```python
import numpy as np

kernel_name = "gpu_gemm_a2a_m4096_k4096_n32768_bf16_silu"


ABSMAX_RTOL = 0.008

COMPUTE_DTYPE = "bfloat16"
EPILOGUE = "silu"
IN_QUANT = None
OUT_QUANT = None
IN_DTYPES = {"x": "bfloat16", "w_mat": "bfloat16"}
WORLD_SIZE = 8

def _to_bf16(x):
    u = np.ascontiguousarray(x, dtype=np.float32).view(np.uint32)
    return ((u + 0x7FFF + ((u >> 16) & 1)) & 0xFFFF0000).view(np.float32)

def setup_inputs():

    x = _to_bf16(np.random.randn(4096, 4096).astype(np.float32))
    w_mat = _to_bf16((np.random.randn(4096, 32768) / np.sqrt(4096)).astype(np.float32))
    return {"x": x, "w_mat": w_mat}

def _epilogue(y):
    return (y / (1.0 + np.exp(-np.clip(y, -60.0, 60.0)))).astype(np.float32)

def reference(x, w_mat):
    y = (x @ w_mat).astype(np.float32)
    return _epilogue(y)
```

```cpp
#include <hip/hip_runtime.h>
#include <rccl/rccl.h>
#include <cstdio>

#define K_MAP 0
#define K_SELF 0
#define K_FIN 2
#define K_PANEL_STRIDE 2359296u
#define K_FLG_OFF 167772160u
#define K_FLG_PANEL 2048u
#define K_ASM \
  "LK_pre:\n" \
  "s_load_dwordx8 s[4:11], %2, 0x0\n" \
  "s_load_dwordx2 s[16:17], %2, 0x20\n" \
  "s_getreg_b32 s25, hwreg(20, 0, 3)\n" \
  "s_lshl_b32 s30, %1, 7\n" \
  "s_add_u32 s30, s30, 134283264\n" \
  "s_andn2_b32 %1, %1, 7\n" \
  "s_or_b32 %1, %1, s25\n" \
  "v_mov_b32 v128, 134316032\n" \
  "v_lshl_add_u32 v128, %1, 7, v128\n" \
  "v_mov_b32 v129, 1\n" \
  "v_mov_b32 v132, 0x1fff0\n" \
  "LK_again:\n" \
  "s_and_b32 s25, %1, 7\n" \
  "s_and_b32 s29, %1, 6\n" \
  "s_lshr_b32 s70, %1, 7\n" \
  "s_or_b32 s29, s29, s70\n" \
  "s_lshl_b32 s70, s29, 4\n" \
  "s_load_dwordx4 s[12:15], %2, s70 offset:0x40\n" \
  "v_lshrrev_b32 v215, 6, %0\n" \
  "v_and_b32 v214, 63, %0\n" \
  "s_lshr_b32 s26, %1, 3\n" \
  "v_readfirstlane_b32 s69, v215\n" \
  "s_getpc_b64 s[78:79]\n" \
  "LK_pc:\n" \
  "v_lshl_add_u32 v215, s26, 2, v215\n" \
  "v_and_b32 v215, 7, v215\n" \
  "v_lshlrev_b32 v216, 6, v214\n" \
  "v_lshl_add_u32 v215, v215, 12, v216\n" \
  "v_min_u32 v215, LK_end-LK_pc, v215\n" \
  "global_load_dword v236, v215, s[78:79]\n" \
  "v_mov_b32 v229, s30\n" \
  "v_lshrrev_b32 v215, 3, v214\n" \
  "v_lshrrev_b32 v216, 4, v214\n" \
  "v_and_b32 v217, 7, v214\n" \
  "v_xor_b32 v216, v216, v217\n" \
  "v_lshlrev_b32 v216, 4, v216\n" \
  "v_lshl_add_u32 v218, v215, 13, v216\n" \
  "s_cmp_lg_u32 s69, 3\n" \
  "s_cbranch_scc1 LK_plain\n" \
  "s_and_b32 s27, s25, 1\n" \
  "s_and_b32 s70, s26, 8\n" \
  "s_lshl4_add_u32 s70, s27, s70\n" \
  "s_lshl_b32 s63, s70, 17\n" \
  "s_and_b32 s70, s26, 7\n" \
  "s_lshl_b32 s70, s70, 16\n" \
  "s_or_b32 s63, s63, s70\n" \
  "s_and_b32 s70, s26, 1\n" \
  "s_lshl_b32 s70, s70, 6\n" \
  "v_xor_b32 v231, s70, v218\n" \
  "v_add_u32 v232, 524288, v231\n" \
  "s_mov_b32 s46, 4194304\n" \
  "s_mov_b32 s47, 0x00020000\n" \
  "s_and_b32 s28, s26, 15\n" \
  "s_mul_i32 s70, s27, 2359296\n" \
  "s_lshl_b32 s71, s28, 10\n" \
  "s_add_u32 s70, s70, s71\n" \
  "v_lshl_add_u32 v233, v214, 4, s70\n" \
  "s_waitcnt lgkmcnt(0)\n" \
  "s_mov_b64 s[44:45], s[4:5]\n" \
  "global_load_dword v220, v229, s[10:11] sc0 sc1\n" \
  "buffer_load_dwordx4 v[160:163], v231, s[44:47], s63 offen offset:0 \n" \
  "buffer_load_dwordx4 v[164:167], v232, s[44:47], s63 offen offset:0 \n" \
  "buffer_load_dwordx4 v[168:171], v231, s[44:47], s63 offen offset:128 \n" \
  "buffer_load_dwordx4 v[172:175], v232, s[44:47], s63 offen offset:128 \n" \
  "buffer_load_dwordx4 v[176:179], v231, s[44:47], s63 offen offset:256 \n" \
  "buffer_load_dwordx4 v[180:183], v232, s[44:47], s63 offen offset:256 \n" \
  "buffer_load_dwordx4 v[184:187], v231, s[44:47], s63 offen offset:384 \n" \
  "buffer_load_dwordx4 v[188:191], v232, s[44:47], s63 offen offset:384 \n" \
  "buffer_load_dwordx4 v[192:195], v231, s[44:47], s63 offen offset:512 \n" \
  "buffer_load_dwordx4 v[196:199], v232, s[44:47], s63 offen offset:512 \n" \
  "buffer_load_dwordx4 v[200:203], v231, s[44:47], s63 offen offset:640 \n" \
  "buffer_load_dwordx4 v[204:207], v232, s[44:47], s63 offen offset:640 \n" \
  "s_add_u32 s63, s63, 768\n" \
  "s_mov_b64 exec, 1\n" \
  "global_atomic_add v130, v128, v129, s[10:11] sc0 sc1\n" \
  "s_mov_b64 exec, -1\n" \
  "s_mul_i32 s71, s27, 2359296\n" \
  "s_lshr_b32 s29, s28, 2\n" \
  "s_mul_i32 s29, s29, 128\n" \
  "s_add_u32 s71, s71, s29\n" \
  "s_and_b32 s29, s28, 3\n" \
  "s_lshl2_add_u32 s71, s29, s71\n" \
  "s_add_u32 s48, s14, s71\n" \
  "s_addc_u32 s49, s15, 0\n" \
  "s_mov_b32 s50, 1540\n" \
  "s_mov_b32 s51, 0x00020000\n" \
  "v_mul_u32_u24 v237, 512, v214\n" \
  "v_mov_b32 v219, 0\n" \
  "s_mov_b32 s82, 0\n" \
  "s_mov_b32 s83, 0\n" \
  "s_lshr_b32 s27, s26, 2\n" \
  "s_and_b32 s70, s25, 1\n" \
  "s_lshl1_add_u32 s29, s27, s70\n" \
  "s_lshr_b32 s70, s25, 1\n" \
  "s_and_b32 s28, s26, 3\n" \
  "s_lshl2_add_u32 s28, s70, s28\n" \
  "s_lshl_b32 s28, s28, 9\n" \
  "v_lshrrev_b32 v215, 3, v214\n" \
  "v_and_b32 v216, 2, v215\n" \
  "s_and_b32 s70, s69, 1\n" \
  "v_or_b32 v216, s70, v216\n" \
  "v_bfe_u32 v217, v214, 1, 2\n" \
  "v_xor_b32 v216, v216, v217\n" \
  "v_lshlrev_b32 v216, 5, v216\n" \
  "v_and_b32 v217, 1, v214\n" \
  "v_lshl_add_u32 v216, v217, 4, v216\n" \
  "v_lshl_add_u32 v216, v215, 16, v216\n" \
  "v_add_u32 v225, 2688, v216\n" \
  "v_subrev_u32 v226, 896, v225\n" \
  "v_subrev_u32 v227, 896, v226\n" \
  "v_subrev_u32 v228, 896, v227\n" \
  "s_add_u32 s40, s6, s28\n" \
  "s_addc_u32 s41, s7, 0\n" \
  "s_sub_u32 s42, s17, s28\n" \
  "s_mov_b32 s43, 0x00020000\n" \
  "s_lshl_b32 s62, s69, 19\n" \
  "s_lshl_b32 s66, s69, 12\n" \
  "s_add_u32 s67, s66, 65536\n" \
  "v_and_b32 v216, 6, v214\n" \
  "v_lshlrev_b32 v216, 6, v216\n" \
  "v_lshl_add_u32 v235, v215, 16, v216\n" \
  "v_add_u32 v235, 2688, v235\n" \
  "s_lshl_b32 s65, s27, 19\n" \
  "s_add_u32 s65, s65, 6291456\n" \
  "s_add_u32 s62, s62, 8388608\n" \
  "v_accvgpr_write_b32 a0, 0\n" \
  "v_accvgpr_write_b32 a1, 0\n" \
  "v_accvgpr_write_b32 a2, 0\n" \
  "v_accvgpr_write_b32 a3, 0\n" \
  "v_accvgpr_write_b32 a4, 0\n" \
  "v_accvgpr_write_b32 a5, 0\n" \
  "v_accvgpr_write_b32 a6, 0\n" \
  "v_accvgpr_write_b32 a7, 0\n" \
  "v_accvgpr_write_b32 a8, 0\n" \
  "v_accvgpr_write_b32 a9, 0\n" \
  "v_accvgpr_write_b32 a10, 0\n" \
  "v_accvgpr_write_b32 a11, 0\n" \
  "v_accvgpr_write_b32 a12, 0\n" \
  "v_accvgpr_write_b32 a13, 0\n" \
  "v_accvgpr_write_b32 a14, 0\n" \
  "v_accvgpr_write_b32 a15, 0\n" \
  "v_accvgpr_write_b32 a16, 0\n" \
  "v_accvgpr_write_b32 a17, 0\n" \
  "v_accvgpr_write_b32 a18, 0\n" \
  "v_accvgpr_write_b32 a19, 0\n" \
  "v_accvgpr_write_b32 a20, 0\n" \
  "v_accvgpr_write_b32 a21, 0\n" \
  "v_accvgpr_write_b32 a22, 0\n" \
  "v_accvgpr_write_b32 a23, 0\n" \
  "v_accvgpr_write_b32 a24, 0\n" \
  "v_accvgpr_write_b32 a25, 0\n" \
  "v_accvgpr_write_b32 a26, 0\n" \
  "v_accvgpr_write_b32 a27, 0\n" \
  "v_accvgpr_write_b32 a28, 0\n" \
  "v_accvgpr_write_b32 a29, 0\n" \
  "v_accvgpr_write_b32 a30, 0\n" \
  "v_accvgpr_write_b32 a31, 0\n" \
  "v_accvgpr_write_b32 a32, 0\n" \
  "v_accvgpr_write_b32 a33, 0\n" \
  "v_accvgpr_write_b32 a34, 0\n" \
  "v_accvgpr_write_b32 a35, 0\n" \
  "v_accvgpr_write_b32 a36, 0\n" \
  "v_accvgpr_write_b32 a37, 0\n" \
  "v_accvgpr_write_b32 a38, 0\n" \
  "v_accvgpr_write_b32 a39, 0\n" \
  "v_accvgpr_write_b32 a40, 0\n" \
  "v_accvgpr_write_b32 a41, 0\n" \
  "v_accvgpr_write_b32 a42, 0\n" \
  "v_accvgpr_write_b32 a43, 0\n" \
  "v_accvgpr_write_b32 a44, 0\n" \
  "v_accvgpr_write_b32 a45, 0\n" \
  "v_accvgpr_write_b32 a46, 0\n" \
  "v_accvgpr_write_b32 a47, 0\n" \
  "v_accvgpr_write_b32 a48, 0\n" \
  "v_accvgpr_write_b32 a49, 0\n" \
  "v_accvgpr_write_b32 a50, 0\n" \
  "v_accvgpr_write_b32 a51, 0\n" \
  "v_accvgpr_write_b32 a52, 0\n" \
  "v_accvgpr_write_b32 a53, 0\n" \
  "v_accvgpr_write_b32 a54, 0\n" \
  "v_accvgpr_write_b32 a55, 0\n" \
  "v_accvgpr_write_b32 a56, 0\n" \
  "v_accvgpr_write_b32 a57, 0\n" \
  "v_accvgpr_write_b32 a58, 0\n" \
  "v_accvgpr_write_b32 a59, 0\n" \
  "v_accvgpr_write_b32 a60, 0\n" \
  "v_accvgpr_write_b32 a61, 0\n" \
  "v_accvgpr_write_b32 a62, 0\n" \
  "v_accvgpr_write_b32 a63, 0\n" \
  "v_accvgpr_write_b32 a64, 0\n" \
  "v_accvgpr_write_b32 a65, 0\n" \
  "v_accvgpr_write_b32 a66, 0\n" \
  "v_accvgpr_write_b32 a67, 0\n" \
  "v_accvgpr_write_b32 a68, 0\n" \
  "v_accvgpr_write_b32 a69, 0\n" \
  "v_accvgpr_write_b32 a70, 0\n" \
  "v_accvgpr_write_b32 a71, 0\n" \
  "v_accvgpr_write_b32 a72, 0\n" \
  "v_accvgpr_write_b32 a73, 0\n" \
  "v_accvgpr_write_b32 a74, 0\n" \
  "v_accvgpr_write_b32 a75, 0\n" \
  "v_accvgpr_write_b32 a76, 0\n" \
  "v_accvgpr_write_b32 a77, 0\n" \
  "v_accvgpr_write_b32 a78, 0\n" \
  "v_accvgpr_write_b32 a79, 0\n" \
  "v_accvgpr_write_b32 a80, 0\n" \
  "v_accvgpr_write_b32 a81, 0\n" \
  "v_accvgpr_write_b32 a82, 0\n" \
  "v_accvgpr_write_b32 a83, 0\n" \
  "v_accvgpr_write_b32 a84, 0\n" \
  "v_accvgpr_write_b32 a85, 0\n" \
  "v_accvgpr_write_b32 a86, 0\n" \
  "v_accvgpr_write_b32 a87, 0\n" \
  "v_accvgpr_write_b32 a88, 0\n" \
  "v_accvgpr_write_b32 a89, 0\n" \
  "v_accvgpr_write_b32 a90, 0\n" \
  "v_accvgpr_write_b32 a91, 0\n" \
  "v_accvgpr_write_b32 a92, 0\n" \
  "v_accvgpr_write_b32 a93, 0\n" \
  "v_accvgpr_write_b32 a94, 0\n" \
  "v_accvgpr_write_b32 a95, 0\n" \
  "v_accvgpr_write_b32 a96, 0\n" \
  "v_accvgpr_write_b32 a97, 0\n" \
  "v_accvgpr_write_b32 a98, 0\n" \
  "v_accvgpr_write_b32 a99, 0\n" \
  "v_accvgpr_write_b32 a100, 0\n" \
  "v_accvgpr_write_b32 a101, 0\n" \
  "v_accvgpr_write_b32 a102, 0\n" \
  "v_accvgpr_write_b32 a103, 0\n" \
  "v_accvgpr_write_b32 a104, 0\n" \
  "v_accvgpr_write_b32 a105, 0\n" \
  "v_accvgpr_write_b32 a106, 0\n" \
  "v_accvgpr_write_b32 a107, 0\n" \
  "v_accvgpr_write_b32 a108, 0\n" \
  "v_accvgpr_write_b32 a109, 0\n" \
  "v_accvgpr_write_b32 a110, 0\n" \
  "v_accvgpr_write_b32 a111, 0\n" \
  "v_accvgpr_write_b32 a112, 0\n" \
  "v_accvgpr_write_b32 a113, 0\n" \
  "v_accvgpr_write_b32 a114, 0\n" \
  "v_accvgpr_write_b32 a115, 0\n" \
  "v_accvgpr_write_b32 a116, 0\n" \
  "v_accvgpr_write_b32 a117, 0\n" \
  "v_accvgpr_write_b32 a118, 0\n" \
  "v_accvgpr_write_b32 a119, 0\n" \
  "v_accvgpr_write_b32 a120, 0\n" \
  "v_accvgpr_write_b32 a121, 0\n" \
  "v_accvgpr_write_b32 a122, 0\n" \
  "v_accvgpr_write_b32 a123, 0\n" \
  "v_accvgpr_write_b32 a124, 0\n" \
  "v_accvgpr_write_b32 a125, 0\n" \
  "v_accvgpr_write_b32 a126, 0\n" \
  "v_accvgpr_write_b32 a127, 0\n" \
  "v_mov_b32 v0, 0\n" \
  "v_mov_b32 v1, 0\n" \
  "v_mov_b32 v2, 0\n" \
  "v_mov_b32 v3, 0\n" \
  "v_mov_b32 v4, 0\n" \
  "v_mov_b32 v5, 0\n" \
  "v_mov_b32 v6, 0\n" \
  "v_mov_b32 v7, 0\n" \
  "v_mov_b32 v8, 0\n" \
  "v_mov_b32 v9, 0\n" \
  "v_mov_b32 v10, 0\n" \
  "v_mov_b32 v11, 0\n" \
  "v_mov_b32 v12, 0\n" \
  "v_mov_b32 v13, 0\n" \
  "v_mov_b32 v14, 0\n" \
  "v_mov_b32 v15, 0\n" \
  "v_mov_b32 v16, 0\n" \
  "v_mov_b32 v17, 0\n" \
  "v_mov_b32 v18, 0\n" \
  "v_mov_b32 v19, 0\n" \
  "v_mov_b32 v20, 0\n" \
  "v_mov_b32 v21, 0\n" \
  "v_mov_b32 v22, 0\n" \
  "v_mov_b32 v23, 0\n" \
  "v_mov_b32 v24, 0\n" \
  "v_mov_b32 v25, 0\n" \
  "v_mov_b32 v26, 0\n" \
  "v_mov_b32 v27, 0\n" \
  "v_mov_b32 v28, 0\n" \
  "v_mov_b32 v29, 0\n" \
  "v_mov_b32 v30, 0\n" \
  "v_mov_b32 v31, 0\n" \
  "v_mov_b32 v32, 0\n" \
  "v_mov_b32 v33, 0\n" \
  "v_mov_b32 v34, 0\n" \
  "v_mov_b32 v35, 0\n" \
  "v_mov_b32 v36, 0\n" \
  "v_mov_b32 v37, 0\n" \
  "v_mov_b32 v38, 0\n" \
  "v_mov_b32 v39, 0\n" \
  "v_mov_b32 v40, 0\n" \
  "v_mov_b32 v41, 0\n" \
  "v_mov_b32 v42, 0\n" \
  "v_mov_b32 v43, 0\n" \
  "v_mov_b32 v44, 0\n" \
  "v_mov_b32 v45, 0\n" \
  "v_mov_b32 v46, 0\n" \
  "v_mov_b32 v47, 0\n" \
  "v_mov_b32 v48, 0\n" \
  "v_mov_b32 v49, 0\n" \
  "v_mov_b32 v50, 0\n" \
  "v_mov_b32 v51, 0\n" \
  "v_mov_b32 v52, 0\n" \
  "v_mov_b32 v53, 0\n" \
  "v_mov_b32 v54, 0\n" \
  "v_mov_b32 v55, 0\n" \
  "v_mov_b32 v56, 0\n" \
  "v_mov_b32 v57, 0\n" \
  "v_mov_b32 v58, 0\n" \
  "v_mov_b32 v59, 0\n" \
  "v_mov_b32 v60, 0\n" \
  "v_mov_b32 v61, 0\n" \
  "v_mov_b32 v62, 0\n" \
  "v_mov_b32 v63, 0\n" \
  "s_waitcnt vmcnt(13) \n" \
  "v_readfirstlane_b32 s72, v220\n" \
  "s_add_u32 s72, s72, 1\n" \
  "s_and_b32 s71, s72, 1\n" \
  "s_lshl_b32 s71, s71, 26\n" \
  "v_add_u32 v233, s71, v233\n" \
  "s_add_u32 s48, s48, s71\n" \
  "s_addc_u32 s49, s49, 0\n" \
  "s_bitcmp1_b32 s25, 0\n" \
  "s_cbranch_scc1 LK_nodly\n" \
  "s_memrealtime s[76:77]\n" \
  "s_mov_b64 exec, 1\n" \
  "global_load_dword v236, v233, s[12:13] sc0 sc1\n" \
  "buffer_load_dword v236, v237, s[48:51], 0 offen sc0 sc1\n" \
  "s_mov_b64 exec, -1\n" \
  "s_waitcnt vmcnt(0) lgkmcnt(0)\n" \
  "s_memrealtime s[76:77]\n" \
  "s_lshr_b32 s70, %1, 1\n" \
  "s_lshl_b32 s70, s70, 15\n" \
  "v_lshlrev_b32 v216, 7, v214\n" \
  "buffer_load_dword v236, v216, s[44:47], s70 offen\n" \
  "s_add_u32 s70, s70, 8192\n" \
  "buffer_load_dword v236, v216, s[44:47], s70 offen\n" \
  "s_add_u32 s70, s70, 8192\n" \
  "buffer_load_dword v236, v216, s[44:47], s70 offen\n" \
  "s_add_u32 s70, s70, 8192\n" \
  "buffer_load_dword v236, v216, s[44:47], s70 offen\n" \
  "s_add_u32 s70, s70, 8192\n" \
  "LK_dly:\n" \
  "s_sleep 1\n" \
  "s_memrealtime s[80:81]\n" \
  "s_waitcnt lgkmcnt(0)\n" \
  "s_sub_u32 s70, s80, s76\n" \
  "s_cmp_lt_u32 s70, 200\n" \
  "s_cbranch_scc1 LK_dly\n" \
  "LK_nodly:\n" \
  "s_waitcnt vmcnt(12) \n" \
  "global_store_dwordx4 v233, v[160:163], s[12:13] sc0 sc1\n" \
  "v_add_u32 v233, 16384, v233\n" \
  "s_waitcnt vmcnt(12) \n" \
  "global_store_dwordx4 v233, v[164:167], s[12:13] sc0 sc1\n" \
  "v_add_u32 v233, 16384, v233\n" \
  "s_barrier\n" \
  "s_lshl_b32 s70, s72, 8\n" \
  "v_mov_b32 v234, s70\n" \
  "s_xor_b32 s24, s27, s16\n" \
  "s_mul_i32 s70, s29, 2359296\n" \
  "s_add_u32 s52, s10, s70\n" \
  "s_addc_u32 s53, s11, 0\n" \
  "s_add_u32 s70, s70, 16384\n" \
  "s_and_b32 s71, s29, 1\n" \
  "s_lshl_b32 s71, s71, 21\n" \
  "s_sub_u32 s38, 4194304, s71\n" \
  "s_cmp_eq_u32 s24, 0\n" \
  "s_cselect_b32 s70, s71, s70\n" \
  "s_cselect_b64 s[36:37], s[4:5], s[10:11]\n" \
  "s_cselect_b32 s38, s38, 2097152\n" \
  "s_add_u32 s36, s36, s70\n" \
  "s_addc_u32 s37, s37, 0\n" \
  "s_mov_b32 s39, 0x00020000\n" \
  "s_add_u32 s20, s10, 134348800\n" \
  "s_addc_u32 s21, s11, 0\n" \
  "s_lshl_b32 s70, s29, 21\n" \
  "s_add_u32 s70, s70, s28\n" \
  "s_add_u32 s18, s8, s70\n" \
  "s_addc_u32 s19, s9, 0\n" \
  "s_cmp_eq_u32 s24, 0\n" \
  "s_cselect_b64 vcc, -1, 0\n" \
  "s_mov_b32 s70, 0\n" \
  "s_cselect_b32 s70, 64512, s70\n" \
  "s_cselect_b32 s71, 64, 0\n" \
  "s_mov_b32 s64, 32768\n" \
  "s_cselect_b32 s64, 128, s64\n" \
  "s_cselect_b32 s74, 0, 1\n" \
  "v_lshlrev_b32 v216, 4, v214\n" \
  "v_cndmask_b32 v221, v216, v218, vcc\n" \
  "v_xor_b32 v222, s71, v221\n" \
  "v_add_u32 v222, s70, v222\n" \
  "v_xor_b32 v223, s71, v222\n" \
  "v_add_u32 v223, s70, v223\n" \
  "v_xor_b32 v224, s71, v223\n" \
  "v_add_u32 v224, s70, v224\n" \
  "s_mul_i32 s60, s69, 4096\n" \
  "s_lshr_b32 s70, s69, 1\n" \
  "s_lshl_b32 s70, s70, 20\n" \
  "s_and_b32 s71, s69, 1\n" \
  "s_lshl_b32 s71, s71, 18\n" \
  "s_add_u32 s70, s70, s71\n" \
  "s_cmp_eq_u32 s24, 0\n" \
  "s_cselect_b32 s60, s70, s60\n" \
  "s_mov_b32 s70, 16384\n" \
  "s_cselect_b32 s70, 524288, s70\n" \
  "s_cselect_b32 s54, 0, 512\n" \
  "s_mov_b32 s55, 0x00020000\n" \
  "s_add_u32 s61, s60, s70\n" \
  "v_and_b32 v230, 3, v214\n" \
  "s_lshl_b32 s70, s69, 2\n" \
  "v_add_u32 v230, s70, v230\n" \
  "v_lshrrev_b32 v216, 2, v230\n" \
  "v_and_b32 v230, 3, v230\n" \
  "v_lshlrev_b32 v230, 2, v230\n" \
  "s_mov_b32 s70, 128\n" \
  "v_mad_u32_u24 v230, v216, s70, v230\n" \
  "s_mul_i32 s70, s69, 128\n" \
  "s_lshr_b32 s25, s25, 1\n" \
  "s_mul_i32 s25, s25, 512\n" \
  "s_add_u32 s70, s70, s25\n" \
  "s_add_u32 s76, s52, s70\n" \
  "s_addc_u32 s77, s53, 0\n" \
  "s_and_b32 s70, s72, 1\n" \
  "s_lshl_b32 s70, s70, 26\n" \
  "s_cmp_eq_u32 s24, 0\n" \
  "s_cselect_b32 s70, 0, s70\n" \
  "s_add_u32 s36, s36, s70\n" \
  "s_addc_u32 s37, s37, 0\n" \
  "s_add_u32 s52, s52, s70\n" \
  "s_addc_u32 s53, s53, 0\n" \
  "s_add_u32 s76, s76, s70\n" \
  "s_addc_u32 s77, s77, 0\n" \
  "s_lshl_b32 s73, s72, 8\n" \
  "s_add_u32 s73, s73, 1\n" \
  "s_mul_i32 s73, s73, s74\n" \
  "v_mov_b32 v229, 0\n" \
  "v_and_b32 v215, 15, v214\n" \
  "v_lshrrev_b32 v216, 1, v215\n" \
  "v_lshrrev_b32 v217, 4, v214\n" \
  "v_xor_b32 v216, v216, v217\n" \
  "v_lshlrev_b32 v216, 4, v216\n" \
  "v_lshl_add_u32 v216, v215, 7, v216\n" \
  "s_lshr_b32 s70, s69, 1\n" \
  "s_lshl_b32 s70, s70, 13\n" \
  "v_add_u32 v208, s70, v216\n" \
  "v_xor_b32 v209, 64, v208\n" \
  "v_lshrrev_b32 v215, 4, v214\n" \
  "v_bfe_u32 v216, v214, 2, 2\n" \
  "v_and_b32 v217, 3, v214\n" \
  "v_lshlrev_b32 v217, 3, v217\n" \
  "v_lshl_add_u32 v217, v216, 7, v217\n" \
  "v_lshl_add_u32 v217, v215, 12, v217\n" \
  "v_and_b32 v216, 2, v216\n" \
  "v_and_b32 v215, 1, v215\n" \
  "v_or_b32 v215, v215, v216\n" \
  "s_and_b32 s70, s69, 1\n" \
  "s_lshl_b32 s70, s70, 11\n" \
  "s_add_u32 s70, s70, 65536\n" \
  "v_add_u32 v217, s70, v217\n" \
  "v_xor_b32 v216, 0, v215\n" \
  "v_lshl_add_u32 v210, v216, 5, v217\n" \
  "v_xor_b32 v216, 1, v215\n" \
  "v_lshl_add_u32 v211, v216, 5, v217\n" \
  "v_xor_b32 v216, 2, v215\n" \
  "v_lshl_add_u32 v212, v216, 5, v217\n" \
  "v_xor_b32 v216, 3, v215\n" \
  "v_lshl_add_u32 v213, v216, 5, v217\n" \
  "s_lshl1_add_u32 s60, s64, s60\n" \
  "s_add_u32 s61, s61, s64\n" \
  "s_mul_i32 s70, s74, 3\n" \
  "s_add_u32 s73, s73, s70\n" \
  "v_mov_b32 v64, 0\n" \
  "v_mov_b32 v65, 0\n" \
  "v_mov_b32 v66, 0\n" \
  "v_mov_b32 v67, 0\n" \
  "v_mov_b32 v68, 0\n" \
  "v_mov_b32 v69, 0\n" \
  "v_mov_b32 v70, 0\n" \
  "v_mov_b32 v71, 0\n" \
  "v_mov_b32 v72, 0\n" \
  "v_mov_b32 v73, 0\n" \
  "v_mov_b32 v74, 0\n" \
  "v_mov_b32 v75, 0\n" \
  "v_mov_b32 v76, 0\n" \
  "v_mov_b32 v77, 0\n" \
  "v_mov_b32 v78, 0\n" \
  "v_mov_b32 v79, 0\n" \
  "v_mov_b32 v80, 0\n" \
  "v_mov_b32 v81, 0\n" \
  "v_mov_b32 v82, 0\n" \
  "v_mov_b32 v83, 0\n" \
  "v_mov_b32 v84, 0\n" \
  "v_mov_b32 v85, 0\n" \
  "v_mov_b32 v86, 0\n" \
  "v_mov_b32 v87, 0\n" \
  "v_mov_b32 v88, 0\n" \
  "v_mov_b32 v89, 0\n" \
  "v_mov_b32 v90, 0\n" \
  "v_mov_b32 v91, 0\n" \
  "v_mov_b32 v92, 0\n" \
  "v_mov_b32 v93, 0\n" \
  "v_mov_b32 v94, 0\n" \
  "v_mov_b32 v95, 0\n" \
  "v_mov_b32 v96, 0\n" \
  "v_mov_b32 v97, 0\n" \
  "v_mov_b32 v98, 0\n" \
  "v_mov_b32 v99, 0\n" \
  "v_mov_b32 v100, 0\n" \
  "v_mov_b32 v101, 0\n" \
  "v_mov_b32 v102, 0\n" \
  "v_mov_b32 v103, 0\n" \
  "v_mov_b32 v104, 0\n" \
  "v_mov_b32 v105, 0\n" \
  "v_mov_b32 v106, 0\n" \
  "v_mov_b32 v107, 0\n" \
  "v_mov_b32 v108, 0\n" \
  "v_mov_b32 v109, 0\n" \
  "v_mov_b32 v110, 0\n" \
  "v_mov_b32 v111, 0\n" \
  "v_mov_b32 v112, 0\n" \
  "v_mov_b32 v113, 0\n" \
  "v_mov_b32 v114, 0\n" \
  "v_mov_b32 v115, 0\n" \
  "v_mov_b32 v116, 0\n" \
  "v_mov_b32 v117, 0\n" \
  "v_mov_b32 v118, 0\n" \
  "v_mov_b32 v119, 0\n" \
  "v_mov_b32 v120, 0\n" \
  "v_mov_b32 v121, 0\n" \
  "v_mov_b32 v122, 0\n" \
  "v_mov_b32 v123, 0\n" \
  "v_mov_b32 v124, 0\n" \
  "v_mov_b32 v125, 0\n" \
  "v_mov_b32 v126, 0\n" \
  "v_mov_b32 v127, 0\n" \
  "s_waitcnt vmcnt(1) \n" \
  "global_store_dwordx4 v233, v[168:171], s[12:13] sc0 sc1\n" \
  "v_add_u32 v233, 16384, v233\n" \
  "v_readfirstlane_b32 s70, v130\n" \
  "s_add_u32 s70, s70, 1\n" \
  "s_xor_b32 s70, s70, s72\n" \
  "v_mov_b32 v131, s70\n" \
  "ds_write_b32 v132, v131 offset:16\n" \
  "s_cbranch_scc1 LK_kbad\n" \
  "s_waitcnt vmcnt(1) \n" \
  "global_store_dwordx4 v233, v[172:175], s[12:13] sc0 sc1\n" \
  "v_add_u32 v233, 16384, v233\n" \
  "v_add_u32 v234, 2, v234\n" \
  "buffer_store_dword v234, v237, s[48:51], 0 offen sc0 sc1\n" \
  "s_waitcnt lgkmcnt(0)\n" \
  "s_barrier\n" \
  "s_waitcnt vmcnt(2) \n" \
  "global_store_dwordx4 v233, v[176:179], s[12:13] sc0 sc1\n" \
  "v_add_u32 v233, 16384, v233\n" \
  "s_waitcnt vmcnt(2) \n" \
  "global_store_dwordx4 v233, v[180:183], s[12:13] sc0 sc1\n" \
  "v_add_u32 v233, 16384, v233\n" \
  "v_add_u32 v234, 2, v234\n" \
  "buffer_store_dword v234, v237, s[48:51], 0 offen sc0 sc1\n" \
  "s_mov_b32 s50, 4\n" \
  "s_barrier\n" \
  "ds_read_b64_tr_b16 a[128:129], v210 offset:0\n" \
  "ds_read_b64_tr_b16 a[130:131], v210 offset:512\n" \
  "ds_read_b64_tr_b16 a[132:133], v211 offset:0\n" \
  "ds_read_b64_tr_b16 a[134:135], v211 offset:512\n" \
  "ds_read_b64_tr_b16 a[136:137], v212 offset:0\n" \
  "ds_read_b64_tr_b16 a[138:139], v212 offset:512\n" \
  "ds_read_b64_tr_b16 a[140:141], v213 offset:0\n" \
  "ds_read_b64_tr_b16 a[142:143], v213 offset:512\n" \
  "ds_read_b64_tr_b16 a[144:145], v210 offset:1024\n" \
  "ds_read_b64_tr_b16 a[146:147], v210 offset:1536\n" \
  "ds_read_b64_tr_b16 a[148:149], v211 offset:1024\n" \
  "ds_read_b64_tr_b16 a[150:151], v211 offset:1536\n" \
  "ds_read_b64_tr_b16 a[152:153], v212 offset:1024\n" \
  "ds_read_b64_tr_b16 a[154:155], v212 offset:1536\n" \
  "ds_read_b64_tr_b16 a[156:157], v213 offset:1024\n" \
  "ds_read_b64_tr_b16 a[158:159], v213 offset:1536\n" \
  "ds_read_b64_tr_b16 a[160:161], v210 offset:16384\n" \
  "ds_read_b64_tr_b16 a[162:163], v210 offset:16896\n" \
  "ds_read_b64_tr_b16 a[164:165], v211 offset:16384\n" \
  "ds_read_b64_tr_b16 a[166:167], v211 offset:16896\n" \
  "ds_read_b64_tr_b16 a[168:169], v212 offset:16384\n" \
  "ds_read_b64_tr_b16 a[170:171], v212 offset:16896\n" \
  "ds_read_b64_tr_b16 a[172:173], v213 offset:16384\n" \
  "ds_read_b64_tr_b16 a[174:175], v213 offset:16896\n" \
  "ds_read_b64_tr_b16 a[176:177], v210 offset:17408\n" \
  "ds_read_b64_tr_b16 a[178:179], v210 offset:17920\n" \
  "ds_read_b64_tr_b16 a[180:181], v211 offset:17408\n" \
  "ds_read_b64_tr_b16 a[182:183], v211 offset:17920\n" \
  "ds_read_b64_tr_b16 a[184:185], v212 offset:17408\n" \
  "ds_read_b64_tr_b16 a[186:187], v212 offset:17920\n" \
  "ds_read_b64_tr_b16 a[188:189], v213 offset:17408\n" \
  "ds_read_b64_tr_b16 a[190:191], v213 offset:17920\n" \
  "s_waitcnt vmcnt(2) \n" \
  "global_store_dwordx4 v233, v[184:187], s[12:13] sc0 sc1\n" \
  "v_add_u32 v233, 16384, v233\n" \
  "s_waitcnt vmcnt(2) \n" \
  "global_store_dwordx4 v233, v[188:191], s[12:13] sc0 sc1\n" \
  "v_add_u32 v233, 16384, v233\n" \
  "v_add_u32 v234, 2, v234\n" \
  "buffer_store_dword v234, v237, s[48:51], 0 offen sc0 sc1\n" \
  "v_cmp_gt_u32 vcc, s73, v229\n" \
  "s_nop 0\n" \
  "s_cbranch_vccnz LK_slowpushp3\n" \
  "LK_retpushp3:\n" \
  "s_add_u32 s73, s73, s74\n" \
  "s_add_u32 s73, s73, s74\n" \
  "s_lshl_b32 s74, s74, 1\n" \
  "s_sub_u32 s76, s76, s25\n" \
  "s_subb_u32 s77, s77, 0\n" \
  "s_barrier\n" \
  "ds_read_b128 v[128:131], v208 offset:0\n" \
  "ds_read_b128 v[144:147], v209 offset:0\n" \
  "ds_read_b128 v[132:135], v208 offset:2048\n" \
  "ds_read_b128 v[148:151], v209 offset:2048\n" \
  "ds_read_b128 v[136:139], v208 offset:4096\n" \
  "ds_read_b128 v[152:155], v209 offset:4096\n" \
  "ds_read_b128 v[140:143], v208 offset:6144\n" \
  "ds_read_b128 v[156:159], v209 offset:6144\n" \
  "s_waitcnt vmcnt(26) lgkmcnt(0)\n" \
  "s_barrier\n" \
  "s_add_u32 m0, s67, 0\n" \
  "s_nop 0\n" \
  "buffer_load_dwordx4 v225, s[40:43], s62 offen offset:0  lds\n" \
  "buffer_load_dwordx4 v226, s[40:43], s62 offen offset:1024  lds\n" \
  "buffer_load_dwordx4 v227, s[40:43], s62 offen offset:2048  lds\n" \
  "buffer_load_dwordx4 v228, s[40:43], s62 offen offset:3072  lds\n" \
  "s_add_u32 s62, s62, 0x200000\n" \
  "buffer_load_dword v236, v235, s[40:43], s65 offen \n" \
  "s_add_u32 s65, s65, 4194304\n" \
  "buffer_load_dword v236, v235, s[40:43], s65 offen \n" \
  "s_add_u32 s65, s65, 4194304\n" \
  "buffer_load_dword v236, v235, s[40:43], s65 offen \n" \
  "s_add_u32 s65, s65, 4194304\n" \
  "buffer_load_dword v236, v235, s[40:43], s65 offen \n" \
  "s_add_u32 s65, s65, 4194304\n" \
  "v_add_u32 v234, 2, v234\n" \
  "v_add_u32 v238, 16384, v233\n" \
  "s_waitcnt vmcnt(9) \n" \
  "s_add_u32 s68, s62, 260046848\n" \
  ".p2align 6\n" \
  "LK_pushloop:\n" \
  "v_mfma_f32_16x16x32_bf16 v[0:3], a[128:131], v[128:131], v[0:3]\n" \
  "buffer_load_dword v229, v230, s[52:55], 0 offen sc0 sc1\n" \
  "v_mfma_f32_16x16x32_bf16 v[0:3], a[160:163], v[144:147], v[0:3]\n" \
  "ds_read_b64_tr_b16 a[192:193], v210 offset:32768\n" \
  "v_mfma_f32_16x16x32_bf16 v[16:19], a[164:167], v[144:147], v[16:19]\n" \
  "buffer_load_dword v236, v235, s[40:43], s65 offen \n" \
  "v_mfma_f32_16x16x32_bf16 v[16:19], a[132:135], v[128:131], v[16:19]\n" \
  "ds_read_b64_tr_b16 a[194:195], v210 offset:33280\n" \
  "v_mfma_f32_16x16x32_bf16 v[20:23], a[132:135], v[132:135], v[20:23]\n" \
  "buffer_store_dword v234, v237, s[48:51], 0 offen sc0 sc1\n" \
  "v_mfma_f32_16x16x32_bf16 v[20:23], a[164:167], v[148:151], v[20:23]\n" \
  "ds_read_b128 v[160:163], v208 offset:16384\n" \
  "v_mfma_f32_16x16x32_bf16 v[4:7], a[160:163], v[148:151], v[4:7]\n" \
  "global_store_dwordx4 v233, v[192:195], s[12:13] sc0 sc1\n" \
  "v_mfma_f32_16x16x32_bf16 v[4:7], a[128:131], v[132:135], v[4:7]\n" \
  "ds_read_b64_tr_b16 a[196:197], v211 offset:32768\n" \
  "v_mfma_f32_16x16x32_bf16 v[8:11], a[128:131], v[136:139], v[8:11]\n" \
  "global_store_dwordx4 v238, v[196:199], s[12:13] sc0 sc1\n" \
  "v_mfma_f32_16x16x32_bf16 v[8:11], a[160:163], v[152:155], v[8:11]\n" \
  "ds_read_b64_tr_b16 a[198:199], v211 offset:33280\n" \
  "v_mfma_f32_16x16x32_bf16 v[24:27], a[164:167], v[152:155], v[24:27]\n" \
  "buffer_load_dwordx4 v[192:195], v231, s[44:47], s63 offen \n" \
  "v_mfma_f32_16x16x32_bf16 v[24:27], a[132:135], v[136:139], v[24:27]\n" \
  "ds_read_b128 v[176:179], v209 offset:16384\n" \
  "v_mfma_f32_16x16x32_bf16 v[28:31], a[132:135], v[140:143], v[28:31]\n" \
  "buffer_load_dwordx4 v[196:199], v232, s[44:47], s63 offen \n" \
  "v_mfma_f32_16x16x32_bf16 v[28:31], a[164:167], v[156:159], v[28:31]\n" \
  "ds_read_b64_tr_b16 a[200:201], v212 offset:32768\n" \
  "v_mfma_f32_16x16x32_bf16 v[12:15], a[160:163], v[156:159], v[12:15]\n" \
  "v_mfma_f32_16x16x32_bf16 v[12:15], a[128:131], v[140:143], v[12:15]\n" \
  "ds_read_b64_tr_b16 a[202:203], v212 offset:33280\n" \
  "v_mfma_f32_16x16x32_bf16 v[44:47], a[136:139], v[140:143], v[44:47]\n" \
  "v_mfma_f32_16x16x32_bf16 v[44:47], a[168:171], v[156:159], v[44:47]\n" \
  "ds_read_b128 v[164:167], v208 offset:18432\n" \
  "v_mfma_f32_16x16x32_bf16 v[60:63], a[172:175], v[156:159], v[60:63]\n" \
  "v_mfma_f32_16x16x32_bf16 v[60:63], a[140:143], v[140:143], v[60:63]\n" \
  "ds_read_b64_tr_b16 a[204:205], v213 offset:32768\n" \
  "v_mfma_f32_16x16x32_bf16 v[56:59], a[140:143], v[136:139], v[56:59]\n" \
  "v_mfma_f32_16x16x32_bf16 v[56:59], a[172:175], v[152:155], v[56:59]\n" \
  "ds_read_b64_tr_b16 a[206:207], v213 offset:33280\n" \
  "v_mfma_f32_16x16x32_bf16 v[40:43], a[168:171], v[152:155], v[40:43]\n" \
  "v_mfma_f32_16x16x32_bf16 v[40:43], a[136:139], v[136:139], v[40:43]\n" \
  "ds_read_b128 v[180:183], v209 offset:18432\n" \
  "v_mfma_f32_16x16x32_bf16 v[36:39], a[136:139], v[132:135], v[36:39]\n" \
  "v_mfma_f32_16x16x32_bf16 v[36:39], a[168:171], v[148:151], v[36:39]\n" \
  "ds_read_b64_tr_b16 a[208:209], v210 offset:33792\n" \
  "v_mfma_f32_16x16x32_bf16 v[52:55], a[172:175], v[148:151], v[52:55]\n" \
  "v_mfma_f32_16x16x32_bf16 v[52:55], a[140:143], v[132:135], v[52:55]\n" \
  "ds_read_b64_tr_b16 a[210:211], v210 offset:34304\n" \
  "v_mfma_f32_16x16x32_bf16 v[48:51], a[140:143], v[128:131], v[48:51]\n" \
  "v_mfma_f32_16x16x32_bf16 v[48:51], a[172:175], v[144:147], v[48:51]\n" \
  "ds_read_b128 v[168:171], v208 offset:20480\n" \
  "v_mfma_f32_16x16x32_bf16 v[32:35], a[168:171], v[144:147], v[32:35]\n" \
  "s_add_u32 m0, s66, 49152\n" \
  "v_mfma_f32_16x16x32_bf16 v[32:35], a[136:139], v[128:131], v[32:35]\n" \
  "ds_read_b64_tr_b16 a[212:213], v211 offset:33792\n" \
  "v_mfma_f32_16x16x32_bf16 v[64:67], a[144:147], v[128:131], v[64:67]\n" \
  "buffer_load_dwordx4 v221, s[36:39], s61 offen offset:0 sc0 sc1 lds\n" \
  "v_mfma_f32_16x16x32_bf16 v[64:67], a[176:179], v[144:147], v[64:67]\n" \
  "ds_read_b64_tr_b16 a[214:215], v211 offset:34304\n" \
  "v_mfma_f32_16x16x32_bf16 v[80:83], a[180:183], v[144:147], v[80:83]\n" \
  "v_mfma_f32_16x16x32_bf16 v[80:83], a[148:151], v[128:131], v[80:83]\n" \
  "ds_read_b128 v[184:187], v209 offset:20480\n" \
  "v_mfma_f32_16x16x32_bf16 v[84:87], a[148:151], v[132:135], v[84:87]\n" \
  "v_mfma_f32_16x16x32_bf16 v[84:87], a[180:183], v[148:151], v[84:87]\n" \
  "ds_read_b64_tr_b16 a[216:217], v212 offset:33792\n" \
  "v_mfma_f32_16x16x32_bf16 v[68:71], a[176:179], v[148:151], v[68:71]\n" \
  "v_mfma_f32_16x16x32_bf16 v[68:71], a[144:147], v[132:135], v[68:71]\n" \
  "ds_read_b64_tr_b16 a[218:219], v212 offset:34304\n" \
  "v_mfma_f32_16x16x32_bf16 v[72:75], a[144:147], v[136:139], v[72:75]\n" \
  "buffer_load_dwordx4 v222, s[36:39], s61 offen offset:1024 sc0 sc1 lds\n" \
  "v_mfma_f32_16x16x32_bf16 v[72:75], a[176:179], v[152:155], v[72:75]\n" \
  "ds_read_b128 v[172:175], v208 offset:22528\n" \
  "v_mfma_f32_16x16x32_bf16 v[88:91], a[180:183], v[152:155], v[88:91]\n" \
  "v_mfma_f32_16x16x32_bf16 v[88:91], a[148:151], v[136:139], v[88:91]\n" \
  "ds_read_b64_tr_b16 a[220:221], v213 offset:33792\n" \
  "v_mfma_f32_16x16x32_bf16 v[92:95], a[148:151], v[140:143], v[92:95]\n" \
  "v_mfma_f32_16x16x32_bf16 v[92:95], a[180:183], v[156:159], v[92:95]\n" \
  "ds_read_b64_tr_b16 a[222:223], v213 offset:34304\n" \
  "v_mfma_f32_16x16x32_bf16 v[76:79], a[176:179], v[156:159], v[76:79]\n" \
  "v_mfma_f32_16x16x32_bf16 v[76:79], a[144:147], v[140:143], v[76:79]\n" \
  "ds_read_b128 v[188:191], v209 offset:22528\n" \
  "v_mfma_f32_16x16x32_bf16 v[108:111], a[152:155], v[140:143], v[108:111]\n" \
  "buffer_load_dwordx4 v223, s[36:39], s61 offen offset:2048 sc0 sc1 lds\n" \
  "v_mfma_f32_16x16x32_bf16 v[108:111], a[184:187], v[156:159], v[108:111]\n" \
  "v_mfma_f32_16x16x32_bf16 v[124:127], a[188:191], v[156:159], v[124:127]\n" \
  "v_mfma_f32_16x16x32_bf16 v[124:127], a[156:159], v[140:143], v[124:127]\n" \
  "v_mfma_f32_16x16x32_bf16 v[120:123], a[156:159], v[136:139], v[120:123]\n" \
  "v_mfma_f32_16x16x32_bf16 v[120:123], a[188:191], v[152:155], v[120:123]\n" \
  "v_mfma_f32_16x16x32_bf16 v[104:107], a[184:187], v[152:155], v[104:107]\n" \
  "v_mfma_f32_16x16x32_bf16 v[104:107], a[152:155], v[136:139], v[104:107]\n" \
  "v_mfma_f32_16x16x32_bf16 v[100:103], a[152:155], v[132:135], v[100:103]\n" \
  "buffer_load_dwordx4 v224, s[36:39], s61 offen offset:3072 sc0 sc1 lds\n" \
  "v_mfma_f32_16x16x32_bf16 v[100:103], a[184:187], v[148:151], v[100:103]\n" \
  "s_add_u32 s61, s61, s64\n" \
  "v_mfma_f32_16x16x32_bf16 v[116:119], a[188:191], v[148:151], v[116:119]\n" \
  "v_mfma_f32_16x16x32_bf16 v[116:119], a[156:159], v[132:135], v[116:119]\n" \
  "v_mfma_f32_16x16x32_bf16 v[112:115], a[156:159], v[128:131], v[112:115]\n" \
  "v_mfma_f32_16x16x32_bf16 v[112:115], a[188:191], v[144:147], v[112:115]\n" \
  "s_waitcnt vmcnt(10) lgkmcnt(0)\n" \
  "v_mfma_f32_16x16x32_bf16 v[96:99], a[184:187], v[144:147], v[96:99]\n" \
  "s_barrier\n" \
  "v_mfma_f32_16x16x32_bf16 v[96:99], a[152:155], v[128:131], v[96:99]\n" \
  "v_mfma_f32_16x16x32_bf16 a[0:3], a[128:131], v[160:163], a[0:3]\n" \
  "v_mfma_f32_16x16x32_bf16 a[0:3], a[160:163], v[176:179], a[0:3]\n" \
  "ds_read_b64_tr_b16 a[224:225], v210 offset:49152\n" \
  "v_mfma_f32_16x16x32_bf16 a[16:19], a[164:167], v[176:179], a[16:19]\n" \
  "v_mfma_f32_16x16x32_bf16 a[16:19], a[132:135], v[160:163], a[16:19]\n" \
  "ds_read_b64_tr_b16 a[226:227], v210 offset:49664\n" \
  "v_mfma_f32_16x16x32_bf16 a[20:23], a[132:135], v[164:167], a[20:23]\n" \
  "v_mfma_f32_16x16x32_bf16 a[20:23], a[164:167], v[180:183], a[20:23]\n" \
  "ds_read_b128 v[128:131], v208 offset:32768\n" \
  "v_mfma_f32_16x16x32_bf16 a[4:7], a[160:163], v[180:183], a[4:7]\n" \
  "v_mfma_f32_16x16x32_bf16 a[4:7], a[128:131], v[164:167], a[4:7]\n" \
  "ds_read_b64_tr_b16 a[228:229], v211 offset:49152\n" \
  "v_mfma_f32_16x16x32_bf16 a[8:11], a[128:131], v[168:171], a[8:11]\n" \
  "v_mfma_f32_16x16x32_bf16 a[8:11], a[160:163], v[184:187], a[8:11]\n" \
  "ds_read_b64_tr_b16 a[230:231], v211 offset:49664\n" \
  "v_mfma_f32_16x16x32_bf16 a[24:27], a[164:167], v[184:187], a[24:27]\n" \
  "s_add_u32 m0, s67, 16384\n" \
  "v_mfma_f32_16x16x32_bf16 a[24:27], a[132:135], v[168:171], a[24:27]\n" \
  "ds_read_b128 v[144:147], v209 offset:32768\n" \
  "v_mfma_f32_16x16x32_bf16 a[28:31], a[132:135], v[172:175], a[28:31]\n" \
  "buffer_load_dwordx4 v225, s[40:43], s62 offen offset:0  lds\n" \
  "v_mfma_f32_16x16x32_bf16 a[28:31], a[164:167], v[188:191], a[28:31]\n" \
  "ds_read_b64_tr_b16 a[232:233], v212 offset:49152\n" \
  "v_mfma_f32_16x16x32_bf16 a[12:15], a[160:163], v[188:191], a[12:15]\n" \
  "v_mfma_f32_16x16x32_bf16 a[12:15], a[128:131], v[172:175], a[12:15]\n" \
  "ds_read_b64_tr_b16 a[234:235], v212 offset:49664\n" \
  "v_mfma_f32_16x16x32_bf16 a[44:47], a[136:139], v[172:175], a[44:47]\n" \
  "buffer_load_dwordx4 v226, s[40:43], s62 offen offset:1024  lds\n" \
  "v_mfma_f32_16x16x32_bf16 a[44:47], a[168:171], v[188:191], a[44:47]\n" \
  "ds_read_b128 v[132:135], v208 offset:34816\n" \
  "v_mfma_f32_16x16x32_bf16 a[60:63], a[172:175], v[188:191], a[60:63]\n" \
  "v_mfma_f32_16x16x32_bf16 a[60:63], a[140:143], v[172:175], a[60:63]\n" \
  "ds_read_b64_tr_b16 a[236:237], v213 offset:49152\n" \
  "v_mfma_f32_16x16x32_bf16 a[56:59], a[140:143], v[168:171], a[56:59]\n" \
  "buffer_load_dwordx4 v227, s[40:43], s62 offen offset:2048  lds\n" \
  "v_mfma_f32_16x16x32_bf16 a[56:59], a[172:175], v[184:187], a[56:59]\n" \
  "ds_read_b64_tr_b16 a[238:239], v213 offset:49664\n" \
  "v_mfma_f32_16x16x32_bf16 a[40:43], a[168:171], v[184:187], a[40:43]\n" \
  "v_mfma_f32_16x16x32_bf16 a[40:43], a[136:139], v[168:171], a[40:43]\n" \
  "ds_read_b128 v[148:151], v209 offset:34816\n" \
  "v_mfma_f32_16x16x32_bf16 a[36:39], a[136:139], v[164:167], a[36:39]\n" \
  "buffer_load_dwordx4 v228, s[40:43], s62 offen offset:3072  lds\n" \
  "v_mfma_f32_16x16x32_bf16 a[36:39], a[168:171], v[180:183], a[36:39]\n" \
  "s_add_u32 s62, s62, 0x200000\n" \
  "v_mfma_f32_16x16x32_bf16 a[52:55], a[172:175], v[180:183], a[52:55]\n" \
  "s_add_u32 m0, s67, 32768\n" \
  "v_mfma_f32_16x16x32_bf16 a[52:55], a[140:143], v[164:167], a[52:55]\n" \
  "ds_read_b64_tr_b16 a[240:241], v210 offset:50176\n" \
  "v_mfma_f32_16x16x32_bf16 a[48:51], a[140:143], v[160:163], a[48:51]\n" \
  "buffer_load_dwordx4 v225, s[40:43], s62 offen offset:0  lds\n" \
  "v_mfma_f32_16x16x32_bf16 a[48:51], a[172:175], v[176:179], a[48:51]\n" \
  "ds_read_b64_tr_b16 a[242:243], v210 offset:50688\n" \
  "v_mfma_f32_16x16x32_bf16 a[32:35], a[168:171], v[176:179], a[32:35]\n" \
  "ds_read_b128 v[136:139], v208 offset:36864\n" \
  "v_mfma_f32_16x16x32_bf16 a[32:35], a[136:139], v[160:163], a[32:35]\n" \
  "ds_read_b64_tr_b16 a[244:245], v211 offset:50176\n" \
  "v_mfma_f32_16x16x32_bf16 a[64:67], a[144:147], v[160:163], a[64:67]\n" \
  "buffer_load_dwordx4 v226, s[40:43], s62 offen offset:1024  lds\n" \
  "v_mfma_f32_16x16x32_bf16 a[64:67], a[176:179], v[176:179], a[64:67]\n" \
  "ds_read_b64_tr_b16 a[246:247], v211 offset:50688\n" \
  "v_mfma_f32_16x16x32_bf16 a[80:83], a[180:183], v[176:179], a[80:83]\n" \
  "v_mfma_f32_16x16x32_bf16 a[80:83], a[148:151], v[160:163], a[80:83]\n" \
  "ds_read_b128 v[152:155], v209 offset:36864\n" \
  "v_mfma_f32_16x16x32_bf16 a[84:87], a[148:151], v[164:167], a[84:87]\n" \
  "v_cmp_gt_u32 vcc, s73, v229\n" \
  "v_mfma_f32_16x16x32_bf16 a[84:87], a[180:183], v[180:183], a[84:87]\n" \
  "buffer_load_dwordx4 v227, s[40:43], s62 offen offset:2048  lds\n" \
  "v_mfma_f32_16x16x32_bf16 a[68:71], a[176:179], v[180:183], a[68:71]\n" \
  "ds_read_b64_tr_b16 a[248:249], v212 offset:50176\n" \
  "v_mfma_f32_16x16x32_bf16 a[68:71], a[144:147], v[164:167], a[68:71]\n" \
  "ds_read_b64_tr_b16 a[250:251], v212 offset:50688\n" \
  "v_mfma_f32_16x16x32_bf16 a[72:75], a[144:147], v[168:171], a[72:75]\n" \
  "s_cbranch_vccnz LK_slowpush1\n" \
  "LK_retpush1:\n" \
  "v_mfma_f32_16x16x32_bf16 a[72:75], a[176:179], v[184:187], a[72:75]\n" \
  "buffer_load_dwordx4 v228, s[40:43], s62 offen offset:3072  lds\n" \
  "v_mfma_f32_16x16x32_bf16 a[88:91], a[180:183], v[184:187], a[88:91]\n" \
  "ds_read_b128 v[140:143], v208 offset:38912\n" \
  "v_mfma_f32_16x16x32_bf16 a[88:91], a[148:151], v[168:171], a[88:91]\n" \
  "ds_read_b64_tr_b16 a[252:253], v213 offset:50176\n" \
  "v_mfma_f32_16x16x32_bf16 a[92:95], a[148:151], v[172:175], a[92:95]\n" \
  "s_add_u32 m0, s66, 0\n" \
  "v_mfma_f32_16x16x32_bf16 a[92:95], a[180:183], v[188:191], a[92:95]\n" \
  "ds_read_b64_tr_b16 a[254:255], v213 offset:50688\n" \
  "v_mfma_f32_16x16x32_bf16 a[76:79], a[176:179], v[188:191], a[76:79]\n" \
  "buffer_load_dwordx4 v221, s[36:39], s60 offen offset:0 sc0 sc1 lds\n" \
  "v_mfma_f32_16x16x32_bf16 a[76:79], a[144:147], v[172:175], a[76:79]\n" \
  "ds_read_b128 v[156:159], v209 offset:38912\n" \
  "v_mfma_f32_16x16x32_bf16 a[108:111], a[152:155], v[172:175], a[108:111]\n" \
  "s_add_u32 s73, s73, s74\n" \
  "v_mfma_f32_16x16x32_bf16 a[108:111], a[184:187], v[188:191], a[108:111]\n" \
  "s_add_u32 s62, s62, 0x200000\n" \
  "v_mfma_f32_16x16x32_bf16 a[124:127], a[188:191], v[188:191], a[124:127]\n" \
  "buffer_load_dwordx4 v222, s[36:39], s60 offen offset:1024 sc0 sc1 lds\n" \
  "v_mfma_f32_16x16x32_bf16 a[124:127], a[156:159], v[172:175], a[124:127]\n" \
  "v_add_u32 v234, 2, v234\n" \
  "v_mfma_f32_16x16x32_bf16 a[120:123], a[156:159], v[168:171], a[120:123]\n" \
  "v_add_u32 v233, 32768, v233\n" \
  "v_mfma_f32_16x16x32_bf16 a[120:123], a[188:191], v[184:187], a[120:123]\n" \
  "v_add_u32 v238, 32768, v238\n" \
  "v_mfma_f32_16x16x32_bf16 a[104:107], a[184:187], v[184:187], a[104:107]\n" \
  "buffer_load_dwordx4 v223, s[36:39], s60 offen offset:2048 sc0 sc1 lds\n" \
  "v_mfma_f32_16x16x32_bf16 a[104:107], a[152:155], v[168:171], a[104:107]\n" \
  "s_add_u32 s63, s63, 128\n" \
  "v_mfma_f32_16x16x32_bf16 a[100:103], a[152:155], v[164:167], a[100:103]\n" \
  "s_add_u32 s65, s65, 4194304\n" \
  "v_mfma_f32_16x16x32_bf16 a[100:103], a[184:187], v[180:183], a[100:103]\n" \
  "v_mfma_f32_16x16x32_bf16 a[116:119], a[188:191], v[180:183], a[116:119]\n" \
  "buffer_load_dwordx4 v224, s[36:39], s60 offen offset:3072 sc0 sc1 lds\n" \
  "v_mfma_f32_16x16x32_bf16 a[116:119], a[156:159], v[164:167], a[116:119]\n" \
  "s_add_u32 s60, s60, s64\n" \
  "v_mfma_f32_16x16x32_bf16 a[112:115], a[156:159], v[160:163], a[112:115]\n" \
  "v_mfma_f32_16x16x32_bf16 a[112:115], a[188:191], v[176:179], a[112:115]\n" \
  "s_waitcnt vmcnt(12) lgkmcnt(0)\n" \
  "v_mfma_f32_16x16x32_bf16 a[96:99], a[184:187], v[176:179], a[96:99]\n" \
  "s_barrier\n" \
  "v_mfma_f32_16x16x32_bf16 a[96:99], a[152:155], v[160:163], a[96:99]\n" \
  "v_mfma_f32_16x16x32_bf16 v[0:3], a[192:195], v[128:131], v[0:3]\n" \
  "buffer_load_dword v229, v230, s[52:55], 0 offen sc0 sc1\n" \
  "v_mfma_f32_16x16x32_bf16 v[0:3], a[224:227], v[144:147], v[0:3]\n" \
  "ds_read_b64_tr_b16 a[128:129], v210 offset:0\n" \
  "v_mfma_f32_16x16x32_bf16 v[16:19], a[228:231], v[144:147], v[16:19]\n" \
  "buffer_load_dword v236, v235, s[40:43], s65 offen \n" \
  "v_mfma_f32_16x16x32_bf16 v[16:19], a[196:199], v[128:131], v[16:19]\n" \
  "ds_read_b64_tr_b16 a[130:131], v210 offset:512\n" \
  "v_mfma_f32_16x16x32_bf16 v[20:23], a[196:199], v[132:135], v[20:23]\n" \
  "buffer_store_dword v234, v237, s[48:51], 0 offen sc0 sc1\n" \
  "v_mfma_f32_16x16x32_bf16 v[20:23], a[228:231], v[148:151], v[20:23]\n" \
  "ds_read_b128 v[160:163], v208 offset:49152\n" \
  "v_mfma_f32_16x16x32_bf16 v[4:7], a[224:227], v[148:151], v[4:7]\n" \
  "global_store_dwordx4 v233, v[200:203], s[12:13] sc0 sc1\n" \
  "v_mfma_f32_16x16x32_bf16 v[4:7], a[192:195], v[132:135], v[4:7]\n" \
  "ds_read_b64_tr_b16 a[132:133], v211 offset:0\n" \
  "v_mfma_f32_16x16x32_bf16 v[8:11], a[192:195], v[136:139], v[8:11]\n" \
  "global_store_dwordx4 v238, v[204:207], s[12:13] sc0 sc1\n" \
  "v_mfma_f32_16x16x32_bf16 v[8:11], a[224:227], v[152:155], v[8:11]\n" \
  "ds_read_b64_tr_b16 a[134:135], v211 offset:512\n" \
  "v_mfma_f32_16x16x32_bf16 v[24:27], a[228:231], v[152:155], v[24:27]\n" \
  "buffer_load_dwordx4 v[200:203], v231, s[44:47], s63 offen \n" \
  "v_mfma_f32_16x16x32_bf16 v[24:27], a[196:199], v[136:139], v[24:27]\n" \
  "ds_read_b128 v[176:179], v209 offset:49152\n" \
  "v_mfma_f32_16x16x32_bf16 v[28:31], a[196:199], v[140:143], v[28:31]\n" \
  "buffer_load_dwordx4 v[204:207], v232, s[44:47], s63 offen \n" \
  "v_mfma_f32_16x16x32_bf16 v[28:31], a[228:231], v[156:159], v[28:31]\n" \
  "ds_read_b64_tr_b16 a[136:137], v212 offset:0\n" \
  "v_mfma_f32_16x16x32_bf16 v[12:15], a[224:227], v[156:159], v[12:15]\n" \
  "v_mfma_f32_16x16x32_bf16 v[12:15], a[192:195], v[140:143], v[12:15]\n" \
  "ds_read_b64_tr_b16 a[138:139], v212 offset:512\n" \
  "v_mfma_f32_16x16x32_bf16 v[44:47], a[200:203], v[140:143], v[44:47]\n" \
  "v_mfma_f32_16x16x32_bf16 v[44:47], a[232:235], v[156:159], v[44:47]\n" \
  "ds_read_b128 v[164:167], v208 offset:51200\n" \
  "v_mfma_f32_16x16x32_bf16 v[60:63], a[236:239], v[156:159], v[60:63]\n" \
  "v_mfma_f32_16x16x32_bf16 v[60:63], a[204:207], v[140:143], v[60:63]\n" \
  "ds_read_b64_tr_b16 a[140:141], v213 offset:0\n" \
  "v_mfma_f32_16x16x32_bf16 v[56:59], a[204:207], v[136:139], v[56:59]\n" \
  "v_mfma_f32_16x16x32_bf16 v[56:59], a[236:239], v[152:155], v[56:59]\n" \
  "ds_read_b64_tr_b16 a[142:143], v213 offset:512\n" \
  "v_mfma_f32_16x16x32_bf16 v[40:43], a[232:235], v[152:155], v[40:43]\n" \
  "v_mfma_f32_16x16x32_bf16 v[40:43], a[200:203], v[136:139], v[40:43]\n" \
  "ds_read_b128 v[180:183], v209 offset:51200\n" \
  "v_mfma_f32_16x16x32_bf16 v[36:39], a[200:203], v[132:135], v[36:39]\n" \
  "v_mfma_f32_16x16x32_bf16 v[36:39], a[232:235], v[148:151], v[36:39]\n" \
  "ds_read_b64_tr_b16 a[144:145], v210 offset:1024\n" \
  "v_mfma_f32_16x16x32_bf16 v[52:55], a[236:239], v[148:151], v[52:55]\n" \
  "v_mfma_f32_16x16x32_bf16 v[52:55], a[204:207], v[132:135], v[52:55]\n" \
  "ds_read_b64_tr_b16 a[146:147], v210 offset:1536\n" \
  "v_mfma_f32_16x16x32_bf16 v[48:51], a[204:207], v[128:131], v[48:51]\n" \
  "v_mfma_f32_16x16x32_bf16 v[48:51], a[236:239], v[144:147], v[48:51]\n" \
  "ds_read_b128 v[168:171], v208 offset:53248\n" \
  "v_mfma_f32_16x16x32_bf16 v[32:35], a[232:235], v[144:147], v[32:35]\n" \
  "s_add_u32 m0, s66, 16384\n" \
  "v_mfma_f32_16x16x32_bf16 v[32:35], a[200:203], v[128:131], v[32:35]\n" \
  "ds_read_b64_tr_b16 a[148:149], v211 offset:1024\n" \
  "v_mfma_f32_16x16x32_bf16 v[64:67], a[208:211], v[128:131], v[64:67]\n" \
  "buffer_load_dwordx4 v221, s[36:39], s61 offen offset:0 sc0 sc1 lds\n" \
  "v_mfma_f32_16x16x32_bf16 v[64:67], a[240:243], v[144:147], v[64:67]\n" \
  "ds_read_b64_tr_b16 a[150:151], v211 offset:1536\n" \
  "v_mfma_f32_16x16x32_bf16 v[80:83], a[244:247], v[144:147], v[80:83]\n" \
  "v_mfma_f32_16x16x32_bf16 v[80:83], a[212:215], v[128:131], v[80:83]\n" \
  "ds_read_b128 v[184:187], v209 offset:53248\n" \
  "v_mfma_f32_16x16x32_bf16 v[84:87], a[212:215], v[132:135], v[84:87]\n" \
  "v_mfma_f32_16x16x32_bf16 v[84:87], a[244:247], v[148:151], v[84:87]\n" \
  "ds_read_b64_tr_b16 a[152:153], v212 offset:1024\n" \
  "v_mfma_f32_16x16x32_bf16 v[68:71], a[240:243], v[148:151], v[68:71]\n" \
  "v_mfma_f32_16x16x32_bf16 v[68:71], a[208:211], v[132:135], v[68:71]\n" \
  "ds_read_b64_tr_b16 a[154:155], v212 offset:1536\n" \
  "v_mfma_f32_16x16x32_bf16 v[72:75], a[208:211], v[136:139], v[72:75]\n" \
  "buffer_load_dwordx4 v222, s[36:39], s61 offen offset:1024 sc0 sc1 lds\n" \
  "v_mfma_f32_16x16x32_bf16 v[72:75], a[240:243], v[152:155], v[72:75]\n" \
  "ds_read_b128 v[172:175], v208 offset:55296\n" \
  "v_mfma_f32_16x16x32_bf16 v[88:91], a[244:247], v[152:155], v[88:91]\n" \
  "v_mfma_f32_16x16x32_bf16 v[88:91], a[212:215], v[136:139], v[88:91]\n" \
  "ds_read_b64_tr_b16 a[156:157], v213 offset:1024\n" \
  "v_mfma_f32_16x16x32_bf16 v[92:95], a[212:215], v[140:143], v[92:95]\n" \
  "v_mfma_f32_16x16x32_bf16 v[92:95], a[244:247], v[156:159], v[92:95]\n" \
  "ds_read_b64_tr_b16 a[158:159], v213 offset:1536\n" \
  "v_mfma_f32_16x16x32_bf16 v[76:79], a[240:243], v[156:159], v[76:79]\n" \
  "v_mfma_f32_16x16x32_bf16 v[76:79], a[208:211], v[140:143], v[76:79]\n" \
  "ds_read_b128 v[188:191], v209 offset:55296\n" \
  "v_mfma_f32_16x16x32_bf16 v[108:111], a[216:219], v[140:143], v[108:111]\n" \
  "buffer_load_dwordx4 v223, s[36:39], s61 offen offset:2048 sc0 sc1 lds\n" \
  "v_mfma_f32_16x16x32_bf16 v[108:111], a[248:251], v[156:159], v[108:111]\n" \
  "v_mfma_f32_16x16x32_bf16 v[124:127], a[252:255], v[156:159], v[124:127]\n" \
  "v_mfma_f32_16x16x32_bf16 v[124:127], a[220:223], v[140:143], v[124:127]\n" \
  "v_mfma_f32_16x16x32_bf16 v[120:123], a[220:223], v[136:139], v[120:123]\n" \
  "v_mfma_f32_16x16x32_bf16 v[120:123], a[252:255], v[152:155], v[120:123]\n" \
  "v_mfma_f32_16x16x32_bf16 v[104:107], a[248:251], v[152:155], v[104:107]\n" \
  "v_mfma_f32_16x16x32_bf16 v[104:107], a[216:219], v[136:139], v[104:107]\n" \
  "v_mfma_f32_16x16x32_bf16 v[100:103], a[216:219], v[132:135], v[100:103]\n" \
  "buffer_load_dwordx4 v224, s[36:39], s61 offen offset:3072 sc0 sc1 lds\n" \
  "v_mfma_f32_16x16x32_bf16 v[100:103], a[248:251], v[148:151], v[100:103]\n" \
  "s_add_u32 s61, s61, s64\n" \
  "v_mfma_f32_16x16x32_bf16 v[116:119], a[252:255], v[148:151], v[116:119]\n" \
  "v_mfma_f32_16x16x32_bf16 v[116:119], a[220:223], v[132:135], v[116:119]\n" \
  "v_mfma_f32_16x16x32_bf16 v[112:115], a[220:223], v[128:131], v[112:115]\n" \
  "v_mfma_f32_16x16x32_bf16 v[112:115], a[252:255], v[144:147], v[112:115]\n" \
  "s_waitcnt vmcnt(10) lgkmcnt(0)\n" \
  "v_mfma_f32_16x16x32_bf16 v[96:99], a[248:251], v[144:147], v[96:99]\n" \
  "s_barrier\n" \
  "v_mfma_f32_16x16x32_bf16 v[96:99], a[216:219], v[128:131], v[96:99]\n" \
  "v_mfma_f32_16x16x32_bf16 a[0:3], a[192:195], v[160:163], a[0:3]\n" \
  "v_mfma_f32_16x16x32_bf16 a[0:3], a[224:227], v[176:179], a[0:3]\n" \
  "ds_read_b64_tr_b16 a[160:161], v210 offset:16384\n" \
  "v_mfma_f32_16x16x32_bf16 a[16:19], a[228:231], v[176:179], a[16:19]\n" \
  "v_mfma_f32_16x16x32_bf16 a[16:19], a[196:199], v[160:163], a[16:19]\n" \
  "ds_read_b64_tr_b16 a[162:163], v210 offset:16896\n" \
  "v_mfma_f32_16x16x32_bf16 a[20:23], a[196:199], v[164:167], a[20:23]\n" \
  "v_mfma_f32_16x16x32_bf16 a[20:23], a[228:231], v[180:183], a[20:23]\n" \
  "ds_read_b128 v[128:131], v208 offset:0\n" \
  "v_mfma_f32_16x16x32_bf16 a[4:7], a[224:227], v[180:183], a[4:7]\n" \
  "v_mfma_f32_16x16x32_bf16 a[4:7], a[192:195], v[164:167], a[4:7]\n" \
  "ds_read_b64_tr_b16 a[164:165], v211 offset:16384\n" \
  "v_mfma_f32_16x16x32_bf16 a[8:11], a[192:195], v[168:171], a[8:11]\n" \
  "v_mfma_f32_16x16x32_bf16 a[8:11], a[224:227], v[184:187], a[8:11]\n" \
  "ds_read_b64_tr_b16 a[166:167], v211 offset:16896\n" \
  "v_mfma_f32_16x16x32_bf16 a[24:27], a[228:231], v[184:187], a[24:27]\n" \
  "s_add_u32 m0, s67, 49152\n" \
  "v_mfma_f32_16x16x32_bf16 a[24:27], a[196:199], v[168:171], a[24:27]\n" \
  "ds_read_b128 v[144:147], v209 offset:0\n" \
  "v_mfma_f32_16x16x32_bf16 a[28:31], a[196:199], v[172:175], a[28:31]\n" \
  "buffer_load_dwordx4 v225, s[40:43], s62 offen offset:0  lds\n" \
  "v_mfma_f32_16x16x32_bf16 a[28:31], a[228:231], v[188:191], a[28:31]\n" \
  "ds_read_b64_tr_b16 a[168:169], v212 offset:16384\n" \
  "v_mfma_f32_16x16x32_bf16 a[12:15], a[224:227], v[188:191], a[12:15]\n" \
  "v_mfma_f32_16x16x32_bf16 a[12:15], a[192:195], v[172:175], a[12:15]\n" \
  "ds_read_b64_tr_b16 a[170:171], v212 offset:16896\n" \
  "v_mfma_f32_16x16x32_bf16 a[44:47], a[200:203], v[172:175], a[44:47]\n" \
  "buffer_load_dwordx4 v226, s[40:43], s62 offen offset:1024  lds\n" \
  "v_mfma_f32_16x16x32_bf16 a[44:47], a[232:235], v[188:191], a[44:47]\n" \
  "ds_read_b128 v[132:135], v208 offset:2048\n" \
  "v_mfma_f32_16x16x32_bf16 a[60:63], a[236:239], v[188:191], a[60:63]\n" \
  "v_mfma_f32_16x16x32_bf16 a[60:63], a[204:207], v[172:175], a[60:63]\n" \
  "ds_read_b64_tr_b16 a[172:173], v213 offset:16384\n" \
  "v_mfma_f32_16x16x32_bf16 a[56:59], a[204:207], v[168:171], a[56:59]\n" \
  "buffer_load_dwordx4 v227, s[40:43], s62 offen offset:2048  lds\n" \
  "v_mfma_f32_16x16x32_bf16 a[56:59], a[236:239], v[184:187], a[56:59]\n" \
  "ds_read_b64_tr_b16 a[174:175], v213 offset:16896\n" \
  "v_mfma_f32_16x16x32_bf16 a[40:43], a[232:235], v[184:187], a[40:43]\n" \
  "v_mfma_f32_16x16x32_bf16 a[40:43], a[200:203], v[168:171], a[40:43]\n" \
  "ds_read_b128 v[148:151], v209 offset:2048\n" \
  "v_mfma_f32_16x16x32_bf16 a[36:39], a[200:203], v[164:167], a[36:39]\n" \
  "buffer_load_dwordx4 v228, s[40:43], s62 offen offset:3072  lds\n" \
  "v_mfma_f32_16x16x32_bf16 a[36:39], a[232:235], v[180:183], a[36:39]\n" \
  "s_add_u32 s62, s62, 0x200000\n" \
  "v_mfma_f32_16x16x32_bf16 a[52:55], a[236:239], v[180:183], a[52:55]\n" \
  "s_add_u32 m0, s67, 0\n" \
  "v_mfma_f32_16x16x32_bf16 a[52:55], a[204:207], v[164:167], a[52:55]\n" \
  "ds_read_b64_tr_b16 a[176:177], v210 offset:17408\n" \
  "v_mfma_f32_16x16x32_bf16 a[48:51], a[204:207], v[160:163], a[48:51]\n" \
  "buffer_load_dwordx4 v225, s[40:43], s62 offen offset:0  lds\n" \
  "v_mfma_f32_16x16x32_bf16 a[48:51], a[236:239], v[176:179], a[48:51]\n" \
  "ds_read_b64_tr_b16 a[178:179], v210 offset:17920\n" \
  "v_mfma_f32_16x16x32_bf16 a[32:35], a[232:235], v[176:179], a[32:35]\n" \
  "ds_read_b128 v[136:139], v208 offset:4096\n" \
  "v_mfma_f32_16x16x32_bf16 a[32:35], a[200:203], v[160:163], a[32:35]\n" \
  "ds_read_b64_tr_b16 a[180:181], v211 offset:17408\n" \
  "v_mfma_f32_16x16x32_bf16 a[64:67], a[208:211], v[160:163], a[64:67]\n" \
  "buffer_load_dwordx4 v226, s[40:43], s62 offen offset:1024  lds\n" \
  "v_mfma_f32_16x16x32_bf16 a[64:67], a[240:243], v[176:179], a[64:67]\n" \
  "ds_read_b64_tr_b16 a[182:183], v211 offset:17920\n" \
  "v_mfma_f32_16x16x32_bf16 a[80:83], a[244:247], v[176:179], a[80:83]\n" \
  "v_mfma_f32_16x16x32_bf16 a[80:83], a[212:215], v[160:163], a[80:83]\n" \
  "ds_read_b128 v[152:155], v209 offset:4096\n" \
  "v_mfma_f32_16x16x32_bf16 a[84:87], a[212:215], v[164:167], a[84:87]\n" \
  "v_cmp_gt_u32 vcc, s73, v229\n" \
  "v_mfma_f32_16x16x32_bf16 a[84:87], a[244:247], v[180:183], a[84:87]\n" \
  "buffer_load_dwordx4 v227, s[40:43], s62 offen offset:2048  lds\n" \
  "v_mfma_f32_16x16x32_bf16 a[68:71], a[240:243], v[180:183], a[68:71]\n" \
  "ds_read_b64_tr_b16 a[184:185], v212 offset:17408\n" \
  "v_mfma_f32_16x16x32_bf16 a[68:71], a[208:211], v[164:167], a[68:71]\n" \
  "ds_read_b64_tr_b16 a[186:187], v212 offset:17920\n" \
  "v_mfma_f32_16x16x32_bf16 a[72:75], a[208:211], v[168:171], a[72:75]\n" \
  "s_cbranch_vccnz LK_slowpush3\n" \
  "LK_retpush3:\n" \
  "v_mfma_f32_16x16x32_bf16 a[72:75], a[240:243], v[184:187], a[72:75]\n" \
  "buffer_load_dwordx4 v228, s[40:43], s62 offen offset:3072  lds\n" \
  "v_mfma_f32_16x16x32_bf16 a[88:91], a[244:247], v[184:187], a[88:91]\n" \
  "ds_read_b128 v[140:143], v208 offset:6144\n" \
  "v_mfma_f32_16x16x32_bf16 a[88:91], a[212:215], v[168:171], a[88:91]\n" \
  "ds_read_b64_tr_b16 a[188:189], v213 offset:17408\n" \
  "v_mfma_f32_16x16x32_bf16 a[92:95], a[212:215], v[172:175], a[92:95]\n" \
  "s_add_u32 m0, s66, 32768\n" \
  "v_mfma_f32_16x16x32_bf16 a[92:95], a[244:247], v[188:191], a[92:95]\n" \
  "ds_read_b64_tr_b16 a[190:191], v213 offset:17920\n" \
  "v_mfma_f32_16x16x32_bf16 a[76:79], a[240:243], v[188:191], a[76:79]\n" \
  "buffer_load_dwordx4 v221, s[36:39], s60 offen offset:0 sc0 sc1 lds\n" \
  "v_mfma_f32_16x16x32_bf16 a[76:79], a[208:211], v[172:175], a[76:79]\n" \
  "ds_read_b128 v[156:159], v209 offset:6144\n" \
  "v_mfma_f32_16x16x32_bf16 a[108:111], a[216:219], v[172:175], a[108:111]\n" \
  "s_add_u32 s73, s73, s74\n" \
  "v_mfma_f32_16x16x32_bf16 a[108:111], a[248:251], v[188:191], a[108:111]\n" \
  "s_add_u32 s62, s62, 0x200000\n" \
  "v_mfma_f32_16x16x32_bf16 a[124:127], a[252:255], v[188:191], a[124:127]\n" \
  "buffer_load_dwordx4 v222, s[36:39], s60 offen offset:1024 sc0 sc1 lds\n" \
  "v_mfma_f32_16x16x32_bf16 a[124:127], a[220:223], v[172:175], a[124:127]\n" \
  "v_add_u32 v234, 2, v234\n" \
  "v_mfma_f32_16x16x32_bf16 a[120:123], a[220:223], v[168:171], a[120:123]\n" \
  "v_add_u32 v233, 32768, v233\n" \
  "v_mfma_f32_16x16x32_bf16 a[120:123], a[252:255], v[184:187], a[120:123]\n" \
  "v_add_u32 v238, 32768, v238\n" \
  "v_mfma_f32_16x16x32_bf16 a[104:107], a[248:251], v[184:187], a[104:107]\n" \
  "buffer_load_dwordx4 v223, s[36:39], s60 offen offset:2048 sc0 sc1 lds\n" \
  "v_mfma_f32_16x16x32_bf16 a[104:107], a[216:219], v[168:171], a[104:107]\n" \
  "s_add_u32 s63, s63, 128\n" \
  "v_mfma_f32_16x16x32_bf16 a[100:103], a[216:219], v[164:167], a[100:103]\n" \
  "s_add_u32 s65, s65, 4194304\n" \
  "v_mfma_f32_16x16x32_bf16 a[100:103], a[248:251], v[180:183], a[100:103]\n" \
  "v_mfma_f32_16x16x32_bf16 a[116:119], a[252:255], v[180:183], a[116:119]\n" \
  "buffer_load_dwordx4 v224, s[36:39], s60 offen offset:3072 sc0 sc1 lds\n" \
  "v_mfma_f32_16x16x32_bf16 a[116:119], a[220:223], v[164:167], a[116:119]\n" \
  "s_add_u32 s60, s60, s64\n" \
  "v_mfma_f32_16x16x32_bf16 a[112:115], a[220:223], v[160:163], a[112:115]\n" \
  "v_mfma_f32_16x16x32_bf16 a[112:115], a[252:255], v[176:179], a[112:115]\n" \
  "s_waitcnt vmcnt(12) lgkmcnt(0)\n" \
  "v_mfma_f32_16x16x32_bf16 a[96:99], a[248:251], v[176:179], a[96:99]\n" \
  "s_barrier\n" \
  "s_cmp_lg_u32 s62, s68\n" \
  "v_mfma_f32_16x16x32_bf16 a[96:99], a[216:219], v[160:163], a[96:99]\n" \
  "s_cbranch_scc1 LK_pushloop\n" \
  "s_branch LK_epi\n" \
  "LK_plain:\n" \
  "v_mov_b32 v219, 0\n" \
  "s_mov_b32 s82, 0\n" \
  "s_mov_b32 s83, 0\n" \
  "s_lshr_b32 s27, s26, 2\n" \
  "s_and_b32 s70, s25, 1\n" \
  "s_lshl1_add_u32 s29, s27, s70\n" \
  "s_lshr_b32 s70, s25, 1\n" \
  "s_and_b32 s28, s26, 3\n" \
  "s_lshl2_add_u32 s28, s70, s28\n" \
  "s_lshl_b32 s28, s28, 9\n" \
  "s_waitcnt lgkmcnt(0)\n" \
  "global_load_dword v220, v229, s[10:11] sc0 sc1\n" \
  "v_lshrrev_b32 v215, 3, v214\n" \
  "v_and_b32 v216, 2, v215\n" \
  "s_and_b32 s70, s69, 1\n" \
  "v_or_b32 v216, s70, v216\n" \
  "v_bfe_u32 v217, v214, 1, 2\n" \
  "v_xor_b32 v216, v216, v217\n" \
  "v_lshlrev_b32 v216, 5, v216\n" \
  "v_and_b32 v217, 1, v214\n" \
  "v_lshl_add_u32 v216, v217, 4, v216\n" \
  "v_lshl_add_u32 v216, v215, 16, v216\n" \
  "v_add_u32 v225, 2688, v216\n" \
  "v_subrev_u32 v226, 896, v225\n" \
  "v_subrev_u32 v227, 896, v226\n" \
  "v_subrev_u32 v228, 896, v227\n" \
  "s_add_u32 s40, s6, s28\n" \
  "s_addc_u32 s41, s7, 0\n" \
  "s_sub_u32 s42, s17, s28\n" \
  "s_mov_b32 s43, 0x00020000\n" \
  "s_lshl_b32 s62, s69, 19\n" \
  "s_lshl_b32 s66, s69, 12\n" \
  "s_add_u32 s67, s66, 65536\n" \
  "v_and_b32 v216, 6, v214\n" \
  "v_lshlrev_b32 v216, 6, v216\n" \
  "v_lshl_add_u32 v235, v215, 16, v216\n" \
  "v_add_u32 v235, 2688, v235\n" \
  "s_lshl_b32 s65, s27, 19\n" \
  "s_add_u32 s65, s65, 6291456\n" \
  "s_cmp_lg_u32 s69, 2\n" \
  "s_cbranch_scc1 LK_xq0\n" \
  "s_lshl_b32 s70, s27, 19\n" \
  "buffer_load_dword v236, v235, s[40:43], s70 offen \n" \
  "s_add_u32 s70, s70, 4194304\n" \
  "buffer_load_dword v236, v235, s[40:43], s70 offen \n" \
  "s_add_u32 s70, s70, 4194304\n" \
  "LK_xq0:\n" \
  "s_cmp_eq_u32 s69, 1\n" \
  "s_cselect_b64 exec, -1, 0\n" \
  "s_add_u32 m0, s67, 0\n" \
  "s_add_u32 m0, m0, 8192\n" \
  "s_add_u32 s70, s62, 1048576\n" \
  "buffer_load_dwordx4 v225, s[40:43], s70 offen offset:0  lds\n" \
  "buffer_load_dwordx4 v226, s[40:43], s70 offen offset:1024  lds\n" \
  "buffer_load_dwordx4 v227, s[40:43], s70 offen offset:2048  lds\n" \
  "buffer_load_dwordx4 v228, s[40:43], s70 offen offset:3072  lds\n" \
  "s_mov_b64 exec, -1\n" \
  "s_add_u32 m0, s67, 0\n" \
  "s_nop 0\n" \
  "buffer_load_dwordx4 v225, s[40:43], s62 offen offset:0  lds\n" \
  "buffer_load_dwordx4 v226, s[40:43], s62 offen offset:1024  lds\n" \
  "buffer_load_dwordx4 v227, s[40:43], s62 offen offset:2048  lds\n" \
  "buffer_load_dwordx4 v228, s[40:43], s62 offen offset:3072  lds\n" \
  "s_add_u32 s62, s62, 0x200000\n" \
  "s_xor_b32 s24, s27, s16\n" \
  "s_mul_i32 s70, s29, 2359296\n" \
  "s_add_u32 s52, s10, s70\n" \
  "s_addc_u32 s53, s11, 0\n" \
  "s_add_u32 s70, s70, 16384\n" \
  "s_and_b32 s71, s29, 1\n" \
  "s_lshl_b32 s71, s71, 21\n" \
  "s_sub_u32 s38, 4194304, s71\n" \
  "s_cmp_eq_u32 s24, 0\n" \
  "s_cselect_b32 s70, s71, s70\n" \
  "s_cselect_b64 s[36:37], s[4:5], s[10:11]\n" \
  "s_cselect_b32 s38, s38, 2097152\n" \
  "s_add_u32 s36, s36, s70\n" \
  "s_addc_u32 s37, s37, 0\n" \
  "s_mov_b32 s39, 0x00020000\n" \
  "s_add_u32 s20, s10, 134348800\n" \
  "s_addc_u32 s21, s11, 0\n" \
  "s_lshl_b32 s70, s29, 21\n" \
  "s_add_u32 s70, s70, s28\n" \
  "s_add_u32 s18, s8, s70\n" \
  "s_addc_u32 s19, s9, 0\n" \
  "s_cmp_eq_u32 s24, 0\n" \
  "s_cselect_b64 vcc, -1, 0\n" \
  "s_mov_b32 s70, 0\n" \
  "s_cselect_b32 s70, 64512, s70\n" \
  "s_cselect_b32 s71, 64, 0\n" \
  "s_mov_b32 s64, 32768\n" \
  "s_cselect_b32 s64, 128, s64\n" \
  "s_cselect_b32 s74, 0, 1\n" \
  "v_lshlrev_b32 v216, 4, v214\n" \
  "v_cndmask_b32 v221, v216, v218, vcc\n" \
  "v_xor_b32 v222, s71, v221\n" \
  "v_add_u32 v222, s70, v222\n" \
  "v_xor_b32 v223, s71, v222\n" \
  "v_add_u32 v223, s70, v223\n" \
  "v_xor_b32 v224, s71, v223\n" \
  "v_add_u32 v224, s70, v224\n" \
  "s_mul_i32 s60, s69, 4096\n" \
  "s_lshr_b32 s70, s69, 1\n" \
  "s_lshl_b32 s70, s70, 20\n" \
  "s_and_b32 s71, s69, 1\n" \
  "s_lshl_b32 s71, s71, 18\n" \
  "s_add_u32 s70, s70, s71\n" \
  "s_cmp_eq_u32 s24, 0\n" \
  "s_cselect_b32 s60, s70, s60\n" \
  "s_mov_b32 s70, 16384\n" \
  "s_cselect_b32 s70, 524288, s70\n" \
  "s_cselect_b32 s54, 0, 512\n" \
  "s_mov_b32 s55, 0x00020000\n" \
  "s_add_u32 s61, s60, s70\n" \
  "v_and_b32 v230, 3, v214\n" \
  "s_lshl_b32 s70, s69, 2\n" \
  "v_add_u32 v230, s70, v230\n" \
  "v_lshrrev_b32 v216, 2, v230\n" \
  "v_and_b32 v230, 3, v230\n" \
  "v_lshlrev_b32 v230, 2, v230\n" \
  "s_mov_b32 s70, 128\n" \
  "v_mad_u32_u24 v230, v216, s70, v230\n" \
  "s_mul_i32 s70, s69, 128\n" \
  "s_lshr_b32 s25, s25, 1\n" \
  "s_mul_i32 s25, s25, 512\n" \
  "s_add_u32 s70, s70, s25\n" \
  "s_add_u32 s76, s52, s70\n" \
  "s_addc_u32 s77, s53, 0\n" \
  "s_cmp_eq_u32 s24, 0\n" \
  "s_mov_b32 s63, 8192\n" \
  "s_cselect_b32 s63, 1048576, s63\n" \
  "s_cmp_eq_u32 s69, 1\n" \
  "s_cselect_b32 s12, 256, 0\n" \
  "v_bfe_u32 v233, v214, 2, 1\n" \
  "v_mad_u32_u24 v233, v233, s12, v230\n" \
  "v_and_b32 v215, 15, v214\n" \
  "v_lshrrev_b32 v216, 1, v215\n" \
  "v_lshrrev_b32 v217, 4, v214\n" \
  "v_xor_b32 v216, v216, v217\n" \
  "v_lshlrev_b32 v216, 4, v216\n" \
  "v_lshl_add_u32 v216, v215, 7, v216\n" \
  "s_lshr_b32 s70, s69, 1\n" \
  "s_lshl_b32 s70, s70, 13\n" \
  "v_add_u32 v208, s70, v216\n" \
  "v_xor_b32 v209, 64, v208\n" \
  "v_lshrrev_b32 v215, 4, v214\n" \
  "v_bfe_u32 v216, v214, 2, 2\n" \
  "v_and_b32 v217, 3, v214\n" \
  "v_lshlrev_b32 v217, 3, v217\n" \
  "v_lshl_add_u32 v217, v216, 7, v217\n" \
  "v_lshl_add_u32 v217, v215, 12, v217\n" \
  "v_and_b32 v216, 2, v216\n" \
  "v_and_b32 v215, 1, v215\n" \
  "v_or_b32 v215, v215, v216\n" \
  "s_and_b32 s70, s69, 1\n" \
  "s_lshl_b32 s70, s70, 11\n" \
  "s_add_u32 s70, s70, 65536\n" \
  "v_add_u32 v217, s70, v217\n" \
  "v_xor_b32 v216, 0, v215\n" \
  "v_lshl_add_u32 v210, v216, 5, v217\n" \
  "v_xor_b32 v216, 1, v215\n" \
  "v_lshl_add_u32 v211, v216, 5, v217\n" \
  "v_xor_b32 v216, 2, v215\n" \
  "v_lshl_add_u32 v212, v216, 5, v217\n" \
  "v_xor_b32 v216, 3, v215\n" \
  "v_lshl_add_u32 v213, v216, 5, v217\n" \
  "v_accvgpr_write_b32 a0, 0\n" \
  "v_accvgpr_write_b32 a1, 0\n" \
  "v_accvgpr_write_b32 a2, 0\n" \
  "v_accvgpr_write_b32 a3, 0\n" \
  "v_accvgpr_write_b32 a4, 0\n" \
  "v_accvgpr_write_b32 a5, 0\n" \
  "v_accvgpr_write_b32 a6, 0\n" \
  "v_accvgpr_write_b32 a7, 0\n" \
  "v_accvgpr_write_b32 a8, 0\n" \
  "v_accvgpr_write_b32 a9, 0\n" \
  "v_accvgpr_write_b32 a10, 0\n" \
  "v_accvgpr_write_b32 a11, 0\n" \
  "v_accvgpr_write_b32 a12, 0\n" \
  "v_accvgpr_write_b32 a13, 0\n" \
  "v_accvgpr_write_b32 a14, 0\n" \
  "v_accvgpr_write_b32 a15, 0\n" \
  "v_accvgpr_write_b32 a16, 0\n" \
  "v_accvgpr_write_b32 a17, 0\n" \
  "v_accvgpr_write_b32 a18, 0\n" \
  "v_accvgpr_write_b32 a19, 0\n" \
  "v_accvgpr_write_b32 a20, 0\n" \
  "v_accvgpr_write_b32 a21, 0\n" \
  "v_accvgpr_write_b32 a22, 0\n" \
  "v_accvgpr_write_b32 a23, 0\n" \
  "v_accvgpr_write_b32 a24, 0\n" \
  "v_accvgpr_write_b32 a25, 0\n" \
  "v_accvgpr_write_b32 a26, 0\n" \
  "v_accvgpr_write_b32 a27, 0\n" \
  "v_accvgpr_write_b32 a28, 0\n" \
  "v_accvgpr_write_b32 a29, 0\n" \
  "v_accvgpr_write_b32 a30, 0\n" \
  "v_accvgpr_write_b32 a31, 0\n" \
  "v_accvgpr_write_b32 a32, 0\n" \
  "v_accvgpr_write_b32 a33, 0\n" \
  "v_accvgpr_write_b32 a34, 0\n" \
  "v_accvgpr_write_b32 a35, 0\n" \
  "v_accvgpr_write_b32 a36, 0\n" \
  "v_accvgpr_write_b32 a37, 0\n" \
  "v_accvgpr_write_b32 a38, 0\n" \
  "v_accvgpr_write_b32 a39, 0\n" \
  "v_accvgpr_write_b32 a40, 0\n" \
  "v_accvgpr_write_b32 a41, 0\n" \
  "v_accvgpr_write_b32 a42, 0\n" \
  "v_accvgpr_write_b32 a43, 0\n" \
  "v_accvgpr_write_b32 a44, 0\n" \
  "v_accvgpr_write_b32 a45, 0\n" \
  "v_accvgpr_write_b32 a46, 0\n" \
  "v_accvgpr_write_b32 a47, 0\n" \
  "v_accvgpr_write_b32 a48, 0\n" \
  "v_accvgpr_write_b32 a49, 0\n" \
  "v_accvgpr_write_b32 a50, 0\n" \
  "v_accvgpr_write_b32 a51, 0\n" \
  "v_accvgpr_write_b32 a52, 0\n" \
  "v_accvgpr_write_b32 a53, 0\n" \
  "v_accvgpr_write_b32 a54, 0\n" \
  "v_accvgpr_write_b32 a55, 0\n" \
  "v_accvgpr_write_b32 a56, 0\n" \
  "v_accvgpr_write_b32 a57, 0\n" \
  "v_accvgpr_write_b32 a58, 0\n" \
  "v_accvgpr_write_b32 a59, 0\n" \
  "v_accvgpr_write_b32 a60, 0\n" \
  "v_accvgpr_write_b32 a61, 0\n" \
  "v_accvgpr_write_b32 a62, 0\n" \
  "v_accvgpr_write_b32 a63, 0\n" \
  "v_accvgpr_write_b32 a64, 0\n" \
  "v_accvgpr_write_b32 a65, 0\n" \
  "v_accvgpr_write_b32 a66, 0\n" \
  "v_accvgpr_write_b32 a67, 0\n" \
  "v_accvgpr_write_b32 a68, 0\n" \
  "v_accvgpr_write_b32 a69, 0\n" \
  "v_accvgpr_write_b32 a70, 0\n" \
  "v_accvgpr_write_b32 a71, 0\n" \
  "v_accvgpr_write_b32 a72, 0\n" \
  "v_accvgpr_write_b32 a73, 0\n" \
  "v_accvgpr_write_b32 a74, 0\n" \
  "v_accvgpr_write_b32 a75, 0\n" \
  "v_accvgpr_write_b32 a76, 0\n" \
  "v_accvgpr_write_b32 a77, 0\n" \
  "v_accvgpr_write_b32 a78, 0\n" \
  "v_accvgpr_write_b32 a79, 0\n" \
  "v_accvgpr_write_b32 a80, 0\n" \
  "v_accvgpr_write_b32 a81, 0\n" \
  "v_accvgpr_write_b32 a82, 0\n" \
  "v_accvgpr_write_b32 a83, 0\n" \
  "v_accvgpr_write_b32 a84, 0\n" \
  "v_accvgpr_write_b32 a85, 0\n" \
  "v_accvgpr_write_b32 a86, 0\n" \
  "v_accvgpr_write_b32 a87, 0\n" \
  "v_accvgpr_write_b32 a88, 0\n" \
  "v_accvgpr_write_b32 a89, 0\n" \
  "v_accvgpr_write_b32 a90, 0\n" \
  "v_accvgpr_write_b32 a91, 0\n" \
  "v_accvgpr_write_b32 a92, 0\n" \
  "v_accvgpr_write_b32 a93, 0\n" \
  "v_accvgpr_write_b32 a94, 0\n" \
  "v_accvgpr_write_b32 a95, 0\n" \
  "v_accvgpr_write_b32 a96, 0\n" \
  "v_accvgpr_write_b32 a97, 0\n" \
  "v_accvgpr_write_b32 a98, 0\n" \
  "v_accvgpr_write_b32 a99, 0\n" \
  "v_accvgpr_write_b32 a100, 0\n" \
  "v_accvgpr_write_b32 a101, 0\n" \
  "v_accvgpr_write_b32 a102, 0\n" \
  "v_accvgpr_write_b32 a103, 0\n" \
  "v_accvgpr_write_b32 a104, 0\n" \
  "v_accvgpr_write_b32 a105, 0\n" \
  "v_accvgpr_write_b32 a106, 0\n" \
  "v_accvgpr_write_b32 a107, 0\n" \
  "v_accvgpr_write_b32 a108, 0\n" \
  "v_accvgpr_write_b32 a109, 0\n" \
  "v_accvgpr_write_b32 a110, 0\n" \
  "v_accvgpr_write_b32 a111, 0\n" \
  "v_accvgpr_write_b32 a112, 0\n" \
  "v_accvgpr_write_b32 a113, 0\n" \
  "v_accvgpr_write_b32 a114, 0\n" \
  "v_accvgpr_write_b32 a115, 0\n" \
  "v_accvgpr_write_b32 a116, 0\n" \
  "v_accvgpr_write_b32 a117, 0\n" \
  "v_accvgpr_write_b32 a118, 0\n" \
  "v_accvgpr_write_b32 a119, 0\n" \
  "v_accvgpr_write_b32 a120, 0\n" \
  "v_accvgpr_write_b32 a121, 0\n" \
  "v_accvgpr_write_b32 a122, 0\n" \
  "v_accvgpr_write_b32 a123, 0\n" \
  "v_accvgpr_write_b32 a124, 0\n" \
  "v_accvgpr_write_b32 a125, 0\n" \
  "v_accvgpr_write_b32 a126, 0\n" \
  "v_accvgpr_write_b32 a127, 0\n" \
  "v_mov_b32 v0, 0\n" \
  "v_mov_b32 v1, 0\n" \
  "v_mov_b32 v2, 0\n" \
  "v_mov_b32 v3, 0\n" \
  "v_mov_b32 v4, 0\n" \
  "v_mov_b32 v5, 0\n" \
  "v_mov_b32 v6, 0\n" \
  "v_mov_b32 v7, 0\n" \
  "v_mov_b32 v8, 0\n" \
  "v_mov_b32 v9, 0\n" \
  "v_mov_b32 v10, 0\n" \
  "v_mov_b32 v11, 0\n" \
  "v_mov_b32 v12, 0\n" \
  "v_mov_b32 v13, 0\n" \
  "v_mov_b32 v14, 0\n" \
  "v_mov_b32 v15, 0\n" \
  "v_mov_b32 v16, 0\n" \
  "v_mov_b32 v17, 0\n" \
  "v_mov_b32 v18, 0\n" \
  "v_mov_b32 v19, 0\n" \
  "v_mov_b32 v20, 0\n" \
  "v_mov_b32 v21, 0\n" \
  "v_mov_b32 v22, 0\n" \
  "v_mov_b32 v23, 0\n" \
  "v_mov_b32 v24, 0\n" \
  "v_mov_b32 v25, 0\n" \
  "v_mov_b32 v26, 0\n" \
  "v_mov_b32 v27, 0\n" \
  "v_mov_b32 v28, 0\n" \
  "v_mov_b32 v29, 0\n" \
  "v_mov_b32 v30, 0\n" \
  "v_mov_b32 v31, 0\n" \
  "v_mov_b32 v32, 0\n" \
  "v_mov_b32 v33, 0\n" \
  "v_mov_b32 v34, 0\n" \
  "v_mov_b32 v35, 0\n" \
  "v_mov_b32 v36, 0\n" \
  "v_mov_b32 v37, 0\n" \
  "v_mov_b32 v38, 0\n" \
  "v_mov_b32 v39, 0\n" \
  "v_mov_b32 v40, 0\n" \
  "v_mov_b32 v41, 0\n" \
  "v_mov_b32 v42, 0\n" \
  "v_mov_b32 v43, 0\n" \
  "v_mov_b32 v44, 0\n" \
  "v_mov_b32 v45, 0\n" \
  "v_mov_b32 v46, 0\n" \
  "v_mov_b32 v47, 0\n" \
  "v_mov_b32 v48, 0\n" \
  "v_mov_b32 v49, 0\n" \
  "v_mov_b32 v50, 0\n" \
  "v_mov_b32 v51, 0\n" \
  "v_mov_b32 v52, 0\n" \
  "v_mov_b32 v53, 0\n" \
  "v_mov_b32 v54, 0\n" \
  "v_mov_b32 v55, 0\n" \
  "v_mov_b32 v56, 0\n" \
  "v_mov_b32 v57, 0\n" \
  "v_mov_b32 v58, 0\n" \
  "v_mov_b32 v59, 0\n" \
  "v_mov_b32 v60, 0\n" \
  "v_mov_b32 v61, 0\n" \
  "v_mov_b32 v62, 0\n" \
  "v_mov_b32 v63, 0\n" \
  "v_mov_b32 v64, 0\n" \
  "v_mov_b32 v65, 0\n" \
  "v_mov_b32 v66, 0\n" \
  "v_mov_b32 v67, 0\n" \
  "v_mov_b32 v68, 0\n" \
  "v_mov_b32 v69, 0\n" \
  "v_mov_b32 v70, 0\n" \
  "v_mov_b32 v71, 0\n" \
  "v_mov_b32 v72, 0\n" \
  "v_mov_b32 v73, 0\n" \
  "v_mov_b32 v74, 0\n" \
  "v_mov_b32 v75, 0\n" \
  "v_mov_b32 v76, 0\n" \
  "v_mov_b32 v77, 0\n" \
  "v_mov_b32 v78, 0\n" \
  "v_mov_b32 v79, 0\n" \
  "v_mov_b32 v80, 0\n" \
  "v_mov_b32 v81, 0\n" \
  "v_mov_b32 v82, 0\n" \
  "v_mov_b32 v83, 0\n" \
  "v_mov_b32 v84, 0\n" \
  "v_mov_b32 v85, 0\n" \
  "v_mov_b32 v86, 0\n" \
  "v_mov_b32 v87, 0\n" \
  "v_mov_b32 v88, 0\n" \
  "v_mov_b32 v89, 0\n" \
  "v_mov_b32 v90, 0\n" \
  "v_mov_b32 v91, 0\n" \
  "v_mov_b32 v92, 0\n" \
  "v_mov_b32 v93, 0\n" \
  "v_mov_b32 v94, 0\n" \
  "v_mov_b32 v95, 0\n" \
  "v_mov_b32 v96, 0\n" \
  "v_mov_b32 v97, 0\n" \
  "v_mov_b32 v98, 0\n" \
  "v_mov_b32 v99, 0\n" \
  "v_mov_b32 v100, 0\n" \
  "v_mov_b32 v101, 0\n" \
  "v_mov_b32 v102, 0\n" \
  "v_mov_b32 v103, 0\n" \
  "v_mov_b32 v104, 0\n" \
  "v_mov_b32 v105, 0\n" \
  "v_mov_b32 v106, 0\n" \
  "v_mov_b32 v107, 0\n" \
  "v_mov_b32 v108, 0\n" \
  "v_mov_b32 v109, 0\n" \
  "v_mov_b32 v110, 0\n" \
  "v_mov_b32 v111, 0\n" \
  "v_mov_b32 v112, 0\n" \
  "v_mov_b32 v113, 0\n" \
  "v_mov_b32 v114, 0\n" \
  "v_mov_b32 v115, 0\n" \
  "v_mov_b32 v116, 0\n" \
  "v_mov_b32 v117, 0\n" \
  "v_mov_b32 v118, 0\n" \
  "v_mov_b32 v119, 0\n" \
  "v_mov_b32 v120, 0\n" \
  "v_mov_b32 v121, 0\n" \
  "v_mov_b32 v122, 0\n" \
  "v_mov_b32 v123, 0\n" \
  "v_mov_b32 v124, 0\n" \
  "v_mov_b32 v125, 0\n" \
  "v_mov_b32 v126, 0\n" \
  "v_mov_b32 v127, 0\n" \
  "s_waitcnt vmcnt(8) \n" \
  "v_readfirstlane_b32 s72, v220\n" \
  "s_add_u32 s72, s72, 1\n" \
  "s_and_b32 s71, s72, 1\n" \
  "s_lshl_b32 s71, s71, 26\n" \
  "s_and_b32 s70, s72, 1\n" \
  "s_lshl_b32 s70, s70, 26\n" \
  "s_cmp_eq_u32 s24, 0\n" \
  "s_cselect_b32 s70, 0, s70\n" \
  "s_add_u32 s36, s36, s70\n" \
  "s_addc_u32 s37, s37, 0\n" \
  "s_add_u32 s52, s52, s70\n" \
  "s_addc_u32 s53, s53, 0\n" \
  "s_add_u32 s76, s76, s70\n" \
  "s_addc_u32 s77, s77, 0\n" \
  "s_lshl_b32 s73, s72, 8\n" \
  "s_add_u32 s73, s73, 1\n" \
  "s_mul_i32 s73, s73, s74\n" \
  "v_mov_b32 v229, 0\n" \
  "buffer_load_dword v236, v230, s[52:55], 0 offen sc0 sc1\n" \
  "s_barrier\n" \
  "s_cmp_eq_u32 s69, 1\n" \
  "s_cselect_b64 exec, -1, 0\n" \
  "s_add_u32 m0, s67, 16384\n" \
  "s_add_u32 m0, m0, 8192\n" \
  "s_add_u32 s70, s62, 1048576\n" \
  "buffer_load_dwordx4 v225, s[40:43], s70 offen offset:0  lds\n" \
  "buffer_load_dwordx4 v226, s[40:43], s70 offen offset:1024  lds\n" \
  "buffer_load_dwordx4 v227, s[40:43], s70 offen offset:2048  lds\n" \
  "buffer_load_dwordx4 v228, s[40:43], s70 offen offset:3072  lds\n" \
  "s_mov_b64 exec, -1\n" \
  "s_add_u32 m0, s67, 16384\n" \
  "s_nop 0\n" \
  "buffer_load_dwordx4 v225, s[40:43], s62 offen offset:0  lds\n" \
  "buffer_load_dwordx4 v226, s[40:43], s62 offen offset:1024  lds\n" \
  "buffer_load_dwordx4 v227, s[40:43], s62 offen offset:2048  lds\n" \
  "buffer_load_dwordx4 v228, s[40:43], s62 offen offset:3072  lds\n" \
  "s_add_u32 s62, s62, 0x200000\n" \
  "s_cmp_eq_u32 s69, 1\n" \
  "s_cselect_b64 exec, -1, 0\n" \
  "s_add_u32 m0, s67, 32768\n" \
  "s_add_u32 m0, m0, 8192\n" \
  "s_add_u32 s70, s62, 1048576\n" \
  "buffer_load_dwordx4 v225, s[40:43], s70 offen offset:0  lds\n" \
  "buffer_load_dwordx4 v226, s[40:43], s70 offen offset:1024  lds\n" \
  "buffer_load_dwordx4 v227, s[40:43], s70 offen offset:2048  lds\n" \
  "buffer_load_dwordx4 v228, s[40:43], s70 offen offset:3072  lds\n" \
  "s_mov_b64 exec, -1\n" \
  "s_add_u32 m0, s67, 32768\n" \
  "s_nop 0\n" \
  "buffer_load_dwordx4 v225, s[40:43], s62 offen offset:0  lds\n" \
  "buffer_load_dwordx4 v226, s[40:43], s62 offen offset:1024  lds\n" \
  "buffer_load_dwordx4 v227, s[40:43], s62 offen offset:2048  lds\n" \
  "buffer_load_dwordx4 v228, s[40:43], s62 offen offset:3072  lds\n" \
  "s_add_u32 s62, s62, 0x200000\n" \
  "s_barrier\n" \
  "ds_read_b32 v131, v132 offset:16\n" \
  "s_cmp_eq_u32 s69, 1\n" \
  "s_cselect_b64 exec, -1, 0\n" \
  "s_add_u32 m0, s67, 49152\n" \
  "s_add_u32 m0, m0, 8192\n" \
  "s_add_u32 s70, s62, 1048576\n" \
  "buffer_load_dwordx4 v225, s[40:43], s70 offen offset:0  lds\n" \
  "buffer_load_dwordx4 v226, s[40:43], s70 offen offset:1024  lds\n" \
  "buffer_load_dwordx4 v227, s[40:43], s70 offen offset:2048  lds\n" \
  "buffer_load_dwordx4 v228, s[40:43], s70 offen offset:3072  lds\n" \
  "s_mov_b64 exec, -1\n" \
  "s_add_u32 m0, s67, 49152\n" \
  "s_nop 0\n" \
  "buffer_load_dwordx4 v225, s[40:43], s62 offen offset:0  lds\n" \
  "buffer_load_dwordx4 v226, s[40:43], s62 offen offset:1024  lds\n" \
  "buffer_load_dwordx4 v227, s[40:43], s62 offen offset:2048  lds\n" \
  "buffer_load_dwordx4 v228, s[40:43], s62 offen offset:3072  lds\n" \
  "s_add_u32 s62, s62, 0x200000\n" \
  "s_waitcnt lgkmcnt(0)\n" \
  "v_readfirstlane_b32 s70, v131\n" \
  "s_cmp_lg_u32 s70, 0\n" \
  "s_cbranch_scc1 LK_redo\n" \
  "s_waitcnt vmcnt(16) \n" \
  "s_barrier\n" \
  "ds_read_b64_tr_b16 a[128:129], v210 offset:0\n" \
  "ds_read_b64_tr_b16 a[130:131], v210 offset:512\n" \
  "ds_read_b64_tr_b16 a[132:133], v211 offset:0\n" \
  "ds_read_b64_tr_b16 a[134:135], v211 offset:512\n" \
  "ds_read_b64_tr_b16 a[136:137], v212 offset:0\n" \
  "ds_read_b64_tr_b16 a[138:139], v212 offset:512\n" \
  "ds_read_b64_tr_b16 a[140:141], v213 offset:0\n" \
  "ds_read_b64_tr_b16 a[142:143], v213 offset:512\n" \
  "ds_read_b64_tr_b16 a[144:145], v210 offset:1024\n" \
  "ds_read_b64_tr_b16 a[146:147], v210 offset:1536\n" \
  "ds_read_b64_tr_b16 a[148:149], v211 offset:1024\n" \
  "ds_read_b64_tr_b16 a[150:151], v211 offset:1536\n" \
  "ds_read_b64_tr_b16 a[152:153], v212 offset:1024\n" \
  "ds_read_b64_tr_b16 a[154:155], v212 offset:1536\n" \
  "ds_read_b64_tr_b16 a[156:157], v213 offset:1024\n" \
  "ds_read_b64_tr_b16 a[158:159], v213 offset:1536\n" \
  "ds_read_b64_tr_b16 a[160:161], v210 offset:16384\n" \
  "ds_read_b64_tr_b16 a[162:163], v210 offset:16896\n" \
  "ds_read_b64_tr_b16 a[164:165], v211 offset:16384\n" \
  "ds_read_b64_tr_b16 a[166:167], v211 offset:16896\n" \
  "ds_read_b64_tr_b16 a[168:169], v212 offset:16384\n" \
  "ds_read_b64_tr_b16 a[170:171], v212 offset:16896\n" \
  "ds_read_b64_tr_b16 a[172:173], v213 offset:16384\n" \
  "ds_read_b64_tr_b16 a[174:175], v213 offset:16896\n" \
  "ds_read_b64_tr_b16 a[176:177], v210 offset:17408\n" \
  "ds_read_b64_tr_b16 a[178:179], v210 offset:17920\n" \
  "ds_read_b64_tr_b16 a[180:181], v211 offset:17408\n" \
  "ds_read_b64_tr_b16 a[182:183], v211 offset:17920\n" \
  "ds_read_b64_tr_b16 a[184:185], v212 offset:17408\n" \
  "ds_read_b64_tr_b16 a[186:187], v212 offset:17920\n" \
  "ds_read_b64_tr_b16 a[188:189], v213 offset:17408\n" \
  "ds_read_b64_tr_b16 a[190:191], v213 offset:17920\n" \
  "v_cmp_gt_u32 vcc, s73, v229\n" \
  "s_nop 0\n" \
  "s_cbranch_vccnz LK_slowplainp0\n" \
  "LK_retplainp0:\n" \
  "s_add_u32 s73, s73, s74\n" \
  "s_cmp_eq_u32 s69, 1\n" \
  "s_cselect_b64 exec, -1, 0\n" \
  "s_add_u32 m0, s66, 0\n" \
  "s_add_u32 m0, m0, 8192\n" \
  "s_add_u32 s70, s60, s63\n" \
  "buffer_load_dwordx4 v221, s[36:39], s70 offen offset:0 sc0 sc1 lds\n" \
  "buffer_load_dwordx4 v222, s[36:39], s70 offen offset:1024 sc0 sc1 lds\n" \
  "buffer_load_dwordx4 v223, s[36:39], s70 offen offset:2048 sc0 sc1 lds\n" \
  "buffer_load_dwordx4 v224, s[36:39], s70 offen offset:3072 sc0 sc1 lds\n" \
  "s_mov_b64 exec, -1\n" \
  "s_add_u32 m0, s66, 0\n" \
  "s_nop 0\n" \
  "buffer_load_dwordx4 v221, s[36:39], s60 offen offset:0 sc0 sc1 lds\n" \
  "buffer_load_dwordx4 v222, s[36:39], s60 offen offset:1024 sc0 sc1 lds\n" \
  "buffer_load_dwordx4 v223, s[36:39], s60 offen offset:2048 sc0 sc1 lds\n" \
  "buffer_load_dwordx4 v224, s[36:39], s60 offen offset:3072 sc0 sc1 lds\n" \
  "s_add_u32 s60, s60, s64\n" \
  "v_cmp_gt_u32 vcc, s73, v229\n" \
  "s_nop 0\n" \
  "s_cbranch_vccnz LK_slowplainp1\n" \
  "LK_retplainp1:\n" \
  "s_add_u32 s73, s73, s74\n" \
  "s_cmp_eq_u32 s69, 1\n" \
  "s_cselect_b64 exec, -1, 0\n" \
  "s_add_u32 m0, s66, 16384\n" \
  "s_add_u32 m0, m0, 8192\n" \
  "s_add_u32 s70, s61, s63\n" \
  "buffer_load_dwordx4 v221, s[36:39], s70 offen offset:0 sc0 sc1 lds\n" \
  "buffer_load_dwordx4 v222, s[36:39], s70 offen offset:1024 sc0 sc1 lds\n" \
  "buffer_load_dwordx4 v223, s[36:39], s70 offen offset:2048 sc0 sc1 lds\n" \
  "buffer_load_dwordx4 v224, s[36:39], s70 offen offset:3072 sc0 sc1 lds\n" \
  "s_mov_b64 exec, -1\n" \
  "s_add_u32 m0, s66, 16384\n" \
  "s_nop 0\n" \
  "buffer_load_dwordx4 v221, s[36:39], s61 offen offset:0 sc0 sc1 lds\n" \
  "buffer_load_dwordx4 v222, s[36:39], s61 offen offset:1024 sc0 sc1 lds\n" \
  "buffer_load_dwordx4 v223, s[36:39], s61 offen offset:2048 sc0 sc1 lds\n" \
  "buffer_load_dwordx4 v224, s[36:39], s61 offen offset:3072 sc0 sc1 lds\n" \
  "s_add_u32 s61, s61, s64\n" \
  "v_cmp_gt_u32 vcc, s73, v229\n" \
  "s_nop 0\n" \
  "s_cbranch_vccnz LK_slowplainp2\n" \
  "LK_retplainp2:\n" \
  "s_add_u32 s73, s73, s74\n" \
  "s_cmp_eq_u32 s69, 1\n" \
  "s_cselect_b64 exec, -1, 0\n" \
  "s_add_u32 m0, s66, 32768\n" \
  "s_add_u32 m0, m0, 8192\n" \
  "s_add_u32 s70, s60, s63\n" \
  "buffer_load_dwordx4 v221, s[36:39], s70 offen offset:0 sc0 sc1 lds\n" \
  "buffer_load_dwordx4 v222, s[36:39], s70 offen offset:1024 sc0 sc1 lds\n" \
  "buffer_load_dwordx4 v223, s[36:39], s70 offen offset:2048 sc0 sc1 lds\n" \
  "buffer_load_dwordx4 v224, s[36:39], s70 offen offset:3072 sc0 sc1 lds\n" \
  "s_mov_b64 exec, -1\n" \
  "s_add_u32 m0, s66, 32768\n" \
  "s_nop 0\n" \
  "buffer_load_dwordx4 v221, s[36:39], s60 offen offset:0 sc0 sc1 lds\n" \
  "buffer_load_dwordx4 v222, s[36:39], s60 offen offset:1024 sc0 sc1 lds\n" \
  "buffer_load_dwordx4 v223, s[36:39], s60 offen offset:2048 sc0 sc1 lds\n" \
  "buffer_load_dwordx4 v224, s[36:39], s60 offen offset:3072 sc0 sc1 lds\n" \
  "s_add_u32 s60, s60, s64\n" \
  "s_waitcnt vmcnt(16) \n" \
  "s_barrier\n" \
  "ds_read_b128 v[128:131], v208 offset:0\n" \
  "ds_read_b128 v[144:147], v209 offset:0\n" \
  "ds_read_b128 v[132:135], v208 offset:2048\n" \
  "ds_read_b128 v[148:151], v209 offset:2048\n" \
  "ds_read_b128 v[136:139], v208 offset:4096\n" \
  "ds_read_b128 v[152:155], v209 offset:4096\n" \
  "ds_read_b128 v[140:143], v208 offset:6144\n" \
  "ds_read_b128 v[156:159], v209 offset:6144\n" \
  "s_waitcnt vmcnt(8) lgkmcnt(0)\n" \
  "v_cmp_gt_u32 vcc, s73, v229\n" \
  "s_nop 0\n" \
  "s_cbranch_vccnz LK_slowplainp3\n" \
  "LK_retplainp3:\n" \
  "s_add_u32 s73, s73, s74\n" \
  "s_add_u32 s73, s73, s74\n" \
  "s_lshl_b32 s74, s74, 1\n" \
  "s_sub_u32 s76, s76, s25\n" \
  "s_subb_u32 s77, s77, 0\n" \
  "s_barrier\n" \
  "s_add_u32 m0, s67, 0\n" \
  "s_nop 0\n" \
  "buffer_load_dwordx4 v225, s[40:43], s62 offen offset:0  lds\n" \
  "buffer_load_dwordx4 v226, s[40:43], s62 offen offset:1024  lds\n" \
  "buffer_load_dwordx4 v227, s[40:43], s62 offen offset:2048  lds\n" \
  "buffer_load_dwordx4 v228, s[40:43], s62 offen offset:3072  lds\n" \
  "s_add_u32 s62, s62, 0x200000\n" \
  "s_add_u32 s68, s62, 260046848\n" \
  ".p2align 6\n" \
  "LK_plainloop:\n" \
  "v_mfma_f32_16x16x32_bf16 v[0:3], a[128:131], v[128:131], v[0:3]\n" \
  "buffer_load_dword v229, v230, s[52:55], 0 offen sc0 sc1\n" \
  "v_mfma_f32_16x16x32_bf16 v[0:3], a[160:163], v[144:147], v[0:3]\n" \
  "ds_read_b64_tr_b16 a[192:193], v210 offset:32768\n" \
  "v_mfma_f32_16x16x32_bf16 v[16:19], a[164:167], v[144:147], v[16:19]\n" \
  "v_mfma_f32_16x16x32_bf16 v[16:19], a[132:135], v[128:131], v[16:19]\n" \
  "ds_read_b64_tr_b16 a[194:195], v210 offset:33280\n" \
  "v_mfma_f32_16x16x32_bf16 v[20:23], a[132:135], v[132:135], v[20:23]\n" \
  "v_mfma_f32_16x16x32_bf16 v[20:23], a[164:167], v[148:151], v[20:23]\n" \
  "ds_read_b128 v[160:163], v208 offset:16384\n" \
  "v_mfma_f32_16x16x32_bf16 v[4:7], a[160:163], v[148:151], v[4:7]\n" \
  "v_mfma_f32_16x16x32_bf16 v[4:7], a[128:131], v[132:135], v[4:7]\n" \
  "ds_read_b64_tr_b16 a[196:197], v211 offset:32768\n" \
  "v_mfma_f32_16x16x32_bf16 v[8:11], a[128:131], v[136:139], v[8:11]\n" \
  "v_mfma_f32_16x16x32_bf16 v[8:11], a[160:163], v[152:155], v[8:11]\n" \
  "ds_read_b64_tr_b16 a[198:199], v211 offset:33280\n" \
  "v_mfma_f32_16x16x32_bf16 v[24:27], a[164:167], v[152:155], v[24:27]\n" \
  "v_mfma_f32_16x16x32_bf16 v[24:27], a[132:135], v[136:139], v[24:27]\n" \
  "ds_read_b128 v[176:179], v209 offset:16384\n" \
  "v_mfma_f32_16x16x32_bf16 v[28:31], a[132:135], v[140:143], v[28:31]\n" \
  "v_mfma_f32_16x16x32_bf16 v[28:31], a[164:167], v[156:159], v[28:31]\n" \
  "ds_read_b64_tr_b16 a[200:201], v212 offset:32768\n" \
  "v_mfma_f32_16x16x32_bf16 v[12:15], a[160:163], v[156:159], v[12:15]\n" \
  "v_mfma_f32_16x16x32_bf16 v[12:15], a[128:131], v[140:143], v[12:15]\n" \
  "ds_read_b64_tr_b16 a[202:203], v212 offset:33280\n" \
  "v_mfma_f32_16x16x32_bf16 v[44:47], a[136:139], v[140:143], v[44:47]\n" \
  "v_mfma_f32_16x16x32_bf16 v[44:47], a[168:171], v[156:159], v[44:47]\n" \
  "ds_read_b128 v[164:167], v208 offset:18432\n" \
  "v_mfma_f32_16x16x32_bf16 v[60:63], a[172:175], v[156:159], v[60:63]\n" \
  "v_mfma_f32_16x16x32_bf16 v[60:63], a[140:143], v[140:143], v[60:63]\n" \
  "ds_read_b64_tr_b16 a[204:205], v213 offset:32768\n" \
  "v_mfma_f32_16x16x32_bf16 v[56:59], a[140:143], v[136:139], v[56:59]\n" \
  "v_mfma_f32_16x16x32_bf16 v[56:59], a[172:175], v[152:155], v[56:59]\n" \
  "ds_read_b64_tr_b16 a[206:207], v213 offset:33280\n" \
  "v_mfma_f32_16x16x32_bf16 v[40:43], a[168:171], v[152:155], v[40:43]\n" \
  "v_mfma_f32_16x16x32_bf16 v[40:43], a[136:139], v[136:139], v[40:43]\n" \
  "ds_read_b128 v[180:183], v209 offset:18432\n" \
  "v_mfma_f32_16x16x32_bf16 v[36:39], a[136:139], v[132:135], v[36:39]\n" \
  "v_mfma_f32_16x16x32_bf16 v[36:39], a[168:171], v[148:151], v[36:39]\n" \
  "ds_read_b64_tr_b16 a[208:209], v210 offset:33792\n" \
  "v_mfma_f32_16x16x32_bf16 v[52:55], a[172:175], v[148:151], v[52:55]\n" \
  "v_mfma_f32_16x16x32_bf16 v[52:55], a[140:143], v[132:135], v[52:55]\n" \
  "ds_read_b64_tr_b16 a[210:211], v210 offset:34304\n" \
  "v_mfma_f32_16x16x32_bf16 v[48:51], a[140:143], v[128:131], v[48:51]\n" \
  "s_add_u32 m0, s66, 49152\n" \
  "v_mfma_f32_16x16x32_bf16 v[48:51], a[172:175], v[144:147], v[48:51]\n" \
  "ds_read_b128 v[168:171], v208 offset:20480\n" \
  "v_mfma_f32_16x16x32_bf16 v[32:35], a[168:171], v[144:147], v[32:35]\n" \
  "buffer_load_dwordx4 v221, s[36:39], s61 offen offset:0 sc0 sc1 lds\n" \
  "v_mfma_f32_16x16x32_bf16 v[32:35], a[136:139], v[128:131], v[32:35]\n" \
  "ds_read_b64_tr_b16 a[212:213], v211 offset:33792\n" \
  "v_mfma_f32_16x16x32_bf16 v[64:67], a[144:147], v[128:131], v[64:67]\n" \
  "v_mfma_f32_16x16x32_bf16 v[64:67], a[176:179], v[144:147], v[64:67]\n" \
  "ds_read_b64_tr_b16 a[214:215], v211 offset:34304\n" \
  "v_mfma_f32_16x16x32_bf16 v[80:83], a[180:183], v[144:147], v[80:83]\n" \
  "v_mfma_f32_16x16x32_bf16 v[80:83], a[148:151], v[128:131], v[80:83]\n" \
  "ds_read_b128 v[184:187], v209 offset:20480\n" \
  "v_mfma_f32_16x16x32_bf16 v[84:87], a[148:151], v[132:135], v[84:87]\n" \
  "v_mfma_f32_16x16x32_bf16 v[84:87], a[180:183], v[148:151], v[84:87]\n" \
  "ds_read_b64_tr_b16 a[216:217], v212 offset:33792\n" \
  "v_mfma_f32_16x16x32_bf16 v[68:71], a[176:179], v[148:151], v[68:71]\n" \
  "buffer_load_dwordx4 v222, s[36:39], s61 offen offset:1024 sc0 sc1 lds\n" \
  "v_mfma_f32_16x16x32_bf16 v[68:71], a[144:147], v[132:135], v[68:71]\n" \
  "ds_read_b64_tr_b16 a[218:219], v212 offset:34304\n" \
  "v_mfma_f32_16x16x32_bf16 v[72:75], a[144:147], v[136:139], v[72:75]\n" \
  "v_mfma_f32_16x16x32_bf16 v[72:75], a[176:179], v[152:155], v[72:75]\n" \
  "ds_read_b128 v[172:175], v208 offset:22528\n" \
  "v_mfma_f32_16x16x32_bf16 v[88:91], a[180:183], v[152:155], v[88:91]\n" \
  "v_mfma_f32_16x16x32_bf16 v[88:91], a[148:151], v[136:139], v[88:91]\n" \
  "ds_read_b64_tr_b16 a[220:221], v213 offset:33792\n" \
  "v_mfma_f32_16x16x32_bf16 v[92:95], a[148:151], v[140:143], v[92:95]\n" \
  "v_mfma_f32_16x16x32_bf16 v[92:95], a[180:183], v[156:159], v[92:95]\n" \
  "ds_read_b64_tr_b16 a[222:223], v213 offset:34304\n" \
  "v_mfma_f32_16x16x32_bf16 v[76:79], a[176:179], v[156:159], v[76:79]\n" \
  "buffer_load_dwordx4 v223, s[36:39], s61 offen offset:2048 sc0 sc1 lds\n" \
  "v_mfma_f32_16x16x32_bf16 v[76:79], a[144:147], v[140:143], v[76:79]\n" \
  "ds_read_b128 v[188:191], v209 offset:22528\n" \
  "v_mfma_f32_16x16x32_bf16 v[108:111], a[152:155], v[140:143], v[108:111]\n" \
  "v_mfma_f32_16x16x32_bf16 v[108:111], a[184:187], v[156:159], v[108:111]\n" \
  "v_mfma_f32_16x16x32_bf16 v[124:127], a[188:191], v[156:159], v[124:127]\n" \
  "v_mfma_f32_16x16x32_bf16 v[124:127], a[156:159], v[140:143], v[124:127]\n" \
  "v_mfma_f32_16x16x32_bf16 v[120:123], a[156:159], v[136:139], v[120:123]\n" \
  "v_mfma_f32_16x16x32_bf16 v[120:123], a[188:191], v[152:155], v[120:123]\n" \
  "v_mfma_f32_16x16x32_bf16 v[104:107], a[184:187], v[152:155], v[104:107]\n" \
  "buffer_load_dwordx4 v224, s[36:39], s61 offen offset:3072 sc0 sc1 lds\n" \
  "v_mfma_f32_16x16x32_bf16 v[104:107], a[152:155], v[136:139], v[104:107]\n" \
  "s_add_u32 s61, s61, s64\n" \
  "v_mfma_f32_16x16x32_bf16 v[100:103], a[152:155], v[132:135], v[100:103]\n" \
  "v_mfma_f32_16x16x32_bf16 v[100:103], a[184:187], v[148:151], v[100:103]\n" \
  "v_mfma_f32_16x16x32_bf16 v[116:119], a[188:191], v[148:151], v[116:119]\n" \
  "v_mfma_f32_16x16x32_bf16 v[116:119], a[156:159], v[132:135], v[116:119]\n" \
  "v_mfma_f32_16x16x32_bf16 v[112:115], a[156:159], v[128:131], v[112:115]\n" \
  "v_mfma_f32_16x16x32_bf16 v[112:115], a[188:191], v[144:147], v[112:115]\n" \
  "s_waitcnt vmcnt(4) lgkmcnt(0)\n" \
  "v_mfma_f32_16x16x32_bf16 v[96:99], a[184:187], v[144:147], v[96:99]\n" \
  "s_barrier\n" \
  "v_mfma_f32_16x16x32_bf16 v[96:99], a[152:155], v[128:131], v[96:99]\n" \
  "v_mfma_f32_16x16x32_bf16 a[0:3], a[128:131], v[160:163], a[0:3]\n" \
  "v_mfma_f32_16x16x32_bf16 a[0:3], a[160:163], v[176:179], a[0:3]\n" \
  "ds_read_b64_tr_b16 a[224:225], v210 offset:49152\n" \
  "v_mfma_f32_16x16x32_bf16 a[16:19], a[164:167], v[176:179], a[16:19]\n" \
  "v_mfma_f32_16x16x32_bf16 a[16:19], a[132:135], v[160:163], a[16:19]\n" \
  "ds_read_b64_tr_b16 a[226:227], v210 offset:49664\n" \
  "v_mfma_f32_16x16x32_bf16 a[20:23], a[132:135], v[164:167], a[20:23]\n" \
  "v_mfma_f32_16x16x32_bf16 a[20:23], a[164:167], v[180:183], a[20:23]\n" \
  "ds_read_b128 v[128:131], v208 offset:32768\n" \
  "v_mfma_f32_16x16x32_bf16 a[4:7], a[160:163], v[180:183], a[4:7]\n" \
  "v_mfma_f32_16x16x32_bf16 a[4:7], a[128:131], v[164:167], a[4:7]\n" \
  "ds_read_b64_tr_b16 a[228:229], v211 offset:49152\n" \
  "v_mfma_f32_16x16x32_bf16 a[8:11], a[128:131], v[168:171], a[8:11]\n" \
  "s_add_u32 m0, s67, 16384\n" \
  "v_mfma_f32_16x16x32_bf16 a[8:11], a[160:163], v[184:187], a[8:11]\n" \
  "ds_read_b64_tr_b16 a[230:231], v211 offset:49664\n" \
  "v_mfma_f32_16x16x32_bf16 a[24:27], a[164:167], v[184:187], a[24:27]\n" \
  "buffer_load_dwordx4 v225, s[40:43], s62 offen offset:0  lds\n" \
  "v_mfma_f32_16x16x32_bf16 a[24:27], a[132:135], v[168:171], a[24:27]\n" \
  "ds_read_b128 v[144:147], v209 offset:32768\n" \
  "v_mfma_f32_16x16x32_bf16 a[28:31], a[132:135], v[172:175], a[28:31]\n" \
  "v_mfma_f32_16x16x32_bf16 a[28:31], a[164:167], v[188:191], a[28:31]\n" \
  "ds_read_b64_tr_b16 a[232:233], v212 offset:49152\n" \
  "v_mfma_f32_16x16x32_bf16 a[12:15], a[160:163], v[188:191], a[12:15]\n" \
  "buffer_load_dwordx4 v226, s[40:43], s62 offen offset:1024  lds\n" \
  "v_mfma_f32_16x16x32_bf16 a[12:15], a[128:131], v[172:175], a[12:15]\n" \
  "ds_read_b64_tr_b16 a[234:235], v212 offset:49664\n" \
  "v_mfma_f32_16x16x32_bf16 a[44:47], a[136:139], v[172:175], a[44:47]\n" \
  "v_mfma_f32_16x16x32_bf16 a[44:47], a[168:171], v[188:191], a[44:47]\n" \
  "ds_read_b128 v[132:135], v208 offset:34816\n" \
  "v_mfma_f32_16x16x32_bf16 a[60:63], a[172:175], v[188:191], a[60:63]\n" \
  "buffer_load_dwordx4 v227, s[40:43], s62 offen offset:2048  lds\n" \
  "v_mfma_f32_16x16x32_bf16 a[60:63], a[140:143], v[172:175], a[60:63]\n" \
  "ds_read_b64_tr_b16 a[236:237], v213 offset:49152\n" \
  "v_mfma_f32_16x16x32_bf16 a[56:59], a[140:143], v[168:171], a[56:59]\n" \
  "v_mfma_f32_16x16x32_bf16 a[56:59], a[172:175], v[184:187], a[56:59]\n" \
  "ds_read_b64_tr_b16 a[238:239], v213 offset:49664\n" \
  "v_mfma_f32_16x16x32_bf16 a[40:43], a[168:171], v[184:187], a[40:43]\n" \
  "buffer_load_dwordx4 v228, s[40:43], s62 offen offset:3072  lds\n" \
  "v_mfma_f32_16x16x32_bf16 a[40:43], a[136:139], v[168:171], a[40:43]\n" \
  "s_add_u32 s62, s62, 0x200000\n" \
  "v_mfma_f32_16x16x32_bf16 a[36:39], a[136:139], v[164:167], a[36:39]\n" \
  "s_add_u32 m0, s67, 32768\n" \
  "v_mfma_f32_16x16x32_bf16 a[36:39], a[168:171], v[180:183], a[36:39]\n" \
  "ds_read_b128 v[148:151], v209 offset:34816\n" \
  "v_mfma_f32_16x16x32_bf16 a[52:55], a[172:175], v[180:183], a[52:55]\n" \
  "buffer_load_dwordx4 v225, s[40:43], s62 offen offset:0  lds\n" \
  "v_mfma_f32_16x16x32_bf16 a[52:55], a[140:143], v[164:167], a[52:55]\n" \
  "ds_read_b64_tr_b16 a[240:241], v210 offset:50176\n" \
  "v_mfma_f32_16x16x32_bf16 a[48:51], a[140:143], v[160:163], a[48:51]\n" \
  "ds_read_b64_tr_b16 a[242:243], v210 offset:50688\n" \
  "v_mfma_f32_16x16x32_bf16 a[48:51], a[172:175], v[176:179], a[48:51]\n" \
  "ds_read_b128 v[136:139], v208 offset:36864\n" \
  "v_mfma_f32_16x16x32_bf16 a[32:35], a[168:171], v[176:179], a[32:35]\n" \
  "buffer_load_dwordx4 v226, s[40:43], s62 offen offset:1024  lds\n" \
  "v_mfma_f32_16x16x32_bf16 a[32:35], a[136:139], v[160:163], a[32:35]\n" \
  "ds_read_b64_tr_b16 a[244:245], v211 offset:50176\n" \
  "v_mfma_f32_16x16x32_bf16 a[64:67], a[144:147], v[160:163], a[64:67]\n" \
  "v_mfma_f32_16x16x32_bf16 a[64:67], a[176:179], v[176:179], a[64:67]\n" \
  "ds_read_b64_tr_b16 a[246:247], v211 offset:50688\n" \
  "v_mfma_f32_16x16x32_bf16 a[80:83], a[180:183], v[176:179], a[80:83]\n" \
  "buffer_load_dwordx4 v227, s[40:43], s62 offen offset:2048  lds\n" \
  "v_mfma_f32_16x16x32_bf16 a[80:83], a[148:151], v[160:163], a[80:83]\n" \
  "ds_read_b128 v[152:155], v209 offset:36864\n" \
  "v_mfma_f32_16x16x32_bf16 a[84:87], a[148:151], v[164:167], a[84:87]\n" \
  "v_cmp_gt_u32 vcc, s73, v229\n" \
  "v_mfma_f32_16x16x32_bf16 a[84:87], a[180:183], v[180:183], a[84:87]\n" \
  "ds_read_b64_tr_b16 a[248:249], v212 offset:50176\n" \
  "v_mfma_f32_16x16x32_bf16 a[68:71], a[176:179], v[180:183], a[68:71]\n" \
  "buffer_load_dwordx4 v228, s[40:43], s62 offen offset:3072  lds\n" \
  "v_mfma_f32_16x16x32_bf16 a[68:71], a[144:147], v[164:167], a[68:71]\n" \
  "ds_read_b64_tr_b16 a[250:251], v212 offset:50688\n" \
  "v_mfma_f32_16x16x32_bf16 a[72:75], a[144:147], v[168:171], a[72:75]\n" \
  "s_cbranch_vccnz LK_slowplain1\n" \
  "LK_retplain1:\n" \
  "v_mfma_f32_16x16x32_bf16 a[72:75], a[176:179], v[184:187], a[72:75]\n" \
  "ds_read_b128 v[140:143], v208 offset:38912\n" \
  "v_mfma_f32_16x16x32_bf16 a[88:91], a[180:183], v[184:187], a[88:91]\n" \
  "s_add_u32 m0, s66, 0\n" \
  "v_mfma_f32_16x16x32_bf16 a[88:91], a[148:151], v[168:171], a[88:91]\n" \
  "ds_read_b64_tr_b16 a[252:253], v213 offset:50176\n" \
  "v_mfma_f32_16x16x32_bf16 a[92:95], a[148:151], v[172:175], a[92:95]\n" \
  "buffer_load_dwordx4 v221, s[36:39], s60 offen offset:0 sc0 sc1 lds\n" \
  "v_mfma_f32_16x16x32_bf16 a[92:95], a[180:183], v[188:191], a[92:95]\n" \
  "ds_read_b64_tr_b16 a[254:255], v213 offset:50688\n" \
  "v_mfma_f32_16x16x32_bf16 a[76:79], a[176:179], v[188:191], a[76:79]\n" \
  "v_mfma_f32_16x16x32_bf16 a[76:79], a[144:147], v[172:175], a[76:79]\n" \
  "ds_read_b128 v[156:159], v209 offset:38912\n" \
  "v_mfma_f32_16x16x32_bf16 a[108:111], a[152:155], v[172:175], a[108:111]\n" \
  "s_add_u32 s73, s73, s74\n" \
  "v_mfma_f32_16x16x32_bf16 a[108:111], a[184:187], v[188:191], a[108:111]\n" \
  "buffer_load_dwordx4 v222, s[36:39], s60 offen offset:1024 sc0 sc1 lds\n" \
  "v_mfma_f32_16x16x32_bf16 a[124:127], a[188:191], v[188:191], a[124:127]\n" \
  "s_add_u32 s62, s62, 0x200000\n" \
  "v_mfma_f32_16x16x32_bf16 a[124:127], a[156:159], v[172:175], a[124:127]\n" \
  "v_mfma_f32_16x16x32_bf16 a[120:123], a[156:159], v[168:171], a[120:123]\n" \
  "buffer_load_dwordx4 v223, s[36:39], s60 offen offset:2048 sc0 sc1 lds\n" \
  "v_mfma_f32_16x16x32_bf16 a[120:123], a[188:191], v[184:187], a[120:123]\n" \
  "v_mfma_f32_16x16x32_bf16 a[104:107], a[184:187], v[184:187], a[104:107]\n" \
  "v_mfma_f32_16x16x32_bf16 a[104:107], a[152:155], v[168:171], a[104:107]\n" \
  "v_mfma_f32_16x16x32_bf16 a[100:103], a[152:155], v[164:167], a[100:103]\n" \
  "buffer_load_dwordx4 v224, s[36:39], s60 offen offset:3072 sc0 sc1 lds\n" \
  "v_mfma_f32_16x16x32_bf16 a[100:103], a[184:187], v[180:183], a[100:103]\n" \
  "s_add_u32 s60, s60, s64\n" \
  "v_mfma_f32_16x16x32_bf16 a[116:119], a[188:191], v[180:183], a[116:119]\n" \
  "v_mfma_f32_16x16x32_bf16 a[116:119], a[156:159], v[164:167], a[116:119]\n" \
  "v_mfma_f32_16x16x32_bf16 a[112:115], a[156:159], v[160:163], a[112:115]\n" \
  "v_mfma_f32_16x16x32_bf16 a[112:115], a[188:191], v[176:179], a[112:115]\n" \
  "s_waitcnt vmcnt(12) lgkmcnt(0)\n" \
  "v_mfma_f32_16x16x32_bf16 a[96:99], a[184:187], v[176:179], a[96:99]\n" \
  "s_barrier\n" \
  "v_mfma_f32_16x16x32_bf16 a[96:99], a[152:155], v[160:163], a[96:99]\n" \
  "v_mfma_f32_16x16x32_bf16 v[0:3], a[192:195], v[128:131], v[0:3]\n" \
  "buffer_load_dword v229, v230, s[52:55], 0 offen sc0 sc1\n" \
  "v_mfma_f32_16x16x32_bf16 v[0:3], a[224:227], v[144:147], v[0:3]\n" \
  "ds_read_b64_tr_b16 a[128:129], v210 offset:0\n" \
  "v_mfma_f32_16x16x32_bf16 v[16:19], a[228:231], v[144:147], v[16:19]\n" \
  "v_mfma_f32_16x16x32_bf16 v[16:19], a[196:199], v[128:131], v[16:19]\n" \
  "ds_read_b64_tr_b16 a[130:131], v210 offset:512\n" \
  "v_mfma_f32_16x16x32_bf16 v[20:23], a[196:199], v[132:135], v[20:23]\n" \
  "v_mfma_f32_16x16x32_bf16 v[20:23], a[228:231], v[148:151], v[20:23]\n" \
  "ds_read_b128 v[160:163], v208 offset:49152\n" \
  "v_mfma_f32_16x16x32_bf16 v[4:7], a[224:227], v[148:151], v[4:7]\n" \
  "v_mfma_f32_16x16x32_bf16 v[4:7], a[192:195], v[132:135], v[4:7]\n" \
  "ds_read_b64_tr_b16 a[132:133], v211 offset:0\n" \
  "v_mfma_f32_16x16x32_bf16 v[8:11], a[192:195], v[136:139], v[8:11]\n" \
  "v_mfma_f32_16x16x32_bf16 v[8:11], a[224:227], v[152:155], v[8:11]\n" \
  "ds_read_b64_tr_b16 a[134:135], v211 offset:512\n" \
  "v_mfma_f32_16x16x32_bf16 v[24:27], a[228:231], v[152:155], v[24:27]\n" \
  "v_mfma_f32_16x16x32_bf16 v[24:27], a[196:199], v[136:139], v[24:27]\n" \
  "ds_read_b128 v[176:179], v209 offset:49152\n" \
  "v_mfma_f32_16x16x32_bf16 v[28:31], a[196:199], v[140:143], v[28:31]\n" \
  "v_mfma_f32_16x16x32_bf16 v[28:31], a[228:231], v[156:159], v[28:31]\n" \
  "ds_read_b64_tr_b16 a[136:137], v212 offset:0\n" \
  "v_mfma_f32_16x16x32_bf16 v[12:15], a[224:227], v[156:159], v[12:15]\n" \
  "v_mfma_f32_16x16x32_bf16 v[12:15], a[192:195], v[140:143], v[12:15]\n" \
  "ds_read_b64_tr_b16 a[138:139], v212 offset:512\n" \
  "v_mfma_f32_16x16x32_bf16 v[44:47], a[200:203], v[140:143], v[44:47]\n" \
  "v_mfma_f32_16x16x32_bf16 v[44:47], a[232:235], v[156:159], v[44:47]\n" \
  "ds_read_b128 v[164:167], v208 offset:51200\n" \
  "v_mfma_f32_16x16x32_bf16 v[60:63], a[236:239], v[156:159], v[60:63]\n" \
  "v_mfma_f32_16x16x32_bf16 v[60:63], a[204:207], v[140:143], v[60:63]\n" \
  "ds_read_b64_tr_b16 a[140:141], v213 offset:0\n" \
  "v_mfma_f32_16x16x32_bf16 v[56:59], a[204:207], v[136:139], v[56:59]\n" \
  "v_mfma_f32_16x16x32_bf16 v[56:59], a[236:239], v[152:155], v[56:59]\n" \
  "ds_read_b64_tr_b16 a[142:143], v213 offset:512\n" \
  "v_mfma_f32_16x16x32_bf16 v[40:43], a[232:235], v[152:155], v[40:43]\n" \
  "v_mfma_f32_16x16x32_bf16 v[40:43], a[200:203], v[136:139], v[40:43]\n" \
  "ds_read_b128 v[180:183], v209 offset:51200\n" \
  "v_mfma_f32_16x16x32_bf16 v[36:39], a[200:203], v[132:135], v[36:39]\n" \
  "v_mfma_f32_16x16x32_bf16 v[36:39], a[232:235], v[148:151], v[36:39]\n" \
  "ds_read_b64_tr_b16 a[144:145], v210 offset:1024\n" \
  "v_mfma_f32_16x16x32_bf16 v[52:55], a[236:239], v[148:151], v[52:55]\n" \
  "v_mfma_f32_16x16x32_bf16 v[52:55], a[204:207], v[132:135], v[52:55]\n" \
  "ds_read_b64_tr_b16 a[146:147], v210 offset:1536\n" \
  "v_mfma_f32_16x16x32_bf16 v[48:51], a[204:207], v[128:131], v[48:51]\n" \
  "s_add_u32 m0, s66, 16384\n" \
  "v_mfma_f32_16x16x32_bf16 v[48:51], a[236:239], v[144:147], v[48:51]\n" \
  "ds_read_b128 v[168:171], v208 offset:53248\n" \
  "v_mfma_f32_16x16x32_bf16 v[32:35], a[232:235], v[144:147], v[32:35]\n" \
  "buffer_load_dwordx4 v221, s[36:39], s61 offen offset:0 sc0 sc1 lds\n" \
  "v_mfma_f32_16x16x32_bf16 v[32:35], a[200:203], v[128:131], v[32:35]\n" \
  "ds_read_b64_tr_b16 a[148:149], v211 offset:1024\n" \
  "v_mfma_f32_16x16x32_bf16 v[64:67], a[208:211], v[128:131], v[64:67]\n" \
  "v_mfma_f32_16x16x32_bf16 v[64:67], a[240:243], v[144:147], v[64:67]\n" \
  "ds_read_b64_tr_b16 a[150:151], v211 offset:1536\n" \
  "v_mfma_f32_16x16x32_bf16 v[80:83], a[244:247], v[144:147], v[80:83]\n" \
  "v_mfma_f32_16x16x32_bf16 v[80:83], a[212:215], v[128:131], v[80:83]\n" \
  "ds_read_b128 v[184:187], v209 offset:53248\n" \
  "v_mfma_f32_16x16x32_bf16 v[84:87], a[212:215], v[132:135], v[84:87]\n" \
  "v_mfma_f32_16x16x32_bf16 v[84:87], a[244:247], v[148:151], v[84:87]\n" \
  "ds_read_b64_tr_b16 a[152:153], v212 offset:1024\n" \
  "v_mfma_f32_16x16x32_bf16 v[68:71], a[240:243], v[148:151], v[68:71]\n" \
  "buffer_load_dwordx4 v222, s[36:39], s61 offen offset:1024 sc0 sc1 lds\n" \
  "v_mfma_f32_16x16x32_bf16 v[68:71], a[208:211], v[132:135], v[68:71]\n" \
  "ds_read_b64_tr_b16 a[154:155], v212 offset:1536\n" \
  "v_mfma_f32_16x16x32_bf16 v[72:75], a[208:211], v[136:139], v[72:75]\n" \
  "v_mfma_f32_16x16x32_bf16 v[72:75], a[240:243], v[152:155], v[72:75]\n" \
  "ds_read_b128 v[172:175], v208 offset:55296\n" \
  "v_mfma_f32_16x16x32_bf16 v[88:91], a[244:247], v[152:155], v[88:91]\n" \
  "v_mfma_f32_16x16x32_bf16 v[88:91], a[212:215], v[136:139], v[88:91]\n" \
  "ds_read_b64_tr_b16 a[156:157], v213 offset:1024\n" \
  "v_mfma_f32_16x16x32_bf16 v[92:95], a[212:215], v[140:143], v[92:95]\n" \
  "v_mfma_f32_16x16x32_bf16 v[92:95], a[244:247], v[156:159], v[92:95]\n" \
  "ds_read_b64_tr_b16 a[158:159], v213 offset:1536\n" \
  "v_mfma_f32_16x16x32_bf16 v[76:79], a[240:243], v[156:159], v[76:79]\n" \
  "buffer_load_dwordx4 v223, s[36:39], s61 offen offset:2048 sc0 sc1 lds\n" \
  "v_mfma_f32_16x16x32_bf16 v[76:79], a[208:211], v[140:143], v[76:79]\n" \
  "ds_read_b128 v[188:191], v209 offset:55296\n" \
  "v_mfma_f32_16x16x32_bf16 v[108:111], a[216:219], v[140:143], v[108:111]\n" \
  "v_mfma_f32_16x16x32_bf16 v[108:111], a[248:251], v[156:159], v[108:111]\n" \
  "v_mfma_f32_16x16x32_bf16 v[124:127], a[252:255], v[156:159], v[124:127]\n" \
  "v_mfma_f32_16x16x32_bf16 v[124:127], a[220:223], v[140:143], v[124:127]\n" \
  "v_mfma_f32_16x16x32_bf16 v[120:123], a[220:223], v[136:139], v[120:123]\n" \
  "v_mfma_f32_16x16x32_bf16 v[120:123], a[252:255], v[152:155], v[120:123]\n" \
  "v_mfma_f32_16x16x32_bf16 v[104:107], a[248:251], v[152:155], v[104:107]\n" \
  "buffer_load_dwordx4 v224, s[36:39], s61 offen offset:3072 sc0 sc1 lds\n" \
  "v_mfma_f32_16x16x32_bf16 v[104:107], a[216:219], v[136:139], v[104:107]\n" \
  "s_add_u32 s61, s61, s64\n" \
  "v_mfma_f32_16x16x32_bf16 v[100:103], a[216:219], v[132:135], v[100:103]\n" \
  "v_mfma_f32_16x16x32_bf16 v[100:103], a[248:251], v[148:151], v[100:103]\n" \
  "v_mfma_f32_16x16x32_bf16 v[116:119], a[252:255], v[148:151], v[116:119]\n" \
  "v_mfma_f32_16x16x32_bf16 v[116:119], a[220:223], v[132:135], v[116:119]\n" \
  "v_mfma_f32_16x16x32_bf16 v[112:115], a[220:223], v[128:131], v[112:115]\n" \
  "v_mfma_f32_16x16x32_bf16 v[112:115], a[252:255], v[144:147], v[112:115]\n" \
  "s_waitcnt vmcnt(4) lgkmcnt(0)\n" \
  "v_mfma_f32_16x16x32_bf16 v[96:99], a[248:251], v[144:147], v[96:99]\n" \
  "s_barrier\n" \
  "v_mfma_f32_16x16x32_bf16 v[96:99], a[216:219], v[128:131], v[96:99]\n" \
  "v_mfma_f32_16x16x32_bf16 a[0:3], a[192:195], v[160:163], a[0:3]\n" \
  "v_mfma_f32_16x16x32_bf16 a[0:3], a[224:227], v[176:179], a[0:3]\n" \
  "ds_read_b64_tr_b16 a[160:161], v210 offset:16384\n" \
  "v_mfma_f32_16x16x32_bf16 a[16:19], a[228:231], v[176:179], a[16:19]\n" \
  "v_mfma_f32_16x16x32_bf16 a[16:19], a[196:199], v[160:163], a[16:19]\n" \
  "ds_read_b64_tr_b16 a[162:163], v210 offset:16896\n" \
  "v_mfma_f32_16x16x32_bf16 a[20:23], a[196:199], v[164:167], a[20:23]\n" \
  "v_mfma_f32_16x16x32_bf16 a[20:23], a[228:231], v[180:183], a[20:23]\n" \
  "ds_read_b128 v[128:131], v208 offset:0\n" \
  "v_mfma_f32_16x16x32_bf16 a[4:7], a[224:227], v[180:183], a[4:7]\n" \
  "v_mfma_f32_16x16x32_bf16 a[4:7], a[192:195], v[164:167], a[4:7]\n" \
  "ds_read_b64_tr_b16 a[164:165], v211 offset:16384\n" \
  "v_mfma_f32_16x16x32_bf16 a[8:11], a[192:195], v[168:171], a[8:11]\n" \
  "s_add_u32 m0, s67, 49152\n" \
  "v_mfma_f32_16x16x32_bf16 a[8:11], a[224:227], v[184:187], a[8:11]\n" \
  "ds_read_b64_tr_b16 a[166:167], v211 offset:16896\n" \
  "v_mfma_f32_16x16x32_bf16 a[24:27], a[228:231], v[184:187], a[24:27]\n" \
  "buffer_load_dwordx4 v225, s[40:43], s62 offen offset:0  lds\n" \
  "v_mfma_f32_16x16x32_bf16 a[24:27], a[196:199], v[168:171], a[24:27]\n" \
  "ds_read_b128 v[144:147], v209 offset:0\n" \
  "v_mfma_f32_16x16x32_bf16 a[28:31], a[196:199], v[172:175], a[28:31]\n" \
  "v_mfma_f32_16x16x32_bf16 a[28:31], a[228:231], v[188:191], a[28:31]\n" \
  "ds_read_b64_tr_b16 a[168:169], v212 offset:16384\n" \
  "v_mfma_f32_16x16x32_bf16 a[12:15], a[224:227], v[188:191], a[12:15]\n" \
  "buffer_load_dwordx4 v226, s[40:43], s62 offen offset:1024  lds\n" \
  "v_mfma_f32_16x16x32_bf16 a[12:15], a[192:195], v[172:175], a[12:15]\n" \
  "ds_read_b64_tr_b16 a[170:171], v212 offset:16896\n" \
  "v_mfma_f32_16x16x32_bf16 a[44:47], a[200:203], v[172:175], a[44:47]\n" \
  "v_mfma_f32_16x16x32_bf16 a[44:47], a[232:235], v[188:191], a[44:47]\n" \
  "ds_read_b128 v[132:135], v208 offset:2048\n" \
  "v_mfma_f32_16x16x32_bf16 a[60:63], a[236:239], v[188:191], a[60:63]\n" \
  "buffer_load_dwordx4 v227, s[40:43], s62 offen offset:2048  lds\n" \
  "v_mfma_f32_16x16x32_bf16 a[60:63], a[204:207], v[172:175], a[60:63]\n" \
  "ds_read_b64_tr_b16 a[172:173], v213 offset:16384\n" \
  "v_mfma_f32_16x16x32_bf16 a[56:59], a[204:207], v[168:171], a[56:59]\n" \
  "v_mfma_f32_16x16x32_bf16 a[56:59], a[236:239], v[184:187], a[56:59]\n" \
  "ds_read_b64_tr_b16 a[174:175], v213 offset:16896\n" \
  "v_mfma_f32_16x16x32_bf16 a[40:43], a[232:235], v[184:187], a[40:43]\n" \
  "buffer_load_dwordx4 v228, s[40:43], s62 offen offset:3072  lds\n" \
  "v_mfma_f32_16x16x32_bf16 a[40:43], a[200:203], v[168:171], a[40:43]\n" \
  "s_add_u32 s62, s62, 0x200000\n" \
  "v_mfma_f32_16x16x32_bf16 a[36:39], a[200:203], v[164:167], a[36:39]\n" \
  "s_add_u32 m0, s67, 0\n" \
  "v_mfma_f32_16x16x32_bf16 a[36:39], a[232:235], v[180:183], a[36:39]\n" \
  "ds_read_b128 v[148:151], v209 offset:2048\n" \
  "v_mfma_f32_16x16x32_bf16 a[52:55], a[236:239], v[180:183], a[52:55]\n" \
  "buffer_load_dwordx4 v225, s[40:43], s62 offen offset:0  lds\n" \
  "v_mfma_f32_16x16x32_bf16 a[52:55], a[204:207], v[164:167], a[52:55]\n" \
  "ds_read_b64_tr_b16 a[176:177], v210 offset:17408\n" \
  "v_mfma_f32_16x16x32_bf16 a[48:51], a[204:207], v[160:163], a[48:51]\n" \
  "ds_read_b64_tr_b16 a[178:179], v210 offset:17920\n" \
  "v_mfma_f32_16x16x32_bf16 a[48:51], a[236:239], v[176:179], a[48:51]\n" \
  "ds_read_b128 v[136:139], v208 offset:4096\n" \
  "v_mfma_f32_16x16x32_bf16 a[32:35], a[232:235], v[176:179], a[32:35]\n" \
  "buffer_load_dwordx4 v226, s[40:43], s62 offen offset:1024  lds\n" \
  "v_mfma_f32_16x16x32_bf16 a[32:35], a[200:203], v[160:163], a[32:35]\n" \
  "ds_read_b64_tr_b16 a[180:181], v211 offset:17408\n" \
  "v_mfma_f32_16x16x32_bf16 a[64:67], a[208:211], v[160:163], a[64:67]\n" \
  "v_mfma_f32_16x16x32_bf16 a[64:67], a[240:243], v[176:179], a[64:67]\n" \
  "ds_read_b64_tr_b16 a[182:183], v211 offset:17920\n" \
  "v_mfma_f32_16x16x32_bf16 a[80:83], a[244:247], v[176:179], a[80:83]\n" \
  "buffer_load_dwordx4 v227, s[40:43], s62 offen offset:2048  lds\n" \
  "v_mfma_f32_16x16x32_bf16 a[80:83], a[212:215], v[160:163], a[80:83]\n" \
  "ds_read_b128 v[152:155], v209 offset:4096\n" \
  "v_mfma_f32_16x16x32_bf16 a[84:87], a[212:215], v[164:167], a[84:87]\n" \
  "v_cmp_gt_u32 vcc, s73, v229\n" \
  "v_mfma_f32_16x16x32_bf16 a[84:87], a[244:247], v[180:183], a[84:87]\n" \
  "ds_read_b64_tr_b16 a[184:185], v212 offset:17408\n" \
  "v_mfma_f32_16x16x32_bf16 a[68:71], a[240:243], v[180:183], a[68:71]\n" \
  "buffer_load_dwordx4 v228, s[40:43], s62 offen offset:3072  lds\n" \
  "v_mfma_f32_16x16x32_bf16 a[68:71], a[208:211], v[164:167], a[68:71]\n" \
  "ds_read_b64_tr_b16 a[186:187], v212 offset:17920\n" \
  "v_mfma_f32_16x16x32_bf16 a[72:75], a[208:211], v[168:171], a[72:75]\n" \
  "s_cbranch_vccnz LK_slowplain3\n" \
  "LK_retplain3:\n" \
  "v_mfma_f32_16x16x32_bf16 a[72:75], a[240:243], v[184:187], a[72:75]\n" \
  "ds_read_b128 v[140:143], v208 offset:6144\n" \
  "v_mfma_f32_16x16x32_bf16 a[88:91], a[244:247], v[184:187], a[88:91]\n" \
  "s_add_u32 m0, s66, 32768\n" \
  "v_mfma_f32_16x16x32_bf16 a[88:91], a[212:215], v[168:171], a[88:91]\n" \
  "ds_read_b64_tr_b16 a[188:189], v213 offset:17408\n" \
  "v_mfma_f32_16x16x32_bf16 a[92:95], a[212:215], v[172:175], a[92:95]\n" \
  "buffer_load_dwordx4 v221, s[36:39], s60 offen offset:0 sc0 sc1 lds\n" \
  "v_mfma_f32_16x16x32_bf16 a[92:95], a[244:247], v[188:191], a[92:95]\n" \
  "ds_read_b64_tr_b16 a[190:191], v213 offset:17920\n" \
  "v_mfma_f32_16x16x32_bf16 a[76:79], a[240:243], v[188:191], a[76:79]\n" \
  "v_mfma_f32_16x16x32_bf16 a[76:79], a[208:211], v[172:175], a[76:79]\n" \
  "ds_read_b128 v[156:159], v209 offset:6144\n" \
  "v_mfma_f32_16x16x32_bf16 a[108:111], a[216:219], v[172:175], a[108:111]\n" \
  "s_add_u32 s73, s73, s74\n" \
  "v_mfma_f32_16x16x32_bf16 a[108:111], a[248:251], v[188:191], a[108:111]\n" \
  "buffer_load_dwordx4 v222, s[36:39], s60 offen offset:1024 sc0 sc1 lds\n" \
  "v_mfma_f32_16x16x32_bf16 a[124:127], a[252:255], v[188:191], a[124:127]\n" \
  "s_add_u32 s62, s62, 0x200000\n" \
  "v_mfma_f32_16x16x32_bf16 a[124:127], a[220:223], v[172:175], a[124:127]\n" \
  "v_mfma_f32_16x16x32_bf16 a[120:123], a[220:223], v[168:171], a[120:123]\n" \
  "buffer_load_dwordx4 v223, s[36:39], s60 offen offset:2048 sc0 sc1 lds\n" \
  "v_mfma_f32_16x16x32_bf16 a[120:123], a[252:255], v[184:187], a[120:123]\n" \
  "v_mfma_f32_16x16x32_bf16 a[104:107], a[248:251], v[184:187], a[104:107]\n" \
  "v_mfma_f32_16x16x32_bf16 a[104:107], a[216:219], v[168:171], a[104:107]\n" \
  "v_mfma_f32_16x16x32_bf16 a[100:103], a[216:219], v[164:167], a[100:103]\n" \
  "buffer_load_dwordx4 v224, s[36:39], s60 offen offset:3072 sc0 sc1 lds\n" \
  "v_mfma_f32_16x16x32_bf16 a[100:103], a[248:251], v[180:183], a[100:103]\n" \
  "s_add_u32 s60, s60, s64\n" \
  "v_mfma_f32_16x16x32_bf16 a[116:119], a[252:255], v[180:183], a[116:119]\n" \
  "v_mfma_f32_16x16x32_bf16 a[116:119], a[220:223], v[164:167], a[116:119]\n" \
  "v_mfma_f32_16x16x32_bf16 a[112:115], a[220:223], v[160:163], a[112:115]\n" \
  "v_mfma_f32_16x16x32_bf16 a[112:115], a[252:255], v[176:179], a[112:115]\n" \
  "s_waitcnt vmcnt(12) lgkmcnt(0)\n" \
  "v_mfma_f32_16x16x32_bf16 a[96:99], a[248:251], v[176:179], a[96:99]\n" \
  "s_barrier\n" \
  "s_cmp_lg_u32 s62, s68\n" \
  "v_mfma_f32_16x16x32_bf16 a[96:99], a[216:219], v[160:163], a[96:99]\n" \
  "s_cbranch_scc1 LK_plainloop\n" \
  "LK_epi:\n" \
  "v_mfma_f32_16x16x32_bf16 v[0:3], a[128:131], v[128:131], v[0:3]\n" \
  "v_mfma_f32_16x16x32_bf16 v[0:3], a[160:163], v[144:147], v[0:3]\n" \
  "ds_read_b64_tr_b16 a[192:193], v210 offset:32768\n" \
  "v_mfma_f32_16x16x32_bf16 v[16:19], a[164:167], v[144:147], v[16:19]\n" \
  "ds_read_b64_tr_b16 a[194:195], v210 offset:33280\n" \
  "s_mov_b64 s[44:45], s[18:19]\n" \
  "v_mfma_f32_16x16x32_bf16 v[16:19], a[132:135], v[128:131], v[16:19]\n" \
  "ds_read_b64_tr_b16 a[196:197], v211 offset:32768\n" \
  "s_mov_b32 s46, 2089472\n" \
  "v_mfma_f32_16x16x32_bf16 v[20:23], a[132:135], v[132:135], v[20:23]\n" \
  "ds_read_b64_tr_b16 a[198:199], v211 offset:33280\n" \
  "s_mov_b32 s47, 0x00020000\n" \
  "v_mfma_f32_16x16x32_bf16 v[20:23], a[164:167], v[148:151], v[20:23]\n" \
  "s_add_u32 m0, s66, 49152\n" \
  "ds_read_b64_tr_b16 a[200:201], v212 offset:32768\n" \
  "v_mfma_f32_16x16x32_bf16 v[4:7], a[160:163], v[148:151], v[4:7]\n" \
  "buffer_load_dwordx4 v221, s[36:39], s61 offen offset:0 sc0 sc1 lds\n" \
  "ds_read_b64_tr_b16 a[202:203], v212 offset:33280\n" \
  "v_mfma_f32_16x16x32_bf16 v[4:7], a[128:131], v[132:135], v[4:7]\n" \
  "ds_read_b64_tr_b16 a[204:205], v213 offset:32768\n" \
  "v_and_b32 v217, 15, v214\n" \
  "v_mfma_f32_16x16x32_bf16 v[8:11], a[128:131], v[136:139], v[8:11]\n" \
  "ds_read_b64_tr_b16 a[206:207], v213 offset:33280\n" \
  "v_lshlrev_b32 v217, 13, v217\n" \
  "v_mfma_f32_16x16x32_bf16 v[8:11], a[160:163], v[152:155], v[8:11]\n" \
  "ds_read_b64_tr_b16 a[208:209], v210 offset:33792\n" \
  "v_bfe_u32 v216, v214, 4, 1\n" \
  "v_mfma_f32_16x16x32_bf16 v[24:27], a[164:167], v[152:155], v[24:27]\n" \
  "ds_read_b64_tr_b16 a[210:211], v210 offset:34304\n" \
  "v_lshl_add_u32 v217, v216, 5, v217\n" \
  "v_mfma_f32_16x16x32_bf16 v[24:27], a[132:135], v[136:139], v[24:27]\n" \
  "ds_read_b64_tr_b16 a[212:213], v211 offset:33792\n" \
  "v_lshrrev_b32 v216, 5, v214\n" \
  "v_mfma_f32_16x16x32_bf16 v[28:31], a[132:135], v[140:143], v[28:31]\n" \
  "ds_read_b64_tr_b16 a[214:215], v211 offset:34304\n" \
  "v_lshl_add_u32 v215, v216, 4, v217\n" \
  "v_mfma_f32_16x16x32_bf16 v[28:31], a[164:167], v[156:159], v[28:31]\n" \
  "ds_read_b64_tr_b16 a[216:217], v212 offset:33792\n" \
  "s_lshr_b32 s63, s69, 1\n" \
  "v_mfma_f32_16x16x32_bf16 v[12:15], a[160:163], v[156:159], v[12:15]\n" \
  "buffer_load_dwordx4 v222, s[36:39], s61 offen offset:1024 sc0 sc1 lds\n" \
  "ds_read_b64_tr_b16 a[218:219], v212 offset:34304\n" \
  "v_mfma_f32_16x16x32_bf16 v[12:15], a[128:131], v[140:143], v[12:15]\n" \
  "ds_read_b64_tr_b16 a[220:221], v213 offset:33792\n" \
  "s_lshl_b32 s63, s63, 20\n" \
  "v_mfma_f32_16x16x32_bf16 v[44:47], a[136:139], v[140:143], v[44:47]\n" \
  "ds_read_b64_tr_b16 a[222:223], v213 offset:34304\n" \
  "s_and_b32 s70, s69, 1\n" \
  "v_mfma_f32_16x16x32_bf16 v[44:47], a[168:171], v[156:159], v[44:47]\n" \
  "s_lshl_b32 s70, s70, 8\n" \
  "s_add_u32 s63, s63, s70\n" \
  "v_mfma_f32_16x16x32_bf16 v[60:63], a[172:175], v[156:159], v[60:63]\n" \
  "v_mov_b32 v226, 0xc2ad1f97\n" \
  "s_mov_b32 s50, 0x6abcedc7\n" \
  "v_mfma_f32_16x16x32_bf16 v[60:63], a[140:143], v[140:143], v[60:63]\n" \
  "s_mov_b32 s48, 0xbfb8aa3b\n" \
  "v_mfma_f32_16x16x32_bf16 v[56:59], a[140:143], v[136:139], v[56:59]\n" \
  "v_mfma_f32_16x16x32_bf16 v[56:59], a[172:175], v[152:155], v[56:59]\n" \
  "v_mfma_f32_16x16x32_bf16 v[40:43], a[168:171], v[152:155], v[40:43]\n" \
  "buffer_load_dwordx4 v223, s[36:39], s61 offen offset:2048 sc0 sc1 lds\n" \
  "v_mfma_f32_16x16x32_bf16 v[40:43], a[136:139], v[136:139], v[40:43]\n" \
  "v_mfma_f32_16x16x32_bf16 v[36:39], a[136:139], v[132:135], v[36:39]\n" \
  "v_mfma_f32_16x16x32_bf16 v[36:39], a[168:171], v[148:151], v[36:39]\n" \
  "v_mfma_f32_16x16x32_bf16 v[52:55], a[172:175], v[148:151], v[52:55]\n" \
  "v_mfma_f32_16x16x32_bf16 v[52:55], a[140:143], v[132:135], v[52:55]\n" \
  "v_mfma_f32_16x16x32_bf16 v[48:51], a[140:143], v[128:131], v[48:51]\n" \
  "v_mfma_f32_16x16x32_bf16 v[48:51], a[172:175], v[144:147], v[48:51]\n" \
  "v_mfma_f32_16x16x32_bf16 v[32:35], a[168:171], v[144:147], v[32:35]\n" \
  "buffer_load_dwordx4 v224, s[36:39], s61 offen offset:3072 sc0 sc1 lds\n" \
  "s_waitcnt vmcnt(4) \n" \
  "v_mfma_f32_16x16x32_bf16 v[32:35], a[136:139], v[128:131], v[32:35]\n" \
  "s_barrier\n" \
  "v_mfma_f32_16x16x32_bf16 v[64:67], a[144:147], v[128:131], v[64:67]\n" \
  "global_load_dword v220, v219, s[20:21] sc1\n" \
  "ds_read_b64_tr_b16 a[224:225], v210 offset:49152\n" \
  "v_mfma_f32_16x16x32_bf16 v[64:67], a[176:179], v[144:147], v[64:67]\n" \
  "ds_read_b64_tr_b16 a[226:227], v210 offset:49664\n" \
  "v_mfma_f32_16x16x32_bf16 v[80:83], a[180:183], v[144:147], v[80:83]\n" \
  "ds_read_b128 v[160:163], v208 offset:32768\n" \
  "v_mfma_f32_16x16x32_bf16 v[80:83], a[148:151], v[128:131], v[80:83]\n" \
  "ds_read_b64_tr_b16 a[228:229], v211 offset:49152\n" \
  "v_mfma_f32_16x16x32_bf16 v[84:87], a[148:151], v[132:135], v[84:87]\n" \
  "ds_read_b64_tr_b16 a[230:231], v211 offset:49664\n" \
  "v_mfma_f32_16x16x32_bf16 v[84:87], a[180:183], v[148:151], v[84:87]\n" \
  "ds_read_b128 v[176:179], v209 offset:32768\n" \
  "v_mfma_f32_16x16x32_bf16 v[68:71], a[176:179], v[148:151], v[68:71]\n" \
  "ds_read_b64_tr_b16 a[232:233], v212 offset:49152\n" \
  "v_mfma_f32_16x16x32_bf16 v[68:71], a[144:147], v[132:135], v[68:71]\n" \
  "ds_read_b64_tr_b16 a[234:235], v212 offset:49664\n" \
  "v_mfma_f32_16x16x32_bf16 v[72:75], a[144:147], v[136:139], v[72:75]\n" \
  "ds_read_b128 v[164:167], v208 offset:34816\n" \
  "v_mfma_f32_16x16x32_bf16 v[72:75], a[176:179], v[152:155], v[72:75]\n" \
  "ds_read_b64_tr_b16 a[236:237], v213 offset:49152\n" \
  "v_mfma_f32_16x16x32_bf16 v[88:91], a[180:183], v[152:155], v[88:91]\n" \
  "ds_read_b64_tr_b16 a[238:239], v213 offset:49664\n" \
  "v_mfma_f32_16x16x32_bf16 v[88:91], a[148:151], v[136:139], v[88:91]\n" \
  "ds_read_b128 v[180:183], v209 offset:34816\n" \
  "v_mfma_f32_16x16x32_bf16 v[92:95], a[148:151], v[140:143], v[92:95]\n" \
  "ds_read_b64_tr_b16 a[240:241], v210 offset:50176\n" \
  "v_mfma_f32_16x16x32_bf16 v[92:95], a[180:183], v[156:159], v[92:95]\n" \
  "ds_read_b64_tr_b16 a[242:243], v210 offset:50688\n" \
  "v_mfma_f32_16x16x32_bf16 v[76:79], a[176:179], v[156:159], v[76:79]\n" \
  "ds_read_b128 v[168:171], v208 offset:36864\n" \
  "v_mfma_f32_16x16x32_bf16 v[76:79], a[144:147], v[140:143], v[76:79]\n" \
  "ds_read_b64_tr_b16 a[244:245], v211 offset:50176\n" \
  "v_mfma_f32_16x16x32_bf16 v[108:111], a[152:155], v[140:143], v[108:111]\n" \
  "ds_read_b64_tr_b16 a[246:247], v211 offset:50688\n" \
  "v_mfma_f32_16x16x32_bf16 v[108:111], a[184:187], v[156:159], v[108:111]\n" \
  "ds_read_b128 v[184:187], v209 offset:36864\n" \
  "v_mfma_f32_16x16x32_bf16 v[124:127], a[188:191], v[156:159], v[124:127]\n" \
  "ds_read_b64_tr_b16 a[248:249], v212 offset:50176\n" \
  "v_mfma_f32_16x16x32_bf16 v[124:127], a[156:159], v[140:143], v[124:127]\n" \
  "ds_read_b64_tr_b16 a[250:251], v212 offset:50688\n" \
  "v_mfma_f32_16x16x32_bf16 v[120:123], a[156:159], v[136:139], v[120:123]\n" \
  "ds_read_b128 v[172:175], v208 offset:38912\n" \
  "v_mfma_f32_16x16x32_bf16 v[120:123], a[188:191], v[152:155], v[120:123]\n" \
  "ds_read_b64_tr_b16 a[252:253], v213 offset:50176\n" \
  "v_mfma_f32_16x16x32_bf16 v[104:107], a[184:187], v[152:155], v[104:107]\n" \
  "ds_read_b64_tr_b16 a[254:255], v213 offset:50688\n" \
  "v_mfma_f32_16x16x32_bf16 v[104:107], a[152:155], v[136:139], v[104:107]\n" \
  "ds_read_b128 v[188:191], v209 offset:38912\n" \
  "v_mfma_f32_16x16x32_bf16 v[100:103], a[152:155], v[132:135], v[100:103]\n" \
  "v_mfma_f32_16x16x32_bf16 v[100:103], a[184:187], v[148:151], v[100:103]\n" \
  "v_mfma_f32_16x16x32_bf16 v[116:119], a[188:191], v[148:151], v[116:119]\n" \
  "v_mfma_f32_16x16x32_bf16 v[116:119], a[156:159], v[132:135], v[116:119]\n" \
  "v_mfma_f32_16x16x32_bf16 v[112:115], a[156:159], v[128:131], v[112:115]\n" \
  "v_mfma_f32_16x16x32_bf16 v[112:115], a[188:191], v[144:147], v[112:115]\n" \
  "s_waitcnt vmcnt(63) lgkmcnt(0)\n" \
  "v_mfma_f32_16x16x32_bf16 v[96:99], a[184:187], v[144:147], v[96:99]\n" \
  "v_mfma_f32_16x16x32_bf16 v[96:99], a[152:155], v[128:131], v[96:99]\n" \
  "v_mfma_f32_16x16x32_bf16 v[0:3], a[192:195], v[160:163], v[0:3]\n" \
  "v_mfma_f32_16x16x32_bf16 v[0:3], a[224:227], v[176:179], v[0:3]\n" \
  "ds_read_b128 v[128:131], v208 offset:16384\n" \
  "v_mfma_f32_16x16x32_bf16 v[16:19], a[228:231], v[176:179], v[16:19]\n" \
  "ds_read_b128 v[144:147], v209 offset:16384\n" \
  "v_mfma_f32_16x16x32_bf16 v[16:19], a[196:199], v[160:163], v[16:19]\n" \
  "ds_read_b128 v[132:135], v208 offset:18432\n" \
  "v_mfma_f32_16x16x32_bf16 v[32:35], a[200:203], v[160:163], v[32:35]\n" \
  "ds_read_b128 v[148:151], v209 offset:18432\n" \
  "v_fma_f32 v200, v0, s48, v226\n" \
  "v_mfma_f32_16x16x32_bf16 v[32:35], a[232:235], v[176:179], v[32:35]\n" \
  "ds_read_b128 v[136:139], v208 offset:20480\n" \
  "v_fma_f32 v201, v1, s48, v226\n" \
  "v_mfma_f32_16x16x32_bf16 v[48:51], a[236:239], v[176:179], v[48:51]\n" \
  "ds_read_b128 v[152:155], v209 offset:20480\n" \
  "v_fma_f32 v202, v2, s48, v226\n" \
  "v_mfma_f32_16x16x32_bf16 v[48:51], a[204:207], v[160:163], v[48:51]\n" \
  "ds_read_b128 v[140:143], v208 offset:22528\n" \
  "v_fma_f32 v203, v3, s48, v226\n" \
  "v_mfma_f32_16x16x32_bf16 v[64:67], a[208:211], v[160:163], v[64:67]\n" \
  "ds_read_b128 v[156:159], v209 offset:22528\n" \
  "v_fma_f32 v204, v16, s48, v226\n" \
  "v_mfma_f32_16x16x32_bf16 v[64:67], a[240:243], v[176:179], v[64:67]\n" \
  "v_fma_f32 v205, v17, s48, v226\n" \
  "v_fma_f32 v206, v18, s48, v226\n" \
  "v_mfma_f32_16x16x32_bf16 v[80:83], a[244:247], v[176:179], v[80:83]\n" \
  "v_fma_f32 v207, v19, s48, v226\n" \
  "v_mfma_f32_16x16x32_bf16 v[80:83], a[212:215], v[160:163], v[80:83]\n" \
  "v_exp_f32 v200, v200 clamp\n" \
  "v_mfma_f32_16x16x32_bf16 v[96:99], a[216:219], v[160:163], v[96:99]\n" \
  "v_exp_f32 v201, v201 clamp\n" \
  "v_mfma_f32_16x16x32_bf16 v[96:99], a[248:251], v[176:179], v[96:99]\n" \
  "v_exp_f32 v202, v202 clamp\n" \
  "v_mfma_f32_16x16x32_bf16 v[112:115], a[252:255], v[176:179], v[112:115]\n" \
  "v_exp_f32 v203, v203 clamp\n" \
  "v_mfma_f32_16x16x32_bf16 v[112:115], a[220:223], v[160:163], v[112:115]\n" \
  "v_exp_f32 v204, v204 clamp\n" \
  "v_mfma_f32_16x16x32_bf16 v[116:119], a[220:223], v[164:167], v[116:119]\n" \
  "v_exp_f32 v205, v205 clamp\n" \
  "v_mfma_f32_16x16x32_bf16 v[116:119], a[252:255], v[180:183], v[116:119]\n" \
  "v_exp_f32 v206, v206 clamp\n" \
  "v_mfma_f32_16x16x32_bf16 v[100:103], a[248:251], v[180:183], v[100:103]\n" \
  "v_exp_f32 v207, v207 clamp\n" \
  "v_mfma_f32_16x16x32_bf16 v[100:103], a[216:219], v[164:167], v[100:103]\n" \
  "v_fma_f32 v200, v200, s50, 1.0\n" \
  "v_fma_f32 v201, v201, s50, 1.0\n" \
  "v_mfma_f32_16x16x32_bf16 v[84:87], a[212:215], v[164:167], v[84:87]\n" \
  "v_fma_f32 v202, v202, s50, 1.0\n" \
  "v_fma_f32 v203, v203, s50, 1.0\n" \
  "v_mfma_f32_16x16x32_bf16 v[84:87], a[244:247], v[180:183], v[84:87]\n" \
  "v_fma_f32 v204, v204, s50, 1.0\n" \
  "v_fma_f32 v205, v205, s50, 1.0\n" \
  "v_mfma_f32_16x16x32_bf16 v[68:71], a[240:243], v[180:183], v[68:71]\n" \
  "v_fma_f32 v206, v206, s50, 1.0\n" \
  "v_fma_f32 v207, v207, s50, 1.0\n" \
  "v_mfma_f32_16x16x32_bf16 v[68:71], a[208:211], v[164:167], v[68:71]\n" \
  "v_rcp_f32 v200, v200\n" \
  "v_mfma_f32_16x16x32_bf16 v[52:55], a[204:207], v[164:167], v[52:55]\n" \
  "v_rcp_f32 v201, v201\n" \
  "v_mfma_f32_16x16x32_bf16 v[52:55], a[236:239], v[180:183], v[52:55]\n" \
  "v_rcp_f32 v202, v202\n" \
  "v_mfma_f32_16x16x32_bf16 v[36:39], a[232:235], v[180:183], v[36:39]\n" \
  "v_rcp_f32 v203, v203\n" \
  "v_mfma_f32_16x16x32_bf16 v[36:39], a[200:203], v[164:167], v[36:39]\n" \
  "v_rcp_f32 v204, v204\n" \
  "v_mfma_f32_16x16x32_bf16 v[20:23], a[196:199], v[164:167], v[20:23]\n" \
  "v_rcp_f32 v205, v205\n" \
  "v_mfma_f32_16x16x32_bf16 v[20:23], a[228:231], v[180:183], v[20:23]\n" \
  "v_rcp_f32 v206, v206\n" \
  "v_mfma_f32_16x16x32_bf16 v[4:7], a[224:227], v[180:183], v[4:7]\n" \
  "v_rcp_f32 v207, v207\n" \
  "v_mfma_f32_16x16x32_bf16 v[4:7], a[192:195], v[164:167], v[4:7]\n" \
  "v_mul_f32 v200, v200, v0\n" \
  "v_mul_f32 v201, v201, v1\n" \
  "v_mfma_f32_16x16x32_bf16 v[8:11], a[192:195], v[168:171], v[8:11]\n" \
  "v_mul_f32 v202, v202, v2\n" \
  "v_mul_f32 v203, v203, v3\n" \
  "v_mfma_f32_16x16x32_bf16 v[8:11], a[224:227], v[184:187], v[8:11]\n" \
  "v_mul_f32 v204, v204, v16\n" \
  "v_mul_f32 v205, v205, v17\n" \
  "v_mfma_f32_16x16x32_bf16 v[24:27], a[228:231], v[184:187], v[24:27]\n" \
  "v_mul_f32 v206, v206, v18\n" \
  "v_mul_f32 v207, v207, v19\n" \
  "v_mfma_f32_16x16x32_bf16 v[24:27], a[196:199], v[168:171], v[24:27]\n" \
  "v_cvt_pk_bf16_f32 v192, v200, v201\n" \
  "v_cvt_pk_bf16_f32 v193, v202, v203\n" \
  "v_mfma_f32_16x16x32_bf16 v[40:43], a[200:203], v[168:171], v[40:43]\n" \
  "v_cvt_pk_bf16_f32 v194, v204, v205\n" \
  "v_cvt_pk_bf16_f32 v195, v206, v207\n" \
  "v_mfma_f32_16x16x32_bf16 v[40:43], a[232:235], v[184:187], v[40:43]\n" \
  "v_fma_f32 v248, v32, s48, v226\n" \
  "v_fma_f32 v249, v33, s48, v226\n" \
  "v_mfma_f32_16x16x32_bf16 v[56:59], a[236:239], v[184:187], v[56:59]\n" \
  "v_fma_f32 v250, v34, s48, v226\n" \
  "v_fma_f32 v251, v35, s48, v226\n" \
  "v_mfma_f32_16x16x32_bf16 v[56:59], a[204:207], v[168:171], v[56:59]\n" \
  "v_fma_f32 v252, v48, s48, v226\n" \
  "v_fma_f32 v253, v49, s48, v226\n" \
  "v_mfma_f32_16x16x32_bf16 v[72:75], a[208:211], v[168:171], v[72:75]\n" \
  "s_waitcnt vmcnt(0) lgkmcnt(0)\n" \
  "v_fma_f32 v254, v50, s48, v226\n" \
  "v_mfma_f32_16x16x32_bf16 v[72:75], a[240:243], v[184:187], v[72:75]\n" \
  "s_barrier\n" \
  "v_readfirstlane_b32 s70, v220\n" \
  "v_mfma_f32_16x16x32_bf16 v[88:91], a[244:247], v[184:187], v[88:91]\n" \
  "s_cmp_eq_u32 s70, s72\n" \
  "v_fma_f32 v255, v51, s48, v226\n" \
  "v_mfma_f32_16x16x32_bf16 v[88:91], a[212:215], v[168:171], v[88:91]\n" \
  "s_cbranch_scc1 LK_nan\n" \
  "v_mfma_f32_16x16x32_bf16 v[104:107], a[216:219], v[168:171], v[104:107]\n" \
  "v_permlane16_swap_b32 v192, v194\n" \
  "v_mfma_f32_16x16x32_bf16 v[104:107], a[248:251], v[184:187], v[104:107]\n" \
  "v_permlane16_swap_b32 v193, v195\n" \
  "v_mfma_f32_16x16x32_bf16 v[120:123], a[252:255], v[184:187], v[120:123]\n" \
  "buffer_store_dwordx4 v[192:195], v215, s[44:47], s63 offen offset:0 sc1 nt\n" \
  "v_mfma_f32_16x16x32_bf16 v[120:123], a[220:223], v[168:171], v[120:123]\n" \
  "v_exp_f32 v248, v248 clamp\n" \
  "v_mfma_f32_16x16x32_bf16 v[124:127], a[220:223], v[172:175], v[124:127]\n" \
  "v_exp_f32 v249, v249 clamp\n" \
  "v_mfma_f32_16x16x32_bf16 v[124:127], a[252:255], v[188:191], v[124:127]\n" \
  "v_exp_f32 v250, v250 clamp\n" \
  "v_mfma_f32_16x16x32_bf16 v[108:111], a[248:251], v[188:191], v[108:111]\n" \
  "v_exp_f32 v251, v251 clamp\n" \
  "v_mfma_f32_16x16x32_bf16 v[108:111], a[216:219], v[172:175], v[108:111]\n" \
  "v_exp_f32 v252, v252 clamp\n" \
  "v_mfma_f32_16x16x32_bf16 v[92:95], a[212:215], v[172:175], v[92:95]\n" \
  "v_exp_f32 v253, v253 clamp\n" \
  "v_mfma_f32_16x16x32_bf16 v[92:95], a[244:247], v[188:191], v[92:95]\n" \
  "v_exp_f32 v254, v254 clamp\n" \
  "v_mfma_f32_16x16x32_bf16 v[76:79], a[240:243], v[188:191], v[76:79]\n" \
  "v_exp_f32 v255, v255 clamp\n" \
  "v_mfma_f32_16x16x32_bf16 v[76:79], a[208:211], v[172:175], v[76:79]\n" \
  "v_fma_f32 v248, v248, s50, 1.0\n" \
  "v_fma_f32 v249, v249, s50, 1.0\n" \
  "v_mfma_f32_16x16x32_bf16 v[60:63], a[204:207], v[172:175], v[60:63]\n" \
  "v_fma_f32 v250, v250, s50, 1.0\n" \
  "v_fma_f32 v251, v251, s50, 1.0\n" \
  "v_mfma_f32_16x16x32_bf16 v[60:63], a[236:239], v[188:191], v[60:63]\n" \
  "v_fma_f32 v252, v252, s50, 1.0\n" \
  "v_fma_f32 v253, v253, s50, 1.0\n" \
  "v_mfma_f32_16x16x32_bf16 v[44:47], a[232:235], v[188:191], v[44:47]\n" \
  "v_fma_f32 v254, v254, s50, 1.0\n" \
  "v_fma_f32 v255, v255, s50, 1.0\n" \
  "v_mfma_f32_16x16x32_bf16 v[44:47], a[200:203], v[172:175], v[44:47]\n" \
  "v_rcp_f32 v248, v248\n" \
  "v_mfma_f32_16x16x32_bf16 v[28:31], a[196:199], v[172:175], v[28:31]\n" \
  "v_rcp_f32 v249, v249\n" \
  "v_mfma_f32_16x16x32_bf16 v[28:31], a[228:231], v[188:191], v[28:31]\n" \
  "v_rcp_f32 v250, v250\n" \
  "v_mfma_f32_16x16x32_bf16 v[12:15], a[224:227], v[188:191], v[12:15]\n" \
  "v_rcp_f32 v251, v251\n" \
  "v_mfma_f32_16x16x32_bf16 v[12:15], a[192:195], v[172:175], v[12:15]\n" \
  "v_rcp_f32 v252, v252\n" \
  "v_rcp_f32 v253, v253\n" \
  "v_rcp_f32 v254, v254\n" \
  "v_rcp_f32 v255, v255\n" \
  "v_pk_mul_f32 v[248:249], v[248:249], v[32:33]\n" \
  "v_pk_mul_f32 v[250:251], v[250:251], v[34:35]\n" \
  "v_pk_mul_f32 v[252:253], v[252:253], v[48:49]\n" \
  "v_pk_mul_f32 v[254:255], v[254:255], v[50:51]\n" \
  "v_cvt_pk_bf16_f32 v240, v248, v249\n" \
  "v_cvt_pk_bf16_f32 v241, v250, v251\n" \
  "v_cvt_pk_bf16_f32 v242, v252, v253\n" \
  "v_cvt_pk_bf16_f32 v243, v254, v255\n" \
  "v_pk_fma_f32 v[200:201], v[64:65], s[48:49], v[226:227] op_sel_hi:[1,0,0]\n" \
  "v_pk_fma_f32 v[202:203], v[66:67], s[48:49], v[226:227] op_sel_hi:[1,0,0]\n" \
  "v_pk_fma_f32 v[204:205], v[80:81], s[48:49], v[226:227] op_sel_hi:[1,0,0]\n" \
  "v_pk_fma_f32 v[206:207], v[82:83], s[48:49], v[226:227] op_sel_hi:[1,0,0]\n" \
  "v_permlane16_swap_b32 v240, v242\n" \
  "v_permlane16_swap_b32 v241, v243\n" \
  "buffer_store_dwordx4 v[240:243], v215, s[44:47], s63 offen offset:64 sc1 nt\n" \
  "v_exp_f32 v200, v200 clamp\n" \
  "v_exp_f32 v201, v201 clamp\n" \
  "v_exp_f32 v202, v202 clamp\n" \
  "v_exp_f32 v203, v203 clamp\n" \
  "v_exp_f32 v204, v204 clamp\n" \
  "v_exp_f32 v205, v205 clamp\n" \
  "v_exp_f32 v206, v206 clamp\n" \
  "v_exp_f32 v207, v207 clamp\n" \
  "v_pk_fma_f32 v[200:201], v[200:201], s[50:51], 1.0 op_sel_hi:[1,0,0]\n" \
  "v_pk_fma_f32 v[202:203], v[202:203], s[50:51], 1.0 op_sel_hi:[1,0,0]\n" \
  "v_pk_fma_f32 v[204:205], v[204:205], s[50:51], 1.0 op_sel_hi:[1,0,0]\n" \
  "v_pk_fma_f32 v[206:207], v[206:207], s[50:51], 1.0 op_sel_hi:[1,0,0]\n" \
  "v_rcp_f32 v200, v200\n" \
  "v_rcp_f32 v201, v201\n" \
  "v_rcp_f32 v202, v202\n" \
  "v_rcp_f32 v203, v203\n" \
  "v_rcp_f32 v204, v204\n" \
  "v_rcp_f32 v205, v205\n" \
  "v_rcp_f32 v206, v206\n" \
  "v_rcp_f32 v207, v207\n" \
  "v_pk_mul_f32 v[200:201], v[200:201], v[64:65]\n" \
  "v_pk_mul_f32 v[202:203], v[202:203], v[66:67]\n" \
  "v_pk_mul_f32 v[204:205], v[204:205], v[80:81]\n" \
  "v_pk_mul_f32 v[206:207], v[206:207], v[82:83]\n" \
  "v_cvt_pk_bf16_f32 v192, v200, v201\n" \
  "v_cvt_pk_bf16_f32 v193, v202, v203\n" \
  "v_cvt_pk_bf16_f32 v194, v204, v205\n" \
  "v_cvt_pk_bf16_f32 v195, v206, v207\n" \
  "v_pk_fma_f32 v[248:249], v[96:97], s[48:49], v[226:227] op_sel_hi:[1,0,0]\n" \
  "v_pk_fma_f32 v[250:251], v[98:99], s[48:49], v[226:227] op_sel_hi:[1,0,0]\n" \
  "v_pk_fma_f32 v[252:253], v[112:113], s[48:49], v[226:227] op_sel_hi:[1,0,0]\n" \
  "v_pk_fma_f32 v[254:255], v[114:115], s[48:49], v[226:227] op_sel_hi:[1,0,0]\n" \
  "v_permlane16_swap_b32 v192, v194\n" \
  "v_permlane16_swap_b32 v193, v195\n" \
  "buffer_store_dwordx4 v[192:195], v215, s[44:47], s63 offen offset:128 sc1 nt\n" \
  "v_exp_f32 v248, v248 clamp\n" \
  "v_exp_f32 v249, v249 clamp\n" \
  "v_exp_f32 v250, v250 clamp\n" \
  "v_exp_f32 v251, v251 clamp\n" \
  "v_exp_f32 v252, v252 clamp\n" \
  "v_exp_f32 v253, v253 clamp\n" \
  "v_exp_f32 v254, v254 clamp\n" \
  "v_exp_f32 v255, v255 clamp\n" \
  "v_pk_fma_f32 v[248:249], v[248:249], s[50:51], 1.0 op_sel_hi:[1,0,0]\n" \
  "v_pk_fma_f32 v[250:251], v[250:251], s[50:51], 1.0 op_sel_hi:[1,0,0]\n" \
  "v_pk_fma_f32 v[252:253], v[252:253], s[50:51], 1.0 op_sel_hi:[1,0,0]\n" \
  "v_pk_fma_f32 v[254:255], v[254:255], s[50:51], 1.0 op_sel_hi:[1,0,0]\n" \
  "v_rcp_f32 v248, v248\n" \
  "v_rcp_f32 v249, v249\n" \
  "v_rcp_f32 v250, v250\n" \
  "v_rcp_f32 v251, v251\n" \
  "v_rcp_f32 v252, v252\n" \
  "v_rcp_f32 v253, v253\n" \
  "v_rcp_f32 v254, v254\n" \
  "v_rcp_f32 v255, v255\n" \
  "v_pk_mul_f32 v[248:249], v[248:249], v[96:97]\n" \
  "v_pk_mul_f32 v[250:251], v[250:251], v[98:99]\n" \
  "v_pk_mul_f32 v[252:253], v[252:253], v[112:113]\n" \
  "v_pk_mul_f32 v[254:255], v[254:255], v[114:115]\n" \
  "v_cvt_pk_bf16_f32 v240, v248, v249\n" \
  "v_cvt_pk_bf16_f32 v241, v250, v251\n" \
  "v_cvt_pk_bf16_f32 v242, v252, v253\n" \
  "v_cvt_pk_bf16_f32 v243, v254, v255\n" \
  "v_pk_fma_f32 v[200:201], v[4:5], s[48:49], v[226:227] op_sel_hi:[1,0,0]\n" \
  "v_pk_fma_f32 v[202:203], v[6:7], s[48:49], v[226:227] op_sel_hi:[1,0,0]\n" \
  "v_pk_fma_f32 v[204:205], v[20:21], s[48:49], v[226:227] op_sel_hi:[1,0,0]\n" \
  "v_pk_fma_f32 v[206:207], v[22:23], s[48:49], v[226:227] op_sel_hi:[1,0,0]\n" \
  "v_permlane16_swap_b32 v240, v242\n" \
  "v_permlane16_swap_b32 v241, v243\n" \
  "buffer_store_dwordx4 v[240:243], v215, s[44:47], s63 offen offset:192 sc1 nt\n" \
  "s_add_u32 s63, s63, 131072\n" \
  "v_exp_f32 v200, v200 clamp\n" \
  "v_exp_f32 v201, v201 clamp\n" \
  "v_exp_f32 v202, v202 clamp\n" \
  "v_exp_f32 v203, v203 clamp\n" \
  "v_exp_f32 v204, v204 clamp\n" \
  "v_exp_f32 v205, v205 clamp\n" \
  "v_exp_f32 v206, v206 clamp\n" \
  "v_exp_f32 v207, v207 clamp\n" \
  "v_pk_fma_f32 v[200:201], v[200:201], s[50:51], 1.0 op_sel_hi:[1,0,0]\n" \
  "v_pk_fma_f32 v[202:203], v[202:203], s[50:51], 1.0 op_sel_hi:[1,0,0]\n" \
  "v_pk_fma_f32 v[204:205], v[204:205], s[50:51], 1.0 op_sel_hi:[1,0,0]\n" \
  "v_pk_fma_f32 v[206:207], v[206:207], s[50:51], 1.0 op_sel_hi:[1,0,0]\n" \
  "v_rcp_f32 v200, v200\n" \
  "v_rcp_f32 v201, v201\n" \
  "v_rcp_f32 v202, v202\n" \
  "v_rcp_f32 v203, v203\n" \
  "v_rcp_f32 v204, v204\n" \
  "v_rcp_f32 v205, v205\n" \
  "v_rcp_f32 v206, v206\n" \
  "v_rcp_f32 v207, v207\n" \
  "v_pk_mul_f32 v[200:201], v[200:201], v[4:5]\n" \
  "v_pk_mul_f32 v[202:203], v[202:203], v[6:7]\n" \
  "v_pk_mul_f32 v[204:205], v[204:205], v[20:21]\n" \
  "v_pk_mul_f32 v[206:207], v[206:207], v[22:23]\n" \
  "v_cvt_pk_bf16_f32 v192, v200, v201\n" \
  "v_cvt_pk_bf16_f32 v193, v202, v203\n" \
  "v_cvt_pk_bf16_f32 v194, v204, v205\n" \
  "v_cvt_pk_bf16_f32 v195, v206, v207\n" \
  "v_pk_fma_f32 v[248:249], v[36:37], s[48:49], v[226:227] op_sel_hi:[1,0,0]\n" \
  "v_pk_fma_f32 v[250:251], v[38:39], s[48:49], v[226:227] op_sel_hi:[1,0,0]\n" \
  "v_pk_fma_f32 v[252:253], v[52:53], s[48:49], v[226:227] op_sel_hi:[1,0,0]\n" \
  "v_pk_fma_f32 v[254:255], v[54:55], s[48:49], v[226:227] op_sel_hi:[1,0,0]\n" \
  "v_permlane16_swap_b32 v192, v194\n" \
  "v_permlane16_swap_b32 v193, v195\n" \
  "buffer_store_dwordx4 v[192:195], v215, s[44:47], s63 offen offset:0 sc1 nt\n" \
  "v_exp_f32 v248, v248 clamp\n" \
  "v_exp_f32 v249, v249 clamp\n" \
  "v_exp_f32 v250, v250 clamp\n" \
  "v_exp_f32 v251, v251 clamp\n" \
  "v_exp_f32 v252, v252 clamp\n" \
  "v_exp_f32 v253, v253 clamp\n" \
  "v_exp_f32 v254, v254 clamp\n" \
  "v_exp_f32 v255, v255 clamp\n" \
  "v_pk_fma_f32 v[248:249], v[248:249], s[50:51], 1.0 op_sel_hi:[1,0,0]\n" \
  "v_pk_fma_f32 v[250:251], v[250:251], s[50:51], 1.0 op_sel_hi:[1,0,0]\n" \
  "v_pk_fma_f32 v[252:253], v[252:253], s[50:51], 1.0 op_sel_hi:[1,0,0]\n" \
  "v_pk_fma_f32 v[254:255], v[254:255], s[50:51], 1.0 op_sel_hi:[1,0,0]\n" \
  "v_rcp_f32 v248, v248\n" \
  "v_rcp_f32 v249, v249\n" \
  "v_rcp_f32 v250, v250\n" \
  "v_rcp_f32 v251, v251\n" \
  "v_rcp_f32 v252, v252\n" \
  "v_rcp_f32 v253, v253\n" \
  "v_rcp_f32 v254, v254\n" \
  "v_rcp_f32 v255, v255\n" \
  "v_pk_mul_f32 v[248:249], v[248:249], v[36:37]\n" \
  "v_pk_mul_f32 v[250:251], v[250:251], v[38:39]\n" \
  "v_pk_mul_f32 v[252:253], v[252:253], v[52:53]\n" \
  "v_pk_mul_f32 v[254:255], v[254:255], v[54:55]\n" \
  "v_cvt_pk_bf16_f32 v240, v248, v249\n" \
  "v_cvt_pk_bf16_f32 v241, v250, v251\n" \
  "v_cvt_pk_bf16_f32 v242, v252, v253\n" \
  "v_cvt_pk_bf16_f32 v243, v254, v255\n" \
  "v_pk_fma_f32 v[200:201], v[68:69], s[48:49], v[226:227] op_sel_hi:[1,0,0]\n" \
  "v_pk_fma_f32 v[202:203], v[70:71], s[48:49], v[226:227] op_sel_hi:[1,0,0]\n" \
  "v_pk_fma_f32 v[204:205], v[84:85], s[48:49], v[226:227] op_sel_hi:[1,0,0]\n" \
  "v_pk_fma_f32 v[206:207], v[86:87], s[48:49], v[226:227] op_sel_hi:[1,0,0]\n" \
  "v_permlane16_swap_b32 v240, v242\n" \
  "v_permlane16_swap_b32 v241, v243\n" \
  "buffer_store_dwordx4 v[240:243], v215, s[44:47], s63 offen offset:64 sc1 nt\n" \
  "v_exp_f32 v200, v200 clamp\n" \
  "v_exp_f32 v201, v201 clamp\n" \
  "v_exp_f32 v202, v202 clamp\n" \
  "v_exp_f32 v203, v203 clamp\n" \
  "v_exp_f32 v204, v204 clamp\n" \
  "v_exp_f32 v205, v205 clamp\n" \
  "v_exp_f32 v206, v206 clamp\n" \
  "v_exp_f32 v207, v207 clamp\n" \
  "v_pk_fma_f32 v[200:201], v[200:201], s[50:51], 1.0 op_sel_hi:[1,0,0]\n" \
  "v_pk_fma_f32 v[202:203], v[202:203], s[50:51], 1.0 op_sel_hi:[1,0,0]\n" \
  "v_pk_fma_f32 v[204:205], v[204:205], s[50:51], 1.0 op_sel_hi:[1,0,0]\n" \
  "v_pk_fma_f32 v[206:207], v[206:207], s[50:51], 1.0 op_sel_hi:[1,0,0]\n" \
  "v_rcp_f32 v200, v200\n" \
  "v_rcp_f32 v201, v201\n" \
  "v_rcp_f32 v202, v202\n" \
  "v_rcp_f32 v203, v203\n" \
  "v_rcp_f32 v204, v204\n" \
  "v_rcp_f32 v205, v205\n" \
  "v_rcp_f32 v206, v206\n" \
  "v_rcp_f32 v207, v207\n" \
  "v_pk_mul_f32 v[200:201], v[200:201], v[68:69]\n" \
  "v_pk_mul_f32 v[202:203], v[202:203], v[70:71]\n" \
  "v_pk_mul_f32 v[204:205], v[204:205], v[84:85]\n" \
  "v_pk_mul_f32 v[206:207], v[206:207], v[86:87]\n" \
  "v_cvt_pk_bf16_f32 v192, v200, v201\n" \
  "v_cvt_pk_bf16_f32 v193, v202, v203\n" \
  "v_cvt_pk_bf16_f32 v194, v204, v205\n" \
  "v_cvt_pk_bf16_f32 v195, v206, v207\n" \
  "v_pk_fma_f32 v[248:249], v[100:101], s[48:49], v[226:227] op_sel_hi:[1,0,0]\n" \
  "v_pk_fma_f32 v[250:251], v[102:103], s[48:49], v[226:227] op_sel_hi:[1,0,0]\n" \
  "v_pk_fma_f32 v[252:253], v[116:117], s[48:49], v[226:227] op_sel_hi:[1,0,0]\n" \
  "v_pk_fma_f32 v[254:255], v[118:119], s[48:49], v[226:227] op_sel_hi:[1,0,0]\n" \
  "v_permlane16_swap_b32 v192, v194\n" \
  "v_permlane16_swap_b32 v193, v195\n" \
  "buffer_store_dwordx4 v[192:195], v215, s[44:47], s63 offen offset:128 sc1 nt\n" \
  "v_exp_f32 v248, v248 clamp\n" \
  "v_exp_f32 v249, v249 clamp\n" \
  "v_exp_f32 v250, v250 clamp\n" \
  "v_exp_f32 v251, v251 clamp\n" \
  "v_exp_f32 v252, v252 clamp\n" \
  "v_exp_f32 v253, v253 clamp\n" \
  "v_exp_f32 v254, v254 clamp\n" \
  "v_exp_f32 v255, v255 clamp\n" \
  "v_pk_fma_f32 v[248:249], v[248:249], s[50:51], 1.0 op_sel_hi:[1,0,0]\n" \
  "v_pk_fma_f32 v[250:251], v[250:251], s[50:51], 1.0 op_sel_hi:[1,0,0]\n" \
  "v_pk_fma_f32 v[252:253], v[252:253], s[50:51], 1.0 op_sel_hi:[1,0,0]\n" \
  "v_pk_fma_f32 v[254:255], v[254:255], s[50:51], 1.0 op_sel_hi:[1,0,0]\n" \
  "v_rcp_f32 v248, v248\n" \
  "v_rcp_f32 v249, v249\n" \
  "v_rcp_f32 v250, v250\n" \
  "v_rcp_f32 v251, v251\n" \
  "v_rcp_f32 v252, v252\n" \
  "v_rcp_f32 v253, v253\n" \
  "v_rcp_f32 v254, v254\n" \
  "v_rcp_f32 v255, v255\n" \
  "v_pk_mul_f32 v[248:249], v[248:249], v[100:101]\n" \
  "v_pk_mul_f32 v[250:251], v[250:251], v[102:103]\n" \
  "v_pk_mul_f32 v[252:253], v[252:253], v[116:117]\n" \
  "v_pk_mul_f32 v[254:255], v[254:255], v[118:119]\n" \
  "v_cvt_pk_bf16_f32 v240, v248, v249\n" \
  "v_cvt_pk_bf16_f32 v241, v250, v251\n" \
  "v_cvt_pk_bf16_f32 v242, v252, v253\n" \
  "v_cvt_pk_bf16_f32 v243, v254, v255\n" \
  "v_pk_fma_f32 v[200:201], v[8:9], s[48:49], v[226:227] op_sel_hi:[1,0,0]\n" \
  "v_pk_fma_f32 v[202:203], v[10:11], s[48:49], v[226:227] op_sel_hi:[1,0,0]\n" \
  "v_pk_fma_f32 v[204:205], v[24:25], s[48:49], v[226:227] op_sel_hi:[1,0,0]\n" \
  "v_pk_fma_f32 v[206:207], v[26:27], s[48:49], v[226:227] op_sel_hi:[1,0,0]\n" \
  "v_permlane16_swap_b32 v240, v242\n" \
  "v_permlane16_swap_b32 v241, v243\n" \
  "buffer_store_dwordx4 v[240:243], v215, s[44:47], s63 offen offset:192 sc1 nt\n" \
  "s_add_u32 s63, s63, 131072\n" \
  "v_exp_f32 v200, v200 clamp\n" \
  "v_exp_f32 v201, v201 clamp\n" \
  "v_exp_f32 v202, v202 clamp\n" \
  "v_exp_f32 v203, v203 clamp\n" \
  "v_exp_f32 v204, v204 clamp\n" \
  "v_exp_f32 v205, v205 clamp\n" \
  "v_exp_f32 v206, v206 clamp\n" \
  "v_exp_f32 v207, v207 clamp\n" \
  "v_pk_fma_f32 v[200:201], v[200:201], s[50:51], 1.0 op_sel_hi:[1,0,0]\n" \
  "v_pk_fma_f32 v[202:203], v[202:203], s[50:51], 1.0 op_sel_hi:[1,0,0]\n" \
  "v_pk_fma_f32 v[204:205], v[204:205], s[50:51], 1.0 op_sel_hi:[1,0,0]\n" \
  "v_pk_fma_f32 v[206:207], v[206:207], s[50:51], 1.0 op_sel_hi:[1,0,0]\n" \
  "v_rcp_f32 v200, v200\n" \
  "v_rcp_f32 v201, v201\n" \
  "v_rcp_f32 v202, v202\n" \
  "v_rcp_f32 v203, v203\n" \
  "v_rcp_f32 v204, v204\n" \
  "v_rcp_f32 v205, v205\n" \
  "v_rcp_f32 v206, v206\n" \
  "v_rcp_f32 v207, v207\n" \
  "v_pk_mul_f32 v[200:201], v[200:201], v[8:9]\n" \
  "v_pk_mul_f32 v[202:203], v[202:203], v[10:11]\n" \
  "v_pk_mul_f32 v[204:205], v[204:205], v[24:25]\n" \
  "v_pk_mul_f32 v[206:207], v[206:207], v[26:27]\n" \
  "v_cvt_pk_bf16_f32 v192, v200, v201\n" \
  "v_cvt_pk_bf16_f32 v193, v202, v203\n" \
  "v_cvt_pk_bf16_f32 v194, v204, v205\n" \
  "v_cvt_pk_bf16_f32 v195, v206, v207\n" \
  "v_pk_fma_f32 v[248:249], v[40:41], s[48:49], v[226:227] op_sel_hi:[1,0,0]\n" \
  "v_pk_fma_f32 v[250:251], v[42:43], s[48:49], v[226:227] op_sel_hi:[1,0,0]\n" \
  "v_pk_fma_f32 v[252:253], v[56:57], s[48:49], v[226:227] op_sel_hi:[1,0,0]\n" \
  "v_pk_fma_f32 v[254:255], v[58:59], s[48:49], v[226:227] op_sel_hi:[1,0,0]\n" \
  "v_permlane16_swap_b32 v192, v194\n" \
  "v_permlane16_swap_b32 v193, v195\n" \
  "buffer_store_dwordx4 v[192:195], v215, s[44:47], s63 offen offset:0 sc1 nt\n" \
  "v_exp_f32 v248, v248 clamp\n" \
  "v_exp_f32 v249, v249 clamp\n" \
  "v_exp_f32 v250, v250 clamp\n" \
  "v_exp_f32 v251, v251 clamp\n" \
  "v_exp_f32 v252, v252 clamp\n" \
  "v_exp_f32 v253, v253 clamp\n" \
  "v_exp_f32 v254, v254 clamp\n" \
  "v_exp_f32 v255, v255 clamp\n" \
  "v_pk_fma_f32 v[248:249], v[248:249], s[50:51], 1.0 op_sel_hi:[1,0,0]\n" \
  "v_pk_fma_f32 v[250:251], v[250:251], s[50:51], 1.0 op_sel_hi:[1,0,0]\n" \
  "v_pk_fma_f32 v[252:253], v[252:253], s[50:51], 1.0 op_sel_hi:[1,0,0]\n" \
  "v_pk_fma_f32 v[254:255], v[254:255], s[50:51], 1.0 op_sel_hi:[1,0,0]\n" \
  "v_rcp_f32 v248, v248\n" \
  "v_rcp_f32 v249, v249\n" \
  "v_rcp_f32 v250, v250\n" \
  "v_rcp_f32 v251, v251\n" \
  "v_rcp_f32 v252, v252\n" \
  "v_rcp_f32 v253, v253\n" \
  "v_rcp_f32 v254, v254\n" \
  "v_rcp_f32 v255, v255\n" \
  "v_pk_mul_f32 v[248:249], v[248:249], v[40:41]\n" \
  "v_pk_mul_f32 v[250:251], v[250:251], v[42:43]\n" \
  "v_pk_mul_f32 v[252:253], v[252:253], v[56:57]\n" \
  "v_pk_mul_f32 v[254:255], v[254:255], v[58:59]\n" \
  "v_cvt_pk_bf16_f32 v240, v248, v249\n" \
  "v_cvt_pk_bf16_f32 v241, v250, v251\n" \
  "v_cvt_pk_bf16_f32 v242, v252, v253\n" \
  "v_cvt_pk_bf16_f32 v243, v254, v255\n" \
  "v_pk_fma_f32 v[200:201], v[72:73], s[48:49], v[226:227] op_sel_hi:[1,0,0]\n" \
  "v_pk_fma_f32 v[202:203], v[74:75], s[48:49], v[226:227] op_sel_hi:[1,0,0]\n" \
  "v_pk_fma_f32 v[204:205], v[88:89], s[48:49], v[226:227] op_sel_hi:[1,0,0]\n" \
  "v_pk_fma_f32 v[206:207], v[90:91], s[48:49], v[226:227] op_sel_hi:[1,0,0]\n" \
  "v_permlane16_swap_b32 v240, v242\n" \
  "v_permlane16_swap_b32 v241, v243\n" \
  "buffer_store_dwordx4 v[240:243], v215, s[44:47], s63 offen offset:64 sc1 nt\n" \
  "v_exp_f32 v200, v200 clamp\n" \
  "v_exp_f32 v201, v201 clamp\n" \
  "v_exp_f32 v202, v202 clamp\n" \
  "v_exp_f32 v203, v203 clamp\n" \
  "v_exp_f32 v204, v204 clamp\n" \
  "v_exp_f32 v205, v205 clamp\n" \
  "v_exp_f32 v206, v206 clamp\n" \
  "v_exp_f32 v207, v207 clamp\n" \
  "v_pk_fma_f32 v[200:201], v[200:201], s[50:51], 1.0 op_sel_hi:[1,0,0]\n" \
  "v_pk_fma_f32 v[202:203], v[202:203], s[50:51], 1.0 op_sel_hi:[1,0,0]\n" \
  "v_pk_fma_f32 v[204:205], v[204:205], s[50:51], 1.0 op_sel_hi:[1,0,0]\n" \
  "v_pk_fma_f32 v[206:207], v[206:207], s[50:51], 1.0 op_sel_hi:[1,0,0]\n" \
  "v_rcp_f32 v200, v200\n" \
  "v_rcp_f32 v201, v201\n" \
  "v_rcp_f32 v202, v202\n" \
  "v_rcp_f32 v203, v203\n" \
  "v_rcp_f32 v204, v204\n" \
  "v_rcp_f32 v205, v205\n" \
  "v_rcp_f32 v206, v206\n" \
  "v_rcp_f32 v207, v207\n" \
  "v_pk_mul_f32 v[200:201], v[200:201], v[72:73]\n" \
  "v_pk_mul_f32 v[202:203], v[202:203], v[74:75]\n" \
  "v_pk_mul_f32 v[204:205], v[204:205], v[88:89]\n" \
  "v_pk_mul_f32 v[206:207], v[206:207], v[90:91]\n" \
  "v_cvt_pk_bf16_f32 v192, v200, v201\n" \
  "v_cvt_pk_bf16_f32 v193, v202, v203\n" \
  "v_cvt_pk_bf16_f32 v194, v204, v205\n" \
  "v_cvt_pk_bf16_f32 v195, v206, v207\n" \
  "v_pk_fma_f32 v[248:249], v[104:105], s[48:49], v[226:227] op_sel_hi:[1,0,0]\n" \
  "v_pk_fma_f32 v[250:251], v[106:107], s[48:49], v[226:227] op_sel_hi:[1,0,0]\n" \
  "v_pk_fma_f32 v[252:253], v[120:121], s[48:49], v[226:227] op_sel_hi:[1,0,0]\n" \
  "v_pk_fma_f32 v[254:255], v[122:123], s[48:49], v[226:227] op_sel_hi:[1,0,0]\n" \
  "v_permlane16_swap_b32 v192, v194\n" \
  "v_permlane16_swap_b32 v193, v195\n" \
  "buffer_store_dwordx4 v[192:195], v215, s[44:47], s63 offen offset:128 sc1 nt\n" \
  "v_exp_f32 v248, v248 clamp\n" \
  "v_exp_f32 v249, v249 clamp\n" \
  "v_exp_f32 v250, v250 clamp\n" \
  "v_exp_f32 v251, v251 clamp\n" \
  "v_exp_f32 v252, v252 clamp\n" \
  "v_exp_f32 v253, v253 clamp\n" \
  "v_exp_f32 v254, v254 clamp\n" \
  "v_exp_f32 v255, v255 clamp\n" \
  "v_pk_fma_f32 v[248:249], v[248:249], s[50:51], 1.0 op_sel_hi:[1,0,0]\n" \
  "v_pk_fma_f32 v[250:251], v[250:251], s[50:51], 1.0 op_sel_hi:[1,0,0]\n" \
  "v_pk_fma_f32 v[252:253], v[252:253], s[50:51], 1.0 op_sel_hi:[1,0,0]\n" \
  "v_pk_fma_f32 v[254:255], v[254:255], s[50:51], 1.0 op_sel_hi:[1,0,0]\n" \
  "v_rcp_f32 v248, v248\n" \
  "v_rcp_f32 v249, v249\n" \
  "v_rcp_f32 v250, v250\n" \
  "v_rcp_f32 v251, v251\n" \
  "v_rcp_f32 v252, v252\n" \
  "v_rcp_f32 v253, v253\n" \
  "v_rcp_f32 v254, v254\n" \
  "v_rcp_f32 v255, v255\n" \
  "v_pk_mul_f32 v[248:249], v[248:249], v[104:105]\n" \
  "v_pk_mul_f32 v[250:251], v[250:251], v[106:107]\n" \
  "v_pk_mul_f32 v[252:253], v[252:253], v[120:121]\n" \
  "v_pk_mul_f32 v[254:255], v[254:255], v[122:123]\n" \
  "v_cvt_pk_bf16_f32 v240, v248, v249\n" \
  "v_cvt_pk_bf16_f32 v241, v250, v251\n" \
  "v_cvt_pk_bf16_f32 v242, v252, v253\n" \
  "v_cvt_pk_bf16_f32 v243, v254, v255\n" \
  "v_pk_fma_f32 v[200:201], v[12:13], s[48:49], v[226:227] op_sel_hi:[1,0,0]\n" \
  "v_pk_fma_f32 v[202:203], v[14:15], s[48:49], v[226:227] op_sel_hi:[1,0,0]\n" \
  "v_pk_fma_f32 v[204:205], v[28:29], s[48:49], v[226:227] op_sel_hi:[1,0,0]\n" \
  "v_pk_fma_f32 v[206:207], v[30:31], s[48:49], v[226:227] op_sel_hi:[1,0,0]\n" \
  "v_permlane16_swap_b32 v240, v242\n" \
  "v_permlane16_swap_b32 v241, v243\n" \
  "buffer_store_dwordx4 v[240:243], v215, s[44:47], s63 offen offset:192 sc1 nt\n" \
  "s_add_u32 s63, s63, 131072\n" \
  "v_exp_f32 v200, v200 clamp\n" \
  "v_exp_f32 v201, v201 clamp\n" \
  "v_exp_f32 v202, v202 clamp\n" \
  "v_exp_f32 v203, v203 clamp\n" \
  "v_exp_f32 v204, v204 clamp\n" \
  "v_exp_f32 v205, v205 clamp\n" \
  "v_exp_f32 v206, v206 clamp\n" \
  "v_exp_f32 v207, v207 clamp\n" \
  "v_pk_fma_f32 v[200:201], v[200:201], s[50:51], 1.0 op_sel_hi:[1,0,0]\n" \
  "v_pk_fma_f32 v[202:203], v[202:203], s[50:51], 1.0 op_sel_hi:[1,0,0]\n" \
  "v_pk_fma_f32 v[204:205], v[204:205], s[50:51], 1.0 op_sel_hi:[1,0,0]\n" \
  "v_pk_fma_f32 v[206:207], v[206:207], s[50:51], 1.0 op_sel_hi:[1,0,0]\n" \
  "v_rcp_f32 v200, v200\n" \
  "v_rcp_f32 v201, v201\n" \
  "v_rcp_f32 v202, v202\n" \
  "v_rcp_f32 v203, v203\n" \
  "v_rcp_f32 v204, v204\n" \
  "v_rcp_f32 v205, v205\n" \
  "v_rcp_f32 v206, v206\n" \
  "v_rcp_f32 v207, v207\n" \
  "v_pk_mul_f32 v[200:201], v[200:201], v[12:13]\n" \
  "v_pk_mul_f32 v[202:203], v[202:203], v[14:15]\n" \
  "v_pk_mul_f32 v[204:205], v[204:205], v[28:29]\n" \
  "v_pk_mul_f32 v[206:207], v[206:207], v[30:31]\n" \
  "v_cvt_pk_bf16_f32 v192, v200, v201\n" \
  "v_cvt_pk_bf16_f32 v193, v202, v203\n" \
  "v_cvt_pk_bf16_f32 v194, v204, v205\n" \
  "v_cvt_pk_bf16_f32 v195, v206, v207\n" \
  "v_pk_fma_f32 v[248:249], v[44:45], s[48:49], v[226:227] op_sel_hi:[1,0,0]\n" \
  "v_pk_fma_f32 v[250:251], v[46:47], s[48:49], v[226:227] op_sel_hi:[1,0,0]\n" \
  "v_pk_fma_f32 v[252:253], v[60:61], s[48:49], v[226:227] op_sel_hi:[1,0,0]\n" \
  "v_pk_fma_f32 v[254:255], v[62:63], s[48:49], v[226:227] op_sel_hi:[1,0,0]\n" \
  "v_permlane16_swap_b32 v192, v194\n" \
  "v_permlane16_swap_b32 v193, v195\n" \
  "buffer_store_dwordx4 v[192:195], v215, s[44:47], s63 offen offset:0 sc1 nt\n" \
  "v_exp_f32 v248, v248 clamp\n" \
  "v_exp_f32 v249, v249 clamp\n" \
  "v_exp_f32 v250, v250 clamp\n" \
  "v_exp_f32 v251, v251 clamp\n" \
  "v_exp_f32 v252, v252 clamp\n" \
  "v_exp_f32 v253, v253 clamp\n" \
  "v_exp_f32 v254, v254 clamp\n" \
  "v_exp_f32 v255, v255 clamp\n" \
  "v_pk_fma_f32 v[248:249], v[248:249], s[50:51], 1.0 op_sel_hi:[1,0,0]\n" \
  "v_pk_fma_f32 v[250:251], v[250:251], s[50:51], 1.0 op_sel_hi:[1,0,0]\n" \
  "v_pk_fma_f32 v[252:253], v[252:253], s[50:51], 1.0 op_sel_hi:[1,0,0]\n" \
  "v_pk_fma_f32 v[254:255], v[254:255], s[50:51], 1.0 op_sel_hi:[1,0,0]\n" \
  "v_rcp_f32 v248, v248\n" \
  "v_rcp_f32 v249, v249\n" \
  "v_rcp_f32 v250, v250\n" \
  "v_rcp_f32 v251, v251\n" \
  "v_rcp_f32 v252, v252\n" \
  "v_rcp_f32 v253, v253\n" \
  "v_rcp_f32 v254, v254\n" \
  "v_rcp_f32 v255, v255\n" \
  "v_pk_mul_f32 v[248:249], v[248:249], v[44:45]\n" \
  "v_pk_mul_f32 v[250:251], v[250:251], v[46:47]\n" \
  "v_pk_mul_f32 v[252:253], v[252:253], v[60:61]\n" \
  "v_pk_mul_f32 v[254:255], v[254:255], v[62:63]\n" \
  "v_cvt_pk_bf16_f32 v240, v248, v249\n" \
  "v_cvt_pk_bf16_f32 v241, v250, v251\n" \
  "v_cvt_pk_bf16_f32 v242, v252, v253\n" \
  "v_cvt_pk_bf16_f32 v243, v254, v255\n" \
  "v_pk_fma_f32 v[200:201], v[76:77], s[48:49], v[226:227] op_sel_hi:[1,0,0]\n" \
  "v_pk_fma_f32 v[202:203], v[78:79], s[48:49], v[226:227] op_sel_hi:[1,0,0]\n" \
  "v_pk_fma_f32 v[204:205], v[92:93], s[48:49], v[226:227] op_sel_hi:[1,0,0]\n" \
  "v_pk_fma_f32 v[206:207], v[94:95], s[48:49], v[226:227] op_sel_hi:[1,0,0]\n" \
  "v_permlane16_swap_b32 v240, v242\n" \
  "v_permlane16_swap_b32 v241, v243\n" \
  "buffer_store_dwordx4 v[240:243], v215, s[44:47], s63 offen offset:64 sc1 nt\n" \
  "v_exp_f32 v200, v200 clamp\n" \
  "v_exp_f32 v201, v201 clamp\n" \
  "v_exp_f32 v202, v202 clamp\n" \
  "v_exp_f32 v203, v203 clamp\n" \
  "v_exp_f32 v204, v204 clamp\n" \
  "v_exp_f32 v205, v205 clamp\n" \
  "v_exp_f32 v206, v206 clamp\n" \
  "v_exp_f32 v207, v207 clamp\n" \
  "v_pk_fma_f32 v[200:201], v[200:201], s[50:51], 1.0 op_sel_hi:[1,0,0]\n" \
  "v_pk_fma_f32 v[202:203], v[202:203], s[50:51], 1.0 op_sel_hi:[1,0,0]\n" \
  "v_pk_fma_f32 v[204:205], v[204:205], s[50:51], 1.0 op_sel_hi:[1,0,0]\n" \
  "v_pk_fma_f32 v[206:207], v[206:207], s[50:51], 1.0 op_sel_hi:[1,0,0]\n" \
  "v_rcp_f32 v200, v200\n" \
  "v_rcp_f32 v201, v201\n" \
  "v_rcp_f32 v202, v202\n" \
  "v_rcp_f32 v203, v203\n" \
  "v_rcp_f32 v204, v204\n" \
  "v_rcp_f32 v205, v205\n" \
  "v_rcp_f32 v206, v206\n" \
  "v_rcp_f32 v207, v207\n" \
  "v_pk_mul_f32 v[200:201], v[200:201], v[76:77]\n" \
  "v_pk_mul_f32 v[202:203], v[202:203], v[78:79]\n" \
  "v_pk_mul_f32 v[204:205], v[204:205], v[92:93]\n" \
  "v_pk_mul_f32 v[206:207], v[206:207], v[94:95]\n" \
  "v_cvt_pk_bf16_f32 v192, v200, v201\n" \
  "v_cvt_pk_bf16_f32 v193, v202, v203\n" \
  "v_cvt_pk_bf16_f32 v194, v204, v205\n" \
  "v_cvt_pk_bf16_f32 v195, v206, v207\n" \
  "v_pk_fma_f32 v[248:249], v[108:109], s[48:49], v[226:227] op_sel_hi:[1,0,0]\n" \
  "v_pk_fma_f32 v[250:251], v[110:111], s[48:49], v[226:227] op_sel_hi:[1,0,0]\n" \
  "v_pk_fma_f32 v[252:253], v[124:125], s[48:49], v[226:227] op_sel_hi:[1,0,0]\n" \
  "v_pk_fma_f32 v[254:255], v[126:127], s[48:49], v[226:227] op_sel_hi:[1,0,0]\n" \
  "v_mfma_f32_16x16x32_bf16 a[112:115], a[156:159], v[128:131], a[112:115]\n" \
  "ds_read_b128 v[160:163], v208 offset:49152\n" \
  "v_mfma_f32_16x16x32_bf16 a[112:115], a[188:191], v[144:147], a[112:115]\n" \
  "ds_read_b128 v[176:179], v209 offset:49152\n" \
  "v_mfma_f32_16x16x32_bf16 a[96:99], a[184:187], v[144:147], a[96:99]\n" \
  "ds_read_b128 v[164:167], v208 offset:51200\n" \
  "v_mfma_f32_16x16x32_bf16 a[96:99], a[152:155], v[128:131], a[96:99]\n" \
  "ds_read_b128 v[180:183], v209 offset:51200\n" \
  "v_mfma_f32_16x16x32_bf16 a[80:83], a[148:151], v[128:131], a[80:83]\n" \
  "ds_read_b128 v[168:171], v208 offset:53248\n" \
  "v_mfma_f32_16x16x32_bf16 a[80:83], a[180:183], v[144:147], a[80:83]\n" \
  "ds_read_b128 v[184:187], v209 offset:53248\n" \
  "v_mfma_f32_16x16x32_bf16 a[64:67], a[176:179], v[144:147], a[64:67]\n" \
  "ds_read_b128 v[172:175], v208 offset:55296\n" \
  "v_mfma_f32_16x16x32_bf16 a[64:67], a[144:147], v[128:131], a[64:67]\n" \
  "ds_read_b128 v[188:191], v209 offset:55296\n" \
  "v_mfma_f32_16x16x32_bf16 a[48:51], a[140:143], v[128:131], a[48:51]\n" \
  "v_permlane16_swap_b32 v192, v194\n" \
  "v_mfma_f32_16x16x32_bf16 a[48:51], a[172:175], v[144:147], a[48:51]\n" \
  "v_permlane16_swap_b32 v193, v195\n" \
  "v_mfma_f32_16x16x32_bf16 a[32:35], a[168:171], v[144:147], a[32:35]\n" \
  "buffer_store_dwordx4 v[192:195], v215, s[44:47], s63 offen offset:128 sc1 nt\n" \
  "v_mfma_f32_16x16x32_bf16 a[32:35], a[136:139], v[128:131], a[32:35]\n" \
  "v_exp_f32 v248, v248 clamp\n" \
  "v_mfma_f32_16x16x32_bf16 a[16:19], a[132:135], v[128:131], a[16:19]\n" \
  "v_exp_f32 v249, v249 clamp\n" \
  "v_mfma_f32_16x16x32_bf16 a[16:19], a[164:167], v[144:147], a[16:19]\n" \
  "v_exp_f32 v250, v250 clamp\n" \
  "v_mfma_f32_16x16x32_bf16 a[0:3], a[160:163], v[144:147], a[0:3]\n" \
  "v_exp_f32 v251, v251 clamp\n" \
  "v_mfma_f32_16x16x32_bf16 a[0:3], a[128:131], v[128:131], a[0:3]\n" \
  "s_waitcnt lgkmcnt(6)\n" \
  "v_mfma_f32_16x16x32_bf16 a[0:3], a[192:195], v[160:163], a[0:3]\n" \
  "v_exp_f32 v252, v252 clamp\n" \
  "v_mfma_f32_16x16x32_bf16 a[0:3], a[224:227], v[176:179], a[0:3]\n" \
  "v_exp_f32 v253, v253 clamp\n" \
  "v_mfma_f32_16x16x32_bf16 a[16:19], a[228:231], v[176:179], a[16:19]\n" \
  "v_exp_f32 v254, v254 clamp\n" \
  "v_mfma_f32_16x16x32_bf16 a[16:19], a[196:199], v[160:163], a[16:19]\n" \
  "v_exp_f32 v255, v255 clamp\n" \
  "v_mfma_f32_16x16x32_bf16 a[32:35], a[200:203], v[160:163], a[32:35]\n" \
  "v_fma_f32 v248, v248, s50, 1.0\n" \
  "v_fma_f32 v249, v249, s50, 1.0\n" \
  "v_mfma_f32_16x16x32_bf16 a[32:35], a[232:235], v[176:179], a[32:35]\n" \
  "v_fma_f32 v250, v250, s50, 1.0\n" \
  "v_fma_f32 v251, v251, s50, 1.0\n" \
  "v_mfma_f32_16x16x32_bf16 a[48:51], a[236:239], v[176:179], a[48:51]\n" \
  "v_fma_f32 v252, v252, s50, 1.0\n" \
  "v_fma_f32 v253, v253, s50, 1.0\n" \
  "v_mfma_f32_16x16x32_bf16 a[48:51], a[204:207], v[160:163], a[48:51]\n" \
  "v_fma_f32 v254, v254, s50, 1.0\n" \
  "v_fma_f32 v255, v255, s50, 1.0\n" \
  "v_mfma_f32_16x16x32_bf16 a[64:67], a[208:211], v[160:163], a[64:67]\n" \
  "v_rcp_f32 v248, v248\n" \
  "v_mfma_f32_16x16x32_bf16 a[64:67], a[240:243], v[176:179], a[64:67]\n" \
  "v_rcp_f32 v249, v249\n" \
  "v_mfma_f32_16x16x32_bf16 a[80:83], a[244:247], v[176:179], a[80:83]\n" \
  "v_rcp_f32 v250, v250\n" \
  "v_mfma_f32_16x16x32_bf16 a[80:83], a[212:215], v[160:163], a[80:83]\n" \
  "v_rcp_f32 v251, v251\n" \
  "v_mfma_f32_16x16x32_bf16 a[96:99], a[216:219], v[160:163], a[96:99]\n" \
  "v_rcp_f32 v252, v252\n" \
  "v_mfma_f32_16x16x32_bf16 a[96:99], a[248:251], v[176:179], a[96:99]\n" \
  "v_rcp_f32 v253, v253\n" \
  "v_mfma_f32_16x16x32_bf16 a[112:115], a[252:255], v[176:179], a[112:115]\n" \
  "v_rcp_f32 v254, v254\n" \
  "v_mfma_f32_16x16x32_bf16 a[112:115], a[220:223], v[160:163], a[112:115]\n" \
  "v_rcp_f32 v255, v255\n" \
  "v_mul_f32 v248, v248, v108\n" \
  "v_mul_f32 v249, v249, v109\n" \
  "v_pk_mul_f32 v[250:251], v[250:251], v[110:111]\n" \
  "v_pk_mul_f32 v[252:253], v[252:253], v[124:125]\n" \
  "v_pk_mul_f32 v[254:255], v[254:255], v[126:127]\n" \
  "v_cvt_pk_bf16_f32 v240, v248, v249\n" \
  "v_cvt_pk_bf16_f32 v241, v250, v251\n" \
  "v_cvt_pk_bf16_f32 v242, v252, v253\n" \
  "v_cvt_pk_bf16_f32 v243, v254, v255\n" \
  "v_accvgpr_read_b32 v192, a0\n" \
  "v_accvgpr_read_b32 v193, a1\n" \
  "v_accvgpr_read_b32 v194, a2\n" \
  "v_accvgpr_read_b32 v195, a3\n" \
  "v_permlane16_swap_b32 v240, v242\n" \
  "v_permlane16_swap_b32 v241, v243\n" \
  "buffer_store_dwordx4 v[240:243], v215, s[44:47], s63 offen offset:192 sc1 nt\n" \
  "s_add_u32 s63, s63, 131072\n" \
  "v_accvgpr_read_b32 v196, a16\n" \
  "v_accvgpr_read_b32 v197, a17\n" \
  "v_accvgpr_read_b32 v198, a18\n" \
  "v_accvgpr_read_b32 v199, a19\n" \
  "v_pk_fma_f32 v[200:201], v[192:193], s[48:49], v[226:227] op_sel_hi:[1,0,0]\n" \
  "v_pk_fma_f32 v[202:203], v[194:195], s[48:49], v[226:227] op_sel_hi:[1,0,0]\n" \
  "v_pk_fma_f32 v[204:205], v[196:197], s[48:49], v[226:227] op_sel_hi:[1,0,0]\n" \
  "v_pk_fma_f32 v[206:207], v[198:199], s[48:49], v[226:227] op_sel_hi:[1,0,0]\n" \
  "v_exp_f32 v200, v200 clamp\n" \
  "v_exp_f32 v201, v201 clamp\n" \
  "v_exp_f32 v202, v202 clamp\n" \
  "v_exp_f32 v203, v203 clamp\n" \
  "v_exp_f32 v204, v204 clamp\n" \
  "v_exp_f32 v205, v205 clamp\n" \
  "v_exp_f32 v206, v206 clamp\n" \
  "v_exp_f32 v207, v207 clamp\n" \
  "v_pk_fma_f32 v[200:201], v[200:201], s[50:51], 1.0 op_sel_hi:[1,0,0]\n" \
  "v_pk_fma_f32 v[202:203], v[202:203], s[50:51], 1.0 op_sel_hi:[1,0,0]\n" \
  "v_pk_fma_f32 v[204:205], v[204:205], s[50:51], 1.0 op_sel_hi:[1,0,0]\n" \
  "v_pk_fma_f32 v[206:207], v[206:207], s[50:51], 1.0 op_sel_hi:[1,0,0]\n" \
  "v_rcp_f32 v200, v200\n" \
  "v_rcp_f32 v201, v201\n" \
  "v_rcp_f32 v202, v202\n" \
  "v_rcp_f32 v203, v203\n" \
  "v_rcp_f32 v204, v204\n" \
  "v_rcp_f32 v205, v205\n" \
  "v_rcp_f32 v206, v206\n" \
  "v_rcp_f32 v207, v207\n" \
  "v_pk_mul_f32 v[200:201], v[200:201], v[192:193]\n" \
  "v_pk_mul_f32 v[202:203], v[202:203], v[194:195]\n" \
  "v_pk_mul_f32 v[204:205], v[204:205], v[196:197]\n" \
  "v_pk_mul_f32 v[206:207], v[206:207], v[198:199]\n" \
  "v_cvt_pk_bf16_f32 v192, v200, v201\n" \
  "v_cvt_pk_bf16_f32 v193, v202, v203\n" \
  "v_cvt_pk_bf16_f32 v194, v204, v205\n" \
  "v_cvt_pk_bf16_f32 v195, v206, v207\n" \
  "v_accvgpr_read_b32 v240, a32\n" \
  "v_accvgpr_read_b32 v241, a33\n" \
  "v_accvgpr_read_b32 v242, a34\n" \
  "v_accvgpr_read_b32 v243, a35\n" \
  "v_permlane16_swap_b32 v192, v194\n" \
  "v_permlane16_swap_b32 v193, v195\n" \
  "buffer_store_dwordx4 v[192:195], v215, s[44:47], s63 offen offset:0 sc1 nt\n" \
  "v_accvgpr_read_b32 v244, a48\n" \
  "v_accvgpr_read_b32 v245, a49\n" \
  "v_accvgpr_read_b32 v246, a50\n" \
  "v_accvgpr_read_b32 v247, a51\n" \
  "v_pk_fma_f32 v[248:249], v[240:241], s[48:49], v[226:227] op_sel_hi:[1,0,0]\n" \
  "v_pk_fma_f32 v[250:251], v[242:243], s[48:49], v[226:227] op_sel_hi:[1,0,0]\n" \
  "v_pk_fma_f32 v[252:253], v[244:245], s[48:49], v[226:227] op_sel_hi:[1,0,0]\n" \
  "v_pk_fma_f32 v[254:255], v[246:247], s[48:49], v[226:227] op_sel_hi:[1,0,0]\n" \
  "v_exp_f32 v248, v248 clamp\n" \
  "v_exp_f32 v249, v249 clamp\n" \
  "v_exp_f32 v250, v250 clamp\n" \
  "v_exp_f32 v251, v251 clamp\n" \
  "v_exp_f32 v252, v252 clamp\n" \
  "v_exp_f32 v253, v253 clamp\n" \
  "v_exp_f32 v254, v254 clamp\n" \
  "v_exp_f32 v255, v255 clamp\n" \
  "v_pk_fma_f32 v[248:249], v[248:249], s[50:51], 1.0 op_sel_hi:[1,0,0]\n" \
  "v_pk_fma_f32 v[250:251], v[250:251], s[50:51], 1.0 op_sel_hi:[1,0,0]\n" \
  "v_pk_fma_f32 v[252:253], v[252:253], s[50:51], 1.0 op_sel_hi:[1,0,0]\n" \
  "v_pk_fma_f32 v[254:255], v[254:255], s[50:51], 1.0 op_sel_hi:[1,0,0]\n" \
  "v_rcp_f32 v248, v248\n" \
  "v_rcp_f32 v249, v249\n" \
  "v_rcp_f32 v250, v250\n" \
  "v_rcp_f32 v251, v251\n" \
  "v_rcp_f32 v252, v252\n" \
  "v_rcp_f32 v253, v253\n" \
  "v_rcp_f32 v254, v254\n" \
  "v_rcp_f32 v255, v255\n" \
  "v_pk_mul_f32 v[248:249], v[248:249], v[240:241]\n" \
  "v_pk_mul_f32 v[250:251], v[250:251], v[242:243]\n" \
  "v_pk_mul_f32 v[252:253], v[252:253], v[244:245]\n" \
  "v_pk_mul_f32 v[254:255], v[254:255], v[246:247]\n" \
  "v_cvt_pk_bf16_f32 v240, v248, v249\n" \
  "v_cvt_pk_bf16_f32 v241, v250, v251\n" \
  "v_cvt_pk_bf16_f32 v242, v252, v253\n" \
  "v_cvt_pk_bf16_f32 v243, v254, v255\n" \
  "v_accvgpr_read_b32 v192, a64\n" \
  "v_accvgpr_read_b32 v193, a65\n" \
  "v_accvgpr_read_b32 v194, a66\n" \
  "v_accvgpr_read_b32 v195, a67\n" \
  "v_permlane16_swap_b32 v240, v242\n" \
  "v_permlane16_swap_b32 v241, v243\n" \
  "buffer_store_dwordx4 v[240:243], v215, s[44:47], s63 offen offset:64 sc1 nt\n" \
  "v_accvgpr_read_b32 v196, a80\n" \
  "v_accvgpr_read_b32 v197, a81\n" \
  "v_accvgpr_read_b32 v198, a82\n" \
  "v_accvgpr_read_b32 v199, a83\n" \
  "v_pk_fma_f32 v[200:201], v[192:193], s[48:49], v[226:227] op_sel_hi:[1,0,0]\n" \
  "v_pk_fma_f32 v[202:203], v[194:195], s[48:49], v[226:227] op_sel_hi:[1,0,0]\n" \
  "v_pk_fma_f32 v[204:205], v[196:197], s[48:49], v[226:227] op_sel_hi:[1,0,0]\n" \
  "v_pk_fma_f32 v[206:207], v[198:199], s[48:49], v[226:227] op_sel_hi:[1,0,0]\n" \
  "v_exp_f32 v200, v200 clamp\n" \
  "v_exp_f32 v201, v201 clamp\n" \
  "v_exp_f32 v202, v202 clamp\n" \
  "v_exp_f32 v203, v203 clamp\n" \
  "v_exp_f32 v204, v204 clamp\n" \
  "v_exp_f32 v205, v205 clamp\n" \
  "v_exp_f32 v206, v206 clamp\n" \
  "v_exp_f32 v207, v207 clamp\n" \
  "v_pk_fma_f32 v[200:201], v[200:201], s[50:51], 1.0 op_sel_hi:[1,0,0]\n" \
  "v_pk_fma_f32 v[202:203], v[202:203], s[50:51], 1.0 op_sel_hi:[1,0,0]\n" \
  "v_pk_fma_f32 v[204:205], v[204:205], s[50:51], 1.0 op_sel_hi:[1,0,0]\n" \
  "v_pk_fma_f32 v[206:207], v[206:207], s[50:51], 1.0 op_sel_hi:[1,0,0]\n" \
  "v_rcp_f32 v200, v200\n" \
  "v_rcp_f32 v201, v201\n" \
  "v_rcp_f32 v202, v202\n" \
  "v_rcp_f32 v203, v203\n" \
  "v_rcp_f32 v204, v204\n" \
  "v_rcp_f32 v205, v205\n" \
  "v_rcp_f32 v206, v206\n" \
  "v_rcp_f32 v207, v207\n" \
  "v_pk_mul_f32 v[200:201], v[200:201], v[192:193]\n" \
  "v_pk_mul_f32 v[202:203], v[202:203], v[194:195]\n" \
  "v_pk_mul_f32 v[204:205], v[204:205], v[196:197]\n" \
  "v_pk_mul_f32 v[206:207], v[206:207], v[198:199]\n" \
  "v_mfma_f32_16x16x32_bf16 a[116:119], a[156:159], v[132:135], a[116:119]\n" \
  "v_cvt_pk_bf16_f32 v192, v200, v201\n" \
  "v_cvt_pk_bf16_f32 v193, v202, v203\n" \
  "v_mfma_f32_16x16x32_bf16 a[116:119], a[188:191], v[148:151], a[116:119]\n" \
  "v_cvt_pk_bf16_f32 v194, v204, v205\n" \
  "v_cvt_pk_bf16_f32 v195, v206, v207\n" \
  "v_mfma_f32_16x16x32_bf16 a[100:103], a[184:187], v[148:151], a[100:103]\n" \
  "v_accvgpr_read_b32 v240, a96\n" \
  "v_accvgpr_read_b32 v241, a97\n" \
  "v_mfma_f32_16x16x32_bf16 a[100:103], a[152:155], v[132:135], a[100:103]\n" \
  "v_accvgpr_read_b32 v242, a98\n" \
  "v_accvgpr_read_b32 v243, a99\n" \
  "v_mfma_f32_16x16x32_bf16 a[84:87], a[148:151], v[132:135], a[84:87]\n" \
  "v_permlane16_swap_b32 v192, v194\n" \
  "v_mfma_f32_16x16x32_bf16 a[84:87], a[180:183], v[148:151], a[84:87]\n" \
  "v_permlane16_swap_b32 v193, v195\n" \
  "v_mfma_f32_16x16x32_bf16 a[68:71], a[176:179], v[148:151], a[68:71]\n" \
  "buffer_store_dwordx4 v[192:195], v215, s[44:47], s63 offen offset:128 sc1 nt\n" \
  "v_accvgpr_read_b32 v244, a112\n" \
  "v_mfma_f32_16x16x32_bf16 a[68:71], a[144:147], v[132:135], a[68:71]\n" \
  "v_accvgpr_read_b32 v245, a113\n" \
  "v_accvgpr_read_b32 v246, a114\n" \
  "v_mfma_f32_16x16x32_bf16 a[52:55], a[140:143], v[132:135], a[52:55]\n" \
  "v_accvgpr_read_b32 v247, a115\n" \
  "v_fma_f32 v248, v240, s48, v226\n" \
  "v_mfma_f32_16x16x32_bf16 a[52:55], a[172:175], v[148:151], a[52:55]\n" \
  "v_fma_f32 v249, v241, s48, v226\n" \
  "v_fma_f32 v250, v242, s48, v226\n" \
  "v_mfma_f32_16x16x32_bf16 a[36:39], a[168:171], v[148:151], a[36:39]\n" \
  "v_fma_f32 v251, v243, s48, v226\n" \
  "v_fma_f32 v252, v244, s48, v226\n" \
  "v_mfma_f32_16x16x32_bf16 a[36:39], a[136:139], v[132:135], a[36:39]\n" \
  "v_fma_f32 v253, v245, s48, v226\n" \
  "v_fma_f32 v254, v246, s48, v226\n" \
  "v_mfma_f32_16x16x32_bf16 a[20:23], a[132:135], v[132:135], a[20:23]\n" \
  "v_fma_f32 v255, v247, s48, v226\n" \
  "v_mfma_f32_16x16x32_bf16 a[20:23], a[164:167], v[148:151], a[20:23]\n" \
  "v_exp_f32 v248, v248 clamp\n" \
  "v_mfma_f32_16x16x32_bf16 a[4:7], a[160:163], v[148:151], a[4:7]\n" \
  "v_exp_f32 v249, v249 clamp\n" \
  "v_mfma_f32_16x16x32_bf16 a[4:7], a[128:131], v[132:135], a[4:7]\n" \
  "s_waitcnt lgkmcnt(0)\n" \
  "v_mfma_f32_16x16x32_bf16 a[4:7], a[192:195], v[164:167], a[4:7]\n" \
  "v_exp_f32 v250, v250 clamp\n" \
  "v_mfma_f32_16x16x32_bf16 a[4:7], a[224:227], v[180:183], a[4:7]\n" \
  "v_exp_f32 v251, v251 clamp\n" \
  "v_mfma_f32_16x16x32_bf16 a[20:23], a[228:231], v[180:183], a[20:23]\n" \
  "v_exp_f32 v252, v252 clamp\n" \
  "v_mfma_f32_16x16x32_bf16 a[20:23], a[196:199], v[164:167], a[20:23]\n" \
  "v_exp_f32 v253, v253 clamp\n" \
  "v_mfma_f32_16x16x32_bf16 a[36:39], a[200:203], v[164:167], a[36:39]\n" \
  "v_exp_f32 v254, v254 clamp\n" \
  "v_mfma_f32_16x16x32_bf16 a[36:39], a[232:235], v[180:183], a[36:39]\n" \
  "v_exp_f32 v255, v255 clamp\n" \
  "v_mfma_f32_16x16x32_bf16 a[52:55], a[236:239], v[180:183], a[52:55]\n" \
  "v_fma_f32 v248, v248, s50, 1.0\n" \
  "v_fma_f32 v249, v249, s50, 1.0\n" \
  "v_mfma_f32_16x16x32_bf16 a[52:55], a[204:207], v[164:167], a[52:55]\n" \
  "v_fma_f32 v250, v250, s50, 1.0\n" \
  "v_fma_f32 v251, v251, s50, 1.0\n" \
  "v_mfma_f32_16x16x32_bf16 a[68:71], a[208:211], v[164:167], a[68:71]\n" \
  "v_fma_f32 v252, v252, s50, 1.0\n" \
  "v_fma_f32 v253, v253, s50, 1.0\n" \
  "v_mfma_f32_16x16x32_bf16 a[68:71], a[240:243], v[180:183], a[68:71]\n" \
  "v_fma_f32 v254, v254, s50, 1.0\n" \
  "v_fma_f32 v255, v255, s50, 1.0\n" \
  "v_mfma_f32_16x16x32_bf16 a[84:87], a[244:247], v[180:183], a[84:87]\n" \
  "v_rcp_f32 v248, v248\n" \
  "v_mfma_f32_16x16x32_bf16 a[84:87], a[212:215], v[164:167], a[84:87]\n" \
  "v_rcp_f32 v249, v249\n" \
  "v_mfma_f32_16x16x32_bf16 a[100:103], a[216:219], v[164:167], a[100:103]\n" \
  "v_rcp_f32 v250, v250\n" \
  "v_mfma_f32_16x16x32_bf16 a[100:103], a[248:251], v[180:183], a[100:103]\n" \
  "v_rcp_f32 v251, v251\n" \
  "v_mfma_f32_16x16x32_bf16 a[116:119], a[252:255], v[180:183], a[116:119]\n" \
  "v_rcp_f32 v252, v252\n" \
  "v_mfma_f32_16x16x32_bf16 a[116:119], a[220:223], v[164:167], a[116:119]\n" \
  "v_rcp_f32 v253, v253\n" \
  "v_rcp_f32 v254, v254\n" \
  "v_rcp_f32 v255, v255\n" \
  "v_pk_mul_f32 v[248:249], v[248:249], v[240:241]\n" \
  "v_pk_mul_f32 v[250:251], v[250:251], v[242:243]\n" \
  "v_pk_mul_f32 v[252:253], v[252:253], v[244:245]\n" \
  "v_pk_mul_f32 v[254:255], v[254:255], v[246:247]\n" \
  "v_cvt_pk_bf16_f32 v240, v248, v249\n" \
  "v_cvt_pk_bf16_f32 v241, v250, v251\n" \
  "v_cvt_pk_bf16_f32 v242, v252, v253\n" \
  "v_cvt_pk_bf16_f32 v243, v254, v255\n" \
  "v_accvgpr_read_b32 v192, a4\n" \
  "v_accvgpr_read_b32 v193, a5\n" \
  "v_accvgpr_read_b32 v194, a6\n" \
  "v_accvgpr_read_b32 v195, a7\n" \
  "v_permlane16_swap_b32 v240, v242\n" \
  "v_permlane16_swap_b32 v241, v243\n" \
  "buffer_store_dwordx4 v[240:243], v215, s[44:47], s63 offen offset:192 sc1 nt\n" \
  "s_add_u32 s63, s63, 131072\n" \
  "v_accvgpr_read_b32 v196, a20\n" \
  "v_accvgpr_read_b32 v197, a21\n" \
  "v_accvgpr_read_b32 v198, a22\n" \
  "v_accvgpr_read_b32 v199, a23\n" \
  "v_pk_fma_f32 v[200:201], v[192:193], s[48:49], v[226:227] op_sel_hi:[1,0,0]\n" \
  "v_pk_fma_f32 v[202:203], v[194:195], s[48:49], v[226:227] op_sel_hi:[1,0,0]\n" \
  "v_pk_fma_f32 v[204:205], v[196:197], s[48:49], v[226:227] op_sel_hi:[1,0,0]\n" \
  "v_pk_fma_f32 v[206:207], v[198:199], s[48:49], v[226:227] op_sel_hi:[1,0,0]\n" \
  "v_exp_f32 v200, v200 clamp\n" \
  "v_exp_f32 v201, v201 clamp\n" \
  "v_exp_f32 v202, v202 clamp\n" \
  "v_exp_f32 v203, v203 clamp\n" \
  "v_exp_f32 v204, v204 clamp\n" \
  "v_exp_f32 v205, v205 clamp\n" \
  "v_exp_f32 v206, v206 clamp\n" \
  "v_exp_f32 v207, v207 clamp\n" \
  "v_pk_fma_f32 v[200:201], v[200:201], s[50:51], 1.0 op_sel_hi:[1,0,0]\n" \
  "v_pk_fma_f32 v[202:203], v[202:203], s[50:51], 1.0 op_sel_hi:[1,0,0]\n" \
  "v_pk_fma_f32 v[204:205], v[204:205], s[50:51], 1.0 op_sel_hi:[1,0,0]\n" \
  "v_pk_fma_f32 v[206:207], v[206:207], s[50:51], 1.0 op_sel_hi:[1,0,0]\n" \
  "v_rcp_f32 v200, v200\n" \
  "v_rcp_f32 v201, v201\n" \
  "v_rcp_f32 v202, v202\n" \
  "v_rcp_f32 v203, v203\n" \
  "v_rcp_f32 v204, v204\n" \
  "v_rcp_f32 v205, v205\n" \
  "v_rcp_f32 v206, v206\n" \
  "v_rcp_f32 v207, v207\n" \
  "v_pk_mul_f32 v[200:201], v[200:201], v[192:193]\n" \
  "v_pk_mul_f32 v[202:203], v[202:203], v[194:195]\n" \
  "v_pk_mul_f32 v[204:205], v[204:205], v[196:197]\n" \
  "v_pk_mul_f32 v[206:207], v[206:207], v[198:199]\n" \
  "v_cvt_pk_bf16_f32 v192, v200, v201\n" \
  "v_cvt_pk_bf16_f32 v193, v202, v203\n" \
  "v_cvt_pk_bf16_f32 v194, v204, v205\n" \
  "v_cvt_pk_bf16_f32 v195, v206, v207\n" \
  "v_accvgpr_read_b32 v240, a36\n" \
  "v_accvgpr_read_b32 v241, a37\n" \
  "v_accvgpr_read_b32 v242, a38\n" \
  "v_accvgpr_read_b32 v243, a39\n" \
  "v_permlane16_swap_b32 v192, v194\n" \
  "v_permlane16_swap_b32 v193, v195\n" \
  "buffer_store_dwordx4 v[192:195], v215, s[44:47], s63 offen offset:0 sc1 nt\n" \
  "v_accvgpr_read_b32 v244, a52\n" \
  "v_accvgpr_read_b32 v245, a53\n" \
  "v_accvgpr_read_b32 v246, a54\n" \
  "v_accvgpr_read_b32 v247, a55\n" \
  "v_pk_fma_f32 v[248:249], v[240:241], s[48:49], v[226:227] op_sel_hi:[1,0,0]\n" \
  "v_pk_fma_f32 v[250:251], v[242:243], s[48:49], v[226:227] op_sel_hi:[1,0,0]\n" \
  "v_pk_fma_f32 v[252:253], v[244:245], s[48:49], v[226:227] op_sel_hi:[1,0,0]\n" \
  "v_pk_fma_f32 v[254:255], v[246:247], s[48:49], v[226:227] op_sel_hi:[1,0,0]\n" \
  "v_exp_f32 v248, v248 clamp\n" \
  "v_exp_f32 v249, v249 clamp\n" \
  "v_exp_f32 v250, v250 clamp\n" \
  "v_exp_f32 v251, v251 clamp\n" \
  "v_exp_f32 v252, v252 clamp\n" \
  "v_exp_f32 v253, v253 clamp\n" \
  "v_exp_f32 v254, v254 clamp\n" \
  "v_exp_f32 v255, v255 clamp\n" \
  "v_pk_fma_f32 v[248:249], v[248:249], s[50:51], 1.0 op_sel_hi:[1,0,0]\n" \
  "v_pk_fma_f32 v[250:251], v[250:251], s[50:51], 1.0 op_sel_hi:[1,0,0]\n" \
  "v_pk_fma_f32 v[252:253], v[252:253], s[50:51], 1.0 op_sel_hi:[1,0,0]\n" \
  "v_pk_fma_f32 v[254:255], v[254:255], s[50:51], 1.0 op_sel_hi:[1,0,0]\n" \
  "v_rcp_f32 v248, v248\n" \
  "v_rcp_f32 v249, v249\n" \
  "v_rcp_f32 v250, v250\n" \
  "v_rcp_f32 v251, v251\n" \
  "v_rcp_f32 v252, v252\n" \
  "v_rcp_f32 v253, v253\n" \
  "v_rcp_f32 v254, v254\n" \
  "v_rcp_f32 v255, v255\n" \
  "v_pk_mul_f32 v[248:249], v[248:249], v[240:241]\n" \
  "v_pk_mul_f32 v[250:251], v[250:251], v[242:243]\n" \
  "v_pk_mul_f32 v[252:253], v[252:253], v[244:245]\n" \
  "v_pk_mul_f32 v[254:255], v[254:255], v[246:247]\n" \
  "v_cvt_pk_bf16_f32 v240, v248, v249\n" \
  "v_cvt_pk_bf16_f32 v241, v250, v251\n" \
  "v_cvt_pk_bf16_f32 v242, v252, v253\n" \
  "v_cvt_pk_bf16_f32 v243, v254, v255\n" \
  "v_accvgpr_read_b32 v192, a68\n" \
  "v_accvgpr_read_b32 v193, a69\n" \
  "v_accvgpr_read_b32 v194, a70\n" \
  "v_accvgpr_read_b32 v195, a71\n" \
  "v_permlane16_swap_b32 v240, v242\n" \
  "v_permlane16_swap_b32 v241, v243\n" \
  "buffer_store_dwordx4 v[240:243], v215, s[44:47], s63 offen offset:64 sc1 nt\n" \
  "v_accvgpr_read_b32 v196, a84\n" \
  "v_accvgpr_read_b32 v197, a85\n" \
  "v_accvgpr_read_b32 v198, a86\n" \
  "v_accvgpr_read_b32 v199, a87\n" \
  "v_pk_fma_f32 v[200:201], v[192:193], s[48:49], v[226:227] op_sel_hi:[1,0,0]\n" \
  "v_pk_fma_f32 v[202:203], v[194:195], s[48:49], v[226:227] op_sel_hi:[1,0,0]\n" \
  "v_pk_fma_f32 v[204:205], v[196:197], s[48:49], v[226:227] op_sel_hi:[1,0,0]\n" \
  "v_pk_fma_f32 v[206:207], v[198:199], s[48:49], v[226:227] op_sel_hi:[1,0,0]\n" \
  "v_exp_f32 v200, v200 clamp\n" \
  "v_exp_f32 v201, v201 clamp\n" \
  "v_exp_f32 v202, v202 clamp\n" \
  "v_exp_f32 v203, v203 clamp\n" \
  "v_exp_f32 v204, v204 clamp\n" \
  "v_exp_f32 v205, v205 clamp\n" \
  "v_exp_f32 v206, v206 clamp\n" \
  "v_exp_f32 v207, v207 clamp\n" \
  "v_pk_fma_f32 v[200:201], v[200:201], s[50:51], 1.0 op_sel_hi:[1,0,0]\n" \
  "v_pk_fma_f32 v[202:203], v[202:203], s[50:51], 1.0 op_sel_hi:[1,0,0]\n" \
  "v_pk_fma_f32 v[204:205], v[204:205], s[50:51], 1.0 op_sel_hi:[1,0,0]\n" \
  "v_pk_fma_f32 v[206:207], v[206:207], s[50:51], 1.0 op_sel_hi:[1,0,0]\n" \
  "v_rcp_f32 v200, v200\n" \
  "v_rcp_f32 v201, v201\n" \
  "v_rcp_f32 v202, v202\n" \
  "v_rcp_f32 v203, v203\n" \
  "v_rcp_f32 v204, v204\n" \
  "v_rcp_f32 v205, v205\n" \
  "v_rcp_f32 v206, v206\n" \
  "v_rcp_f32 v207, v207\n" \
  "v_pk_mul_f32 v[200:201], v[200:201], v[192:193]\n" \
  "v_pk_mul_f32 v[202:203], v[202:203], v[194:195]\n" \
  "v_pk_mul_f32 v[204:205], v[204:205], v[196:197]\n" \
  "v_pk_mul_f32 v[206:207], v[206:207], v[198:199]\n" \
  "v_mfma_f32_16x16x32_bf16 a[120:123], a[156:159], v[136:139], a[120:123]\n" \
  "v_cvt_pk_bf16_f32 v192, v200, v201\n" \
  "v_cvt_pk_bf16_f32 v193, v202, v203\n" \
  "v_mfma_f32_16x16x32_bf16 a[120:123], a[188:191], v[152:155], a[120:123]\n" \
  "v_cvt_pk_bf16_f32 v194, v204, v205\n" \
  "v_cvt_pk_bf16_f32 v195, v206, v207\n" \
  "v_mfma_f32_16x16x32_bf16 a[104:107], a[184:187], v[152:155], a[104:107]\n" \
  "v_accvgpr_read_b32 v240, a100\n" \
  "v_accvgpr_read_b32 v241, a101\n" \
  "v_mfma_f32_16x16x32_bf16 a[104:107], a[152:155], v[136:139], a[104:107]\n" \
  "v_accvgpr_read_b32 v242, a102\n" \
  "v_accvgpr_read_b32 v243, a103\n" \
  "v_mfma_f32_16x16x32_bf16 a[88:91], a[148:151], v[136:139], a[88:91]\n" \
  "v_permlane16_swap_b32 v192, v194\n" \
  "v_mfma_f32_16x16x32_bf16 a[88:91], a[180:183], v[152:155], a[88:91]\n" \
  "v_permlane16_swap_b32 v193, v195\n" \
  "v_mfma_f32_16x16x32_bf16 a[72:75], a[176:179], v[152:155], a[72:75]\n" \
  "buffer_store_dwordx4 v[192:195], v215, s[44:47], s63 offen offset:128 sc1 nt\n" \
  "v_accvgpr_read_b32 v244, a116\n" \
  "v_mfma_f32_16x16x32_bf16 a[72:75], a[144:147], v[136:139], a[72:75]\n" \
  "v_accvgpr_read_b32 v245, a117\n" \
  "v_accvgpr_read_b32 v246, a118\n" \
  "v_mfma_f32_16x16x32_bf16 a[56:59], a[140:143], v[136:139], a[56:59]\n" \
  "v_accvgpr_read_b32 v247, a119\n" \
  "v_fma_f32 v248, v240, s48, v226\n" \
  "v_mfma_f32_16x16x32_bf16 a[56:59], a[172:175], v[152:155], a[56:59]\n" \
  "v_fma_f32 v249, v241, s48, v226\n" \
  "v_fma_f32 v250, v242, s48, v226\n" \
  "v_mfma_f32_16x16x32_bf16 a[40:43], a[168:171], v[152:155], a[40:43]\n" \
  "v_fma_f32 v251, v243, s48, v226\n" \
  "v_fma_f32 v252, v244, s48, v226\n" \
  "v_mfma_f32_16x16x32_bf16 a[40:43], a[136:139], v[136:139], a[40:43]\n" \
  "v_fma_f32 v253, v245, s48, v226\n" \
  "v_fma_f32 v254, v246, s48, v226\n" \
  "v_mfma_f32_16x16x32_bf16 a[24:27], a[132:135], v[136:139], a[24:27]\n" \
  "v_fma_f32 v255, v247, s48, v226\n" \
  "v_mfma_f32_16x16x32_bf16 a[24:27], a[164:167], v[152:155], a[24:27]\n" \
  "v_exp_f32 v248, v248 clamp\n" \
  "v_mfma_f32_16x16x32_bf16 a[8:11], a[160:163], v[152:155], a[8:11]\n" \
  "v_exp_f32 v249, v249 clamp\n" \
  "v_mfma_f32_16x16x32_bf16 a[8:11], a[128:131], v[136:139], a[8:11]\n" \
  "v_exp_f32 v250, v250 clamp\n" \
  "v_mfma_f32_16x16x32_bf16 a[8:11], a[192:195], v[168:171], a[8:11]\n" \
  "v_exp_f32 v251, v251 clamp\n" \
  "v_mfma_f32_16x16x32_bf16 a[8:11], a[224:227], v[184:187], a[8:11]\n" \
  "v_exp_f32 v252, v252 clamp\n" \
  "v_mfma_f32_16x16x32_bf16 a[24:27], a[228:231], v[184:187], a[24:27]\n" \
  "v_exp_f32 v253, v253 clamp\n" \
  "v_mfma_f32_16x16x32_bf16 a[24:27], a[196:199], v[168:171], a[24:27]\n" \
  "v_exp_f32 v254, v254 clamp\n" \
  "v_mfma_f32_16x16x32_bf16 a[40:43], a[200:203], v[168:171], a[40:43]\n" \
  "v_exp_f32 v255, v255 clamp\n" \
  "v_mfma_f32_16x16x32_bf16 a[40:43], a[232:235], v[184:187], a[40:43]\n" \
  "v_fma_f32 v248, v248, s50, 1.0\n" \
  "v_fma_f32 v249, v249, s50, 1.0\n" \
  "v_mfma_f32_16x16x32_bf16 a[56:59], a[236:239], v[184:187], a[56:59]\n" \
  "v_fma_f32 v250, v250, s50, 1.0\n" \
  "v_fma_f32 v251, v251, s50, 1.0\n" \
  "v_mfma_f32_16x16x32_bf16 a[56:59], a[204:207], v[168:171], a[56:59]\n" \
  "v_fma_f32 v252, v252, s50, 1.0\n" \
  "v_fma_f32 v253, v253, s50, 1.0\n" \
  "v_mfma_f32_16x16x32_bf16 a[72:75], a[208:211], v[168:171], a[72:75]\n" \
  "v_fma_f32 v254, v254, s50, 1.0\n" \
  "v_fma_f32 v255, v255, s50, 1.0\n" \
  "v_mfma_f32_16x16x32_bf16 a[72:75], a[240:243], v[184:187], a[72:75]\n" \
  "v_rcp_f32 v248, v248\n" \
  "v_mfma_f32_16x16x32_bf16 a[88:91], a[244:247], v[184:187], a[88:91]\n" \
  "v_rcp_f32 v249, v249\n" \
  "v_mfma_f32_16x16x32_bf16 a[88:91], a[212:215], v[168:171], a[88:91]\n" \
  "v_rcp_f32 v250, v250\n" \
  "v_mfma_f32_16x16x32_bf16 a[104:107], a[216:219], v[168:171], a[104:107]\n" \
  "v_rcp_f32 v251, v251\n" \
  "v_mfma_f32_16x16x32_bf16 a[104:107], a[248:251], v[184:187], a[104:107]\n" \
  "v_rcp_f32 v252, v252\n" \
  "v_mfma_f32_16x16x32_bf16 a[120:123], a[252:255], v[184:187], a[120:123]\n" \
  "v_rcp_f32 v253, v253\n" \
  "v_mfma_f32_16x16x32_bf16 a[120:123], a[220:223], v[168:171], a[120:123]\n" \
  "v_rcp_f32 v254, v254\n" \
  "v_rcp_f32 v255, v255\n" \
  "v_pk_mul_f32 v[248:249], v[248:249], v[240:241]\n" \
  "v_pk_mul_f32 v[250:251], v[250:251], v[242:243]\n" \
  "v_pk_mul_f32 v[252:253], v[252:253], v[244:245]\n" \
  "v_pk_mul_f32 v[254:255], v[254:255], v[246:247]\n" \
  "v_cvt_pk_bf16_f32 v240, v248, v249\n" \
  "v_cvt_pk_bf16_f32 v241, v250, v251\n" \
  "v_cvt_pk_bf16_f32 v242, v252, v253\n" \
  "v_cvt_pk_bf16_f32 v243, v254, v255\n" \
  "v_accvgpr_read_b32 v192, a8\n" \
  "v_accvgpr_read_b32 v193, a9\n" \
  "v_accvgpr_read_b32 v194, a10\n" \
  "v_accvgpr_read_b32 v195, a11\n" \
  "v_permlane16_swap_b32 v240, v242\n" \
  "v_permlane16_swap_b32 v241, v243\n" \
  "buffer_store_dwordx4 v[240:243], v215, s[44:47], s63 offen offset:192 sc1 nt\n" \
  "s_add_u32 s63, s63, 131072\n" \
  "v_accvgpr_read_b32 v196, a24\n" \
  "v_accvgpr_read_b32 v197, a25\n" \
  "v_accvgpr_read_b32 v198, a26\n" \
  "v_accvgpr_read_b32 v199, a27\n" \
  "v_pk_fma_f32 v[200:201], v[192:193], s[48:49], v[226:227] op_sel_hi:[1,0,0]\n" \
  "v_pk_fma_f32 v[202:203], v[194:195], s[48:49], v[226:227] op_sel_hi:[1,0,0]\n" \
  "v_pk_fma_f32 v[204:205], v[196:197], s[48:49], v[226:227] op_sel_hi:[1,0,0]\n" \
  "v_pk_fma_f32 v[206:207], v[198:199], s[48:49], v[226:227] op_sel_hi:[1,0,0]\n" \
  "v_exp_f32 v200, v200 clamp\n" \
  "v_exp_f32 v201, v201 clamp\n" \
  "v_exp_f32 v202, v202 clamp\n" \
  "v_exp_f32 v203, v203 clamp\n" \
  "v_exp_f32 v204, v204 clamp\n" \
  "v_exp_f32 v205, v205 clamp\n" \
  "v_exp_f32 v206, v206 clamp\n" \
  "v_exp_f32 v207, v207 clamp\n" \
  "v_pk_fma_f32 v[200:201], v[200:201], s[50:51], 1.0 op_sel_hi:[1,0,0]\n" \
  "v_pk_fma_f32 v[202:203], v[202:203], s[50:51], 1.0 op_sel_hi:[1,0,0]\n" \
  "v_pk_fma_f32 v[204:205], v[204:205], s[50:51], 1.0 op_sel_hi:[1,0,0]\n" \
  "v_pk_fma_f32 v[206:207], v[206:207], s[50:51], 1.0 op_sel_hi:[1,0,0]\n" \
  "v_rcp_f32 v200, v200\n" \
  "v_rcp_f32 v201, v201\n" \
  "v_rcp_f32 v202, v202\n" \
  "v_rcp_f32 v203, v203\n" \
  "v_rcp_f32 v204, v204\n" \
  "v_rcp_f32 v205, v205\n" \
  "v_rcp_f32 v206, v206\n" \
  "v_rcp_f32 v207, v207\n" \
  "v_pk_mul_f32 v[200:201], v[200:201], v[192:193]\n" \
  "v_pk_mul_f32 v[202:203], v[202:203], v[194:195]\n" \
  "v_pk_mul_f32 v[204:205], v[204:205], v[196:197]\n" \
  "v_pk_mul_f32 v[206:207], v[206:207], v[198:199]\n" \
  "v_cvt_pk_bf16_f32 v192, v200, v201\n" \
  "v_cvt_pk_bf16_f32 v193, v202, v203\n" \
  "v_cvt_pk_bf16_f32 v194, v204, v205\n" \
  "v_cvt_pk_bf16_f32 v195, v206, v207\n" \
  "v_accvgpr_read_b32 v240, a40\n" \
  "v_accvgpr_read_b32 v241, a41\n" \
  "v_accvgpr_read_b32 v242, a42\n" \
  "v_accvgpr_read_b32 v243, a43\n" \
  "v_permlane16_swap_b32 v192, v194\n" \
  "v_permlane16_swap_b32 v193, v195\n" \
  "buffer_store_dwordx4 v[192:195], v215, s[44:47], s63 offen offset:0 sc1 nt\n" \
  "v_accvgpr_read_b32 v244, a56\n" \
  "v_accvgpr_read_b32 v245, a57\n" \
  "v_accvgpr_read_b32 v246, a58\n" \
  "v_accvgpr_read_b32 v247, a59\n" \
  "v_pk_fma_f32 v[248:249], v[240:241], s[48:49], v[226:227] op_sel_hi:[1,0,0]\n" \
  "v_pk_fma_f32 v[250:251], v[242:243], s[48:49], v[226:227] op_sel_hi:[1,0,0]\n" \
  "v_pk_fma_f32 v[252:253], v[244:245], s[48:49], v[226:227] op_sel_hi:[1,0,0]\n" \
  "v_pk_fma_f32 v[254:255], v[246:247], s[48:49], v[226:227] op_sel_hi:[1,0,0]\n" \
  "v_exp_f32 v248, v248 clamp\n" \
  "v_exp_f32 v249, v249 clamp\n" \
  "v_exp_f32 v250, v250 clamp\n" \
  "v_exp_f32 v251, v251 clamp\n" \
  "v_exp_f32 v252, v252 clamp\n" \
  "v_exp_f32 v253, v253 clamp\n" \
  "v_exp_f32 v254, v254 clamp\n" \
  "v_exp_f32 v255, v255 clamp\n" \
  "v_pk_fma_f32 v[248:249], v[248:249], s[50:51], 1.0 op_sel_hi:[1,0,0]\n" \
  "v_pk_fma_f32 v[250:251], v[250:251], s[50:51], 1.0 op_sel_hi:[1,0,0]\n" \
  "v_pk_fma_f32 v[252:253], v[252:253], s[50:51], 1.0 op_sel_hi:[1,0,0]\n" \
  "v_pk_fma_f32 v[254:255], v[254:255], s[50:51], 1.0 op_sel_hi:[1,0,0]\n" \
  "v_rcp_f32 v248, v248\n" \
  "v_rcp_f32 v249, v249\n" \
  "v_rcp_f32 v250, v250\n" \
  "v_rcp_f32 v251, v251\n" \
  "v_rcp_f32 v252, v252\n" \
  "v_rcp_f32 v253, v253\n" \
  "v_rcp_f32 v254, v254\n" \
  "v_rcp_f32 v255, v255\n" \
  "v_pk_mul_f32 v[248:249], v[248:249], v[240:241]\n" \
  "v_pk_mul_f32 v[250:251], v[250:251], v[242:243]\n" \
  "v_pk_mul_f32 v[252:253], v[252:253], v[244:245]\n" \
  "v_pk_mul_f32 v[254:255], v[254:255], v[246:247]\n" \
  "v_cvt_pk_bf16_f32 v240, v248, v249\n" \
  "v_cvt_pk_bf16_f32 v241, v250, v251\n" \
  "v_cvt_pk_bf16_f32 v242, v252, v253\n" \
  "v_cvt_pk_bf16_f32 v243, v254, v255\n" \
  "v_accvgpr_read_b32 v192, a72\n" \
  "v_accvgpr_read_b32 v193, a73\n" \
  "v_accvgpr_read_b32 v194, a74\n" \
  "v_accvgpr_read_b32 v195, a75\n" \
  "v_permlane16_swap_b32 v240, v242\n" \
  "v_permlane16_swap_b32 v241, v243\n" \
  "buffer_store_dwordx4 v[240:243], v215, s[44:47], s63 offen offset:64 sc1 nt\n" \
  "v_accvgpr_read_b32 v196, a88\n" \
  "v_accvgpr_read_b32 v197, a89\n" \
  "v_accvgpr_read_b32 v198, a90\n" \
  "v_accvgpr_read_b32 v199, a91\n" \
  "v_pk_fma_f32 v[200:201], v[192:193], s[48:49], v[226:227] op_sel_hi:[1,0,0]\n" \
  "v_pk_fma_f32 v[202:203], v[194:195], s[48:49], v[226:227] op_sel_hi:[1,0,0]\n" \
  "v_pk_fma_f32 v[204:205], v[196:197], s[48:49], v[226:227] op_sel_hi:[1,0,0]\n" \
  "v_pk_fma_f32 v[206:207], v[198:199], s[48:49], v[226:227] op_sel_hi:[1,0,0]\n" \
  "v_exp_f32 v200, v200 clamp\n" \
  "v_exp_f32 v201, v201 clamp\n" \
  "v_exp_f32 v202, v202 clamp\n" \
  "v_exp_f32 v203, v203 clamp\n" \
  "v_exp_f32 v204, v204 clamp\n" \
  "v_exp_f32 v205, v205 clamp\n" \
  "v_exp_f32 v206, v206 clamp\n" \
  "v_exp_f32 v207, v207 clamp\n" \
  "v_pk_fma_f32 v[200:201], v[200:201], s[50:51], 1.0 op_sel_hi:[1,0,0]\n" \
  "v_pk_fma_f32 v[202:203], v[202:203], s[50:51], 1.0 op_sel_hi:[1,0,0]\n" \
  "v_pk_fma_f32 v[204:205], v[204:205], s[50:51], 1.0 op_sel_hi:[1,0,0]\n" \
  "v_pk_fma_f32 v[206:207], v[206:207], s[50:51], 1.0 op_sel_hi:[1,0,0]\n" \
  "v_rcp_f32 v200, v200\n" \
  "v_rcp_f32 v201, v201\n" \
  "v_rcp_f32 v202, v202\n" \
  "v_rcp_f32 v203, v203\n" \
  "v_rcp_f32 v204, v204\n" \
  "v_rcp_f32 v205, v205\n" \
  "v_rcp_f32 v206, v206\n" \
  "v_rcp_f32 v207, v207\n" \
  "v_pk_mul_f32 v[200:201], v[200:201], v[192:193]\n" \
  "v_pk_mul_f32 v[202:203], v[202:203], v[194:195]\n" \
  "v_pk_mul_f32 v[204:205], v[204:205], v[196:197]\n" \
  "v_pk_mul_f32 v[206:207], v[206:207], v[198:199]\n" \
  "v_mfma_f32_16x16x32_bf16 a[124:127], a[156:159], v[140:143], a[124:127]\n" \
  "v_cvt_pk_bf16_f32 v192, v200, v201\n" \
  "v_cvt_pk_bf16_f32 v193, v202, v203\n" \
  "v_mfma_f32_16x16x32_bf16 a[124:127], a[188:191], v[156:159], a[124:127]\n" \
  "v_cvt_pk_bf16_f32 v194, v204, v205\n" \
  "v_cvt_pk_bf16_f32 v195, v206, v207\n" \
  "v_mfma_f32_16x16x32_bf16 a[108:111], a[184:187], v[156:159], a[108:111]\n" \
  "v_accvgpr_read_b32 v240, a104\n" \
  "v_accvgpr_read_b32 v241, a105\n" \
  "v_mfma_f32_16x16x32_bf16 a[108:111], a[152:155], v[140:143], a[108:111]\n" \
  "v_accvgpr_read_b32 v242, a106\n" \
  "v_accvgpr_read_b32 v243, a107\n" \
  "v_mfma_f32_16x16x32_bf16 a[92:95], a[148:151], v[140:143], a[92:95]\n" \
  "v_permlane16_swap_b32 v192, v194\n" \
  "v_mfma_f32_16x16x32_bf16 a[92:95], a[180:183], v[156:159], a[92:95]\n" \
  "v_permlane16_swap_b32 v193, v195\n" \
  "v_mfma_f32_16x16x32_bf16 a[76:79], a[176:179], v[156:159], a[76:79]\n" \
  "buffer_store_dwordx4 v[192:195], v215, s[44:47], s63 offen offset:128 sc1 nt\n" \
  "v_accvgpr_read_b32 v244, a120\n" \
  "v_mfma_f32_16x16x32_bf16 a[76:79], a[144:147], v[140:143], a[76:79]\n" \
  "v_accvgpr_read_b32 v245, a121\n" \
  "v_accvgpr_read_b32 v246, a122\n" \
  "v_mfma_f32_16x16x32_bf16 a[60:63], a[140:143], v[140:143], a[60:63]\n" \
  "v_accvgpr_read_b32 v247, a123\n" \
  "v_fma_f32 v248, v240, s48, v226\n" \
  "v_mfma_f32_16x16x32_bf16 a[60:63], a[172:175], v[156:159], a[60:63]\n" \
  "v_fma_f32 v249, v241, s48, v226\n" \
  "v_fma_f32 v250, v242, s48, v226\n" \
  "v_mfma_f32_16x16x32_bf16 a[44:47], a[168:171], v[156:159], a[44:47]\n" \
  "v_fma_f32 v251, v243, s48, v226\n" \
  "v_fma_f32 v252, v244, s48, v226\n" \
  "v_mfma_f32_16x16x32_bf16 a[44:47], a[136:139], v[140:143], a[44:47]\n" \
  "v_fma_f32 v253, v245, s48, v226\n" \
  "v_fma_f32 v254, v246, s48, v226\n" \
  "v_mfma_f32_16x16x32_bf16 a[28:31], a[132:135], v[140:143], a[28:31]\n" \
  "v_fma_f32 v255, v247, s48, v226\n" \
  "v_mfma_f32_16x16x32_bf16 a[28:31], a[164:167], v[156:159], a[28:31]\n" \
  "v_exp_f32 v248, v248 clamp\n" \
  "v_mfma_f32_16x16x32_bf16 a[12:15], a[160:163], v[156:159], a[12:15]\n" \
  "v_exp_f32 v249, v249 clamp\n" \
  "v_mfma_f32_16x16x32_bf16 a[12:15], a[128:131], v[140:143], a[12:15]\n" \
  "v_exp_f32 v250, v250 clamp\n" \
  "v_mfma_f32_16x16x32_bf16 a[12:15], a[192:195], v[172:175], a[12:15]\n" \
  "v_exp_f32 v251, v251 clamp\n" \
  "v_mfma_f32_16x16x32_bf16 a[12:15], a[224:227], v[188:191], a[12:15]\n" \
  "v_exp_f32 v252, v252 clamp\n" \
  "v_mfma_f32_16x16x32_bf16 a[28:31], a[228:231], v[188:191], a[28:31]\n" \
  "v_exp_f32 v253, v253 clamp\n" \
  "v_mfma_f32_16x16x32_bf16 a[28:31], a[196:199], v[172:175], a[28:31]\n" \
  "v_exp_f32 v254, v254 clamp\n" \
  "v_mfma_f32_16x16x32_bf16 a[44:47], a[200:203], v[172:175], a[44:47]\n" \
  "v_exp_f32 v255, v255 clamp\n" \
  "v_mfma_f32_16x16x32_bf16 a[44:47], a[232:235], v[188:191], a[44:47]\n" \
  "v_fma_f32 v248, v248, s50, 1.0\n" \
  "v_fma_f32 v249, v249, s50, 1.0\n" \
  "v_mfma_f32_16x16x32_bf16 a[60:63], a[236:239], v[188:191], a[60:63]\n" \
  "v_fma_f32 v250, v250, s50, 1.0\n" \
  "v_fma_f32 v251, v251, s50, 1.0\n" \
  "v_mfma_f32_16x16x32_bf16 a[60:63], a[204:207], v[172:175], a[60:63]\n" \
  "v_fma_f32 v252, v252, s50, 1.0\n" \
  "v_fma_f32 v253, v253, s50, 1.0\n" \
  "v_mfma_f32_16x16x32_bf16 a[76:79], a[208:211], v[172:175], a[76:79]\n" \
  "v_fma_f32 v254, v254, s50, 1.0\n" \
  "v_fma_f32 v255, v255, s50, 1.0\n" \
  "v_mfma_f32_16x16x32_bf16 a[76:79], a[240:243], v[188:191], a[76:79]\n" \
  "v_rcp_f32 v248, v248\n" \
  "v_mfma_f32_16x16x32_bf16 a[92:95], a[244:247], v[188:191], a[92:95]\n" \
  "v_rcp_f32 v249, v249\n" \
  "v_mfma_f32_16x16x32_bf16 a[92:95], a[212:215], v[172:175], a[92:95]\n" \
  "v_rcp_f32 v250, v250\n" \
  "v_mfma_f32_16x16x32_bf16 a[108:111], a[216:219], v[172:175], a[108:111]\n" \
  "v_rcp_f32 v251, v251\n" \
  "v_mfma_f32_16x16x32_bf16 a[108:111], a[248:251], v[188:191], a[108:111]\n" \
  "v_rcp_f32 v252, v252\n" \
  "v_mfma_f32_16x16x32_bf16 a[124:127], a[252:255], v[188:191], a[124:127]\n" \
  "v_rcp_f32 v253, v253\n" \
  "v_mfma_f32_16x16x32_bf16 a[124:127], a[220:223], v[172:175], a[124:127]\n" \
  "v_rcp_f32 v254, v254\n" \
  "v_rcp_f32 v255, v255\n" \
  "v_pk_mul_f32 v[248:249], v[248:249], v[240:241]\n" \
  "v_pk_mul_f32 v[250:251], v[250:251], v[242:243]\n" \
  "v_pk_mul_f32 v[252:253], v[252:253], v[244:245]\n" \
  "v_pk_mul_f32 v[254:255], v[254:255], v[246:247]\n" \
  "v_cvt_pk_bf16_f32 v240, v248, v249\n" \
  "v_cvt_pk_bf16_f32 v241, v250, v251\n" \
  "v_cvt_pk_bf16_f32 v242, v252, v253\n" \
  "v_cvt_pk_bf16_f32 v243, v254, v255\n" \
  "v_accvgpr_read_b32 v192, a12\n" \
  "v_accvgpr_read_b32 v193, a13\n" \
  "v_accvgpr_read_b32 v194, a14\n" \
  "v_accvgpr_read_b32 v195, a15\n" \
  "v_permlane16_swap_b32 v240, v242\n" \
  "v_permlane16_swap_b32 v241, v243\n" \
  "buffer_store_dwordx4 v[240:243], v215, s[44:47], s63 offen offset:192 sc1 nt\n" \
  "s_add_u32 s63, s63, 131072\n" \
  "v_accvgpr_read_b32 v196, a28\n" \
  "v_accvgpr_read_b32 v197, a29\n" \
  "v_accvgpr_read_b32 v198, a30\n" \
  "v_accvgpr_read_b32 v199, a31\n" \
  "v_pk_fma_f32 v[200:201], v[192:193], s[48:49], v[226:227] op_sel_hi:[1,0,0]\n" \
  "v_pk_fma_f32 v[202:203], v[194:195], s[48:49], v[226:227] op_sel_hi:[1,0,0]\n" \
  "v_pk_fma_f32 v[204:205], v[196:197], s[48:49], v[226:227] op_sel_hi:[1,0,0]\n" \
  "v_pk_fma_f32 v[206:207], v[198:199], s[48:49], v[226:227] op_sel_hi:[1,0,0]\n" \
  "v_exp_f32 v200, v200 clamp\n" \
  "v_exp_f32 v201, v201 clamp\n" \
  "v_exp_f32 v202, v202 clamp\n" \
  "v_exp_f32 v203, v203 clamp\n" \
  "v_exp_f32 v204, v204 clamp\n" \
  "v_exp_f32 v205, v205 clamp\n" \
  "v_exp_f32 v206, v206 clamp\n" \
  "v_exp_f32 v207, v207 clamp\n" \
  "v_pk_fma_f32 v[200:201], v[200:201], s[50:51], 1.0 op_sel_hi:[1,0,0]\n" \
  "v_pk_fma_f32 v[202:203], v[202:203], s[50:51], 1.0 op_sel_hi:[1,0,0]\n" \
  "v_pk_fma_f32 v[204:205], v[204:205], s[50:51], 1.0 op_sel_hi:[1,0,0]\n" \
  "v_pk_fma_f32 v[206:207], v[206:207], s[50:51], 1.0 op_sel_hi:[1,0,0]\n" \
  "v_rcp_f32 v200, v200\n" \
  "v_rcp_f32 v201, v201\n" \
  "v_rcp_f32 v202, v202\n" \
  "v_rcp_f32 v203, v203\n" \
  "v_rcp_f32 v204, v204\n" \
  "v_rcp_f32 v205, v205\n" \
  "v_rcp_f32 v206, v206\n" \
  "v_rcp_f32 v207, v207\n" \
  "v_pk_mul_f32 v[200:201], v[200:201], v[192:193]\n" \
  "v_pk_mul_f32 v[202:203], v[202:203], v[194:195]\n" \
  "v_pk_mul_f32 v[204:205], v[204:205], v[196:197]\n" \
  "v_pk_mul_f32 v[206:207], v[206:207], v[198:199]\n" \
  "v_cvt_pk_bf16_f32 v192, v200, v201\n" \
  "v_cvt_pk_bf16_f32 v193, v202, v203\n" \
  "v_cvt_pk_bf16_f32 v194, v204, v205\n" \
  "v_cvt_pk_bf16_f32 v195, v206, v207\n" \
  "v_accvgpr_read_b32 v240, a44\n" \
  "v_accvgpr_read_b32 v241, a45\n" \
  "v_accvgpr_read_b32 v242, a46\n" \
  "v_accvgpr_read_b32 v243, a47\n" \
  "v_permlane16_swap_b32 v192, v194\n" \
  "v_permlane16_swap_b32 v193, v195\n" \
  "buffer_store_dwordx4 v[192:195], v215, s[44:47], s63 offen offset:0 sc1 nt\n" \
  "v_accvgpr_read_b32 v244, a60\n" \
  "v_accvgpr_read_b32 v245, a61\n" \
  "v_accvgpr_read_b32 v246, a62\n" \
  "v_accvgpr_read_b32 v247, a63\n" \
  "v_pk_fma_f32 v[248:249], v[240:241], s[48:49], v[226:227] op_sel_hi:[1,0,0]\n" \
  "v_pk_fma_f32 v[250:251], v[242:243], s[48:49], v[226:227] op_sel_hi:[1,0,0]\n" \
  "v_pk_fma_f32 v[252:253], v[244:245], s[48:49], v[226:227] op_sel_hi:[1,0,0]\n" \
  "v_pk_fma_f32 v[254:255], v[246:247], s[48:49], v[226:227] op_sel_hi:[1,0,0]\n" \
  "v_exp_f32 v248, v248 clamp\n" \
  "v_exp_f32 v249, v249 clamp\n" \
  "v_exp_f32 v250, v250 clamp\n" \
  "v_exp_f32 v251, v251 clamp\n" \
  "v_exp_f32 v252, v252 clamp\n" \
  "v_exp_f32 v253, v253 clamp\n" \
  "v_exp_f32 v254, v254 clamp\n" \
  "v_exp_f32 v255, v255 clamp\n" \
  "v_pk_fma_f32 v[248:249], v[248:249], s[50:51], 1.0 op_sel_hi:[1,0,0]\n" \
  "v_pk_fma_f32 v[250:251], v[250:251], s[50:51], 1.0 op_sel_hi:[1,0,0]\n" \
  "v_pk_fma_f32 v[252:253], v[252:253], s[50:51], 1.0 op_sel_hi:[1,0,0]\n" \
  "v_pk_fma_f32 v[254:255], v[254:255], s[50:51], 1.0 op_sel_hi:[1,0,0]\n" \
  "v_rcp_f32 v248, v248\n" \
  "v_rcp_f32 v249, v249\n" \
  "v_rcp_f32 v250, v250\n" \
  "v_rcp_f32 v251, v251\n" \
  "v_rcp_f32 v252, v252\n" \
  "v_rcp_f32 v253, v253\n" \
  "v_rcp_f32 v254, v254\n" \
  "v_rcp_f32 v255, v255\n" \
  "v_pk_mul_f32 v[248:249], v[248:249], v[240:241]\n" \
  "v_pk_mul_f32 v[250:251], v[250:251], v[242:243]\n" \
  "v_pk_mul_f32 v[252:253], v[252:253], v[244:245]\n" \
  "v_pk_mul_f32 v[254:255], v[254:255], v[246:247]\n" \
  "v_cvt_pk_bf16_f32 v240, v248, v249\n" \
  "v_cvt_pk_bf16_f32 v241, v250, v251\n" \
  "v_cvt_pk_bf16_f32 v242, v252, v253\n" \
  "v_cvt_pk_bf16_f32 v243, v254, v255\n" \
  "v_accvgpr_read_b32 v192, a76\n" \
  "v_accvgpr_read_b32 v193, a77\n" \
  "v_accvgpr_read_b32 v194, a78\n" \
  "v_accvgpr_read_b32 v195, a79\n" \
  "v_permlane16_swap_b32 v240, v242\n" \
  "v_permlane16_swap_b32 v241, v243\n" \
  "buffer_store_dwordx4 v[240:243], v215, s[44:47], s63 offen offset:64 sc1 nt\n" \
  "v_accvgpr_read_b32 v196, a92\n" \
  "v_accvgpr_read_b32 v197, a93\n" \
  "v_accvgpr_read_b32 v198, a94\n" \
  "v_accvgpr_read_b32 v199, a95\n" \
  "v_pk_fma_f32 v[200:201], v[192:193], s[48:49], v[226:227] op_sel_hi:[1,0,0]\n" \
  "v_pk_fma_f32 v[202:203], v[194:195], s[48:49], v[226:227] op_sel_hi:[1,0,0]\n" \
  "v_pk_fma_f32 v[204:205], v[196:197], s[48:49], v[226:227] op_sel_hi:[1,0,0]\n" \
  "v_pk_fma_f32 v[206:207], v[198:199], s[48:49], v[226:227] op_sel_hi:[1,0,0]\n" \
  "v_exp_f32 v200, v200 clamp\n" \
  "v_exp_f32 v201, v201 clamp\n" \
  "v_exp_f32 v202, v202 clamp\n" \
  "v_exp_f32 v203, v203 clamp\n" \
  "v_exp_f32 v204, v204 clamp\n" \
  "v_exp_f32 v205, v205 clamp\n" \
  "v_exp_f32 v206, v206 clamp\n" \
  "v_exp_f32 v207, v207 clamp\n" \
  "v_pk_fma_f32 v[200:201], v[200:201], s[50:51], 1.0 op_sel_hi:[1,0,0]\n" \
  "v_pk_fma_f32 v[202:203], v[202:203], s[50:51], 1.0 op_sel_hi:[1,0,0]\n" \
  "v_pk_fma_f32 v[204:205], v[204:205], s[50:51], 1.0 op_sel_hi:[1,0,0]\n" \
  "v_pk_fma_f32 v[206:207], v[206:207], s[50:51], 1.0 op_sel_hi:[1,0,0]\n" \
  "v_rcp_f32 v200, v200\n" \
  "v_rcp_f32 v201, v201\n" \
  "v_rcp_f32 v202, v202\n" \
  "v_rcp_f32 v203, v203\n" \
  "v_rcp_f32 v204, v204\n" \
  "v_rcp_f32 v205, v205\n" \
  "v_rcp_f32 v206, v206\n" \
  "v_rcp_f32 v207, v207\n" \
  "v_pk_mul_f32 v[200:201], v[200:201], v[192:193]\n" \
  "v_pk_mul_f32 v[202:203], v[202:203], v[194:195]\n" \
  "v_pk_mul_f32 v[204:205], v[204:205], v[196:197]\n" \
  "v_pk_mul_f32 v[206:207], v[206:207], v[198:199]\n" \
  "v_cvt_pk_bf16_f32 v192, v200, v201\n" \
  "v_cvt_pk_bf16_f32 v193, v202, v203\n" \
  "v_cvt_pk_bf16_f32 v194, v204, v205\n" \
  "v_cvt_pk_bf16_f32 v195, v206, v207\n" \
  "v_accvgpr_read_b32 v240, a108\n" \
  "v_accvgpr_read_b32 v241, a109\n" \
  "v_accvgpr_read_b32 v242, a110\n" \
  "v_accvgpr_read_b32 v243, a111\n" \
  "v_permlane16_swap_b32 v192, v194\n" \
  "v_permlane16_swap_b32 v193, v195\n" \
  "buffer_store_dwordx4 v[192:195], v215, s[44:47], s63 offen offset:128 sc1 nt\n" \
  "v_accvgpr_read_b32 v244, a124\n" \
  "v_accvgpr_read_b32 v245, a125\n" \
  "v_accvgpr_read_b32 v246, a126\n" \
  "v_accvgpr_read_b32 v247, a127\n" \
  "v_pk_fma_f32 v[248:249], v[240:241], s[48:49], v[226:227] op_sel_hi:[1,0,0]\n" \
  "v_pk_fma_f32 v[250:251], v[242:243], s[48:49], v[226:227] op_sel_hi:[1,0,0]\n" \
  "v_pk_fma_f32 v[252:253], v[244:245], s[48:49], v[226:227] op_sel_hi:[1,0,0]\n" \
  "v_pk_fma_f32 v[254:255], v[246:247], s[48:49], v[226:227] op_sel_hi:[1,0,0]\n" \
  "v_exp_f32 v248, v248 clamp\n" \
  "v_exp_f32 v249, v249 clamp\n" \
  "v_exp_f32 v250, v250 clamp\n" \
  "v_exp_f32 v251, v251 clamp\n" \
  "v_exp_f32 v252, v252 clamp\n" \
  "v_exp_f32 v253, v253 clamp\n" \
  "v_exp_f32 v254, v254 clamp\n" \
  "v_exp_f32 v255, v255 clamp\n" \
  "v_pk_fma_f32 v[248:249], v[248:249], s[50:51], 1.0 op_sel_hi:[1,0,0]\n" \
  "v_pk_fma_f32 v[250:251], v[250:251], s[50:51], 1.0 op_sel_hi:[1,0,0]\n" \
  "v_pk_fma_f32 v[252:253], v[252:253], s[50:51], 1.0 op_sel_hi:[1,0,0]\n" \
  "v_pk_fma_f32 v[254:255], v[254:255], s[50:51], 1.0 op_sel_hi:[1,0,0]\n" \
  "v_rcp_f32 v248, v248\n" \
  "v_rcp_f32 v249, v249\n" \
  "v_rcp_f32 v250, v250\n" \
  "v_rcp_f32 v251, v251\n" \
  "v_rcp_f32 v252, v252\n" \
  "v_rcp_f32 v253, v253\n" \
  "v_rcp_f32 v254, v254\n" \
  "v_rcp_f32 v255, v255\n" \
  "v_pk_mul_f32 v[248:249], v[248:249], v[240:241]\n" \
  "v_pk_mul_f32 v[250:251], v[250:251], v[242:243]\n" \
  "v_pk_mul_f32 v[252:253], v[252:253], v[244:245]\n" \
  "v_pk_mul_f32 v[254:255], v[254:255], v[246:247]\n" \
  "v_cvt_pk_bf16_f32 v240, v248, v249\n" \
  "v_cvt_pk_bf16_f32 v241, v250, v251\n" \
  "v_cvt_pk_bf16_f32 v242, v252, v253\n" \
  "v_cvt_pk_bf16_f32 v243, v254, v255\n" \
  "s_nop 1\n" \
  "v_permlane16_swap_b32 v240, v242\n" \
  "v_permlane16_swap_b32 v241, v243\n" \
  "buffer_store_dwordx4 v[240:243], v215, s[44:47], s63 offen offset:192 sc1 nt\n" \
  "LK_fin:\n" \
  "s_cmp_lg_u32 s69, 0\n" \
  "s_cbranch_scc1 LK_end1\n" \
  "v_mov_b32 v216, s72\n" \
  "s_mov_b64 exec, 1\n" \
  "v_mov_b32 v217, s30\n" \
  "global_store_dword v217, v216, s[10:11] sc0 sc1\n" \
  "s_mov_b64 exec, -1\n" \
  "LK_end1:\n" \
  "s_branch LK_end\n" \
  "LK_nan:\n" \
  "v_mov_b32 v240, 0x7fc07fc0\n" \
  "v_mov_b32 v241, 0x7fc07fc0\n" \
  "v_mov_b32 v242, 0x7fc07fc0\n" \
  "v_mov_b32 v243, 0x7fc07fc0\n" \
  "s_mov_b32 s68, 8\n" \
  "LK_nanl:\n" \
  "buffer_store_dwordx4 v[240:243], v215, s[44:47], s63 offen offset:0\n" \
  "buffer_store_dwordx4 v[240:243], v215, s[44:47], s63 offen offset:64\n" \
  "buffer_store_dwordx4 v[240:243], v215, s[44:47], s63 offen offset:128\n" \
  "buffer_store_dwordx4 v[240:243], v215, s[44:47], s63 offen offset:192\n" \
  "s_add_u32 s63, s63, 131072\n" \
  "s_sub_u32 s68, s68, 1\n" \
  "s_cmp_lg_u32 s68, 0\n" \
  "s_cbranch_scc1 LK_nanl\n" \
  "s_branch LK_fin\n" \
  "LK_kbad:\n" \
  "s_waitcnt lgkmcnt(0)\n" \
  "s_barrier\n" \
  "LK_redo:\n" \
  "s_waitcnt vmcnt(0) lgkmcnt(0)\n" \
  "s_cmp_lg_u32 s69, 3\n" \
  "s_cbranch_scc1 LK_kw\n" \
  "s_mov_b64 exec, 1\n" \
  "s_mov_b32 s68, 256\n" \
  "v_mov_b32 v131, -1\n" \
  "LK_kn:\n" \
  "global_atomic_add v128, v131, s[10:11] sc1\n" \
  "s_add_u32 %1, %1, 1\n" \
  "s_and_b32 %1, %1, 255\n" \
  "s_lshl_b32 s70, %1, 7\n" \
  "s_add_u32 s70, s70, 134316032\n" \
  "s_waitcnt vmcnt(0)\n" \
  "v_mov_b32 v128, s70\n" \
  "global_atomic_add v130, v128, v129, s[10:11] sc0 sc1\n" \
  "s_waitcnt vmcnt(0)\n" \
  "v_readfirstlane_b32 s70, v130\n" \
  "s_add_u32 s70, s70, 1\n" \
  "s_cmp_eq_u32 s70, s72\n" \
  "s_cbranch_scc1 LK_kf\n" \
  "s_sub_u32 s68, s68, 1\n" \
  "s_cbranch_scc0 LK_kn\n" \
  "LK_kf:\n" \
  "v_mov_b32 v131, %1\n" \
  "ds_write_b32 v132, v131 offset:20\n" \
  "s_mov_b64 exec, -1\n" \
  "s_waitcnt lgkmcnt(0)\n" \
  "LK_kw:\n" \
  "s_barrier\n" \
  "ds_read_b32 v131, v132 offset:20\n" \
  "v_mov_b32 v128, s30\n" \
  "v_mov_b32 v129, 0\n" \
  "s_waitcnt lgkmcnt(0)\n" \
  "v_readfirstlane_b32 %1, v131\n" \
  "s_nop 4\n" \
  "s_branch LK_again\n" \
  "LK_slowplain1:\n" \
  "s_memrealtime s[78:79]\n" \
  "s_add_u32 s82, s82, 1\n" \
  "s_mov_b32 s71, 39\n" \
  "LK_ssplain1:\n" \
  "s_load_dwordx4 s[56:59], s[76:77], 0x0 glc\n" \
  "s_waitcnt lgkmcnt(0)\n" \
  "s_min_u32 s56, s56, s57\n" \
  "s_min_u32 s58, s58, s59\n" \
  "s_min_u32 s56, s56, s58\n" \
  "v_mov_b32 v229, s56\n" \
  "s_cmp_ge_u32 s56, s73\n" \
  "s_cbranch_scc1 LK_retplain1\n" \
  "s_sub_u32 s71, s71, 1\n" \
  "s_cbranch_scc0 LK_ssplain1\n" \
  "LK_spinplain1:\n" \
  "buffer_load_dword v229, v230, s[52:55], 0 offen sc0 sc1\n" \
  "s_waitcnt vmcnt(0)\n" \
  "s_memrealtime s[80:81]\n" \
  "v_cmp_gt_u32 vcc, s73, v229\n" \
  "s_nop 0\n" \
  "s_cbranch_vccz LK_retplain1\n" \
  "s_waitcnt lgkmcnt(0)\n" \
  "s_sub_u32 s70, s80, s78\n" \
  "s_cmp_lt_u32 s70, 100000\n" \
  "s_cbranch_scc1 LK_spinplain1\n" \
  "global_load_dword v220, v219, s[20:21] sc1\n" \
  "s_sleep 8\n" \
  "s_waitcnt vmcnt(0)\n" \
  "v_readfirstlane_b32 s71, v220\n" \
  "s_cmp_eq_u32 s71, s72\n" \
  "s_cbranch_scc1 LK_toplain1\n" \
  "s_cmp_lt_u32 s70, 30000000\n" \
  "s_cbranch_scc1 LK_spinplain1\n" \
  "LK_toplain1:\n" \
  "v_mov_b32 v220, s72\n" \
  "s_mov_b32 s73, 0\n" \
  "s_mov_b32 s74, 0\n" \
  "global_store_dword v219, v220, s[20:21] sc1\n" \
  "s_waitcnt vmcnt(0)\n" \
  "s_branch LK_retplain1\n" \
  "LK_slowplain3:\n" \
  "s_memrealtime s[78:79]\n" \
  "s_add_u32 s82, s82, 1\n" \
  "s_mov_b32 s71, 39\n" \
  "LK_ssplain3:\n" \
  "s_load_dwordx4 s[56:59], s[76:77], 0x0 glc\n" \
  "s_waitcnt lgkmcnt(0)\n" \
  "s_min_u32 s56, s56, s57\n" \
  "s_min_u32 s58, s58, s59\n" \
  "s_min_u32 s56, s56, s58\n" \
  "v_mov_b32 v229, s56\n" \
  "s_cmp_ge_u32 s56, s73\n" \
  "s_cbranch_scc1 LK_retplain3\n" \
  "s_sub_u32 s71, s71, 1\n" \
  "s_cbranch_scc0 LK_ssplain3\n" \
  "LK_spinplain3:\n" \
  "buffer_load_dword v229, v230, s[52:55], 0 offen sc0 sc1\n" \
  "s_waitcnt vmcnt(0)\n" \
  "s_memrealtime s[80:81]\n" \
  "v_cmp_gt_u32 vcc, s73, v229\n" \
  "s_nop 0\n" \
  "s_cbranch_vccz LK_retplain3\n" \
  "s_waitcnt lgkmcnt(0)\n" \
  "s_sub_u32 s70, s80, s78\n" \
  "s_cmp_lt_u32 s70, 100000\n" \
  "s_cbranch_scc1 LK_spinplain3\n" \
  "global_load_dword v220, v219, s[20:21] sc1\n" \
  "s_sleep 8\n" \
  "s_waitcnt vmcnt(0)\n" \
  "v_readfirstlane_b32 s71, v220\n" \
  "s_cmp_eq_u32 s71, s72\n" \
  "s_cbranch_scc1 LK_toplain3\n" \
  "s_cmp_lt_u32 s70, 30000000\n" \
  "s_cbranch_scc1 LK_spinplain3\n" \
  "LK_toplain3:\n" \
  "v_mov_b32 v220, s72\n" \
  "s_mov_b32 s73, 0\n" \
  "s_mov_b32 s74, 0\n" \
  "global_store_dword v219, v220, s[20:21] sc1\n" \
  "s_waitcnt vmcnt(0)\n" \
  "s_branch LK_retplain3\n" \
  "LK_slowplainp0:\n" \
  "s_memrealtime s[78:79]\n" \
  "s_add_u32 s82, s82, 1\n" \
  "s_mov_b32 s71, 59\n" \
  "LK_ssplainp0:\n" \
  "s_load_dwordx4 s[56:59], s[76:77], 0x0 glc\n" \
  "s_load_dwordx4 s[48:51], s[76:77], s12 glc\n" \
  "s_waitcnt lgkmcnt(0)\n" \
  "s_min_u32 s56, s56, s48\n" \
  "s_min_u32 s57, s57, s49\n" \
  "s_min_u32 s58, s58, s50\n" \
  "s_min_u32 s59, s59, s51\n" \
  "s_min_u32 s56, s56, s57\n" \
  "s_min_u32 s58, s58, s59\n" \
  "s_min_u32 s56, s56, s58\n" \
  "v_mov_b32 v229, s56\n" \
  "s_cmp_ge_u32 s56, s73\n" \
  "s_cbranch_scc1 LK_retplainp0\n" \
  "s_sub_u32 s71, s71, 1\n" \
  "s_cbranch_scc0 LK_ssplainp0\n" \
  "LK_spinplainp0:\n" \
  "buffer_load_dword v229, v233, s[52:55], 0 offen sc0 sc1\n" \
  "s_waitcnt vmcnt(0)\n" \
  "s_memrealtime s[80:81]\n" \
  "v_cmp_gt_u32 vcc, s73, v229\n" \
  "s_nop 0\n" \
  "s_cbranch_vccz LK_retplainp0\n" \
  "s_waitcnt lgkmcnt(0)\n" \
  "s_sub_u32 s70, s80, s78\n" \
  "s_cmp_lt_u32 s70, 100000\n" \
  "s_cbranch_scc1 LK_spinplainp0\n" \
  "global_load_dword v220, v219, s[20:21] sc1\n" \
  "s_sleep 8\n" \
  "s_waitcnt vmcnt(0)\n" \
  "v_readfirstlane_b32 s71, v220\n" \
  "s_cmp_eq_u32 s71, s72\n" \
  "s_cbranch_scc1 LK_toplainp0\n" \
  "s_cmp_lt_u32 s70, 30000000\n" \
  "s_cbranch_scc1 LK_spinplainp0\n" \
  "LK_toplainp0:\n" \
  "v_mov_b32 v220, s72\n" \
  "s_mov_b32 s73, 0\n" \
  "s_mov_b32 s74, 0\n" \
  "global_store_dword v219, v220, s[20:21] sc1\n" \
  "s_waitcnt vmcnt(0)\n" \
  "s_branch LK_retplainp0\n" \
  "LK_slowplainp1:\n" \
  "s_memrealtime s[78:79]\n" \
  "s_add_u32 s82, s82, 1\n" \
  "s_mov_b32 s71, 59\n" \
  "LK_ssplainp1:\n" \
  "s_load_dwordx4 s[56:59], s[76:77], 0x0 glc\n" \
  "s_load_dwordx4 s[48:51], s[76:77], s12 glc\n" \
  "s_waitcnt lgkmcnt(0)\n" \
  "s_min_u32 s56, s56, s48\n" \
  "s_min_u32 s57, s57, s49\n" \
  "s_min_u32 s58, s58, s50\n" \
  "s_min_u32 s59, s59, s51\n" \
  "s_min_u32 s56, s56, s57\n" \
  "s_min_u32 s58, s58, s59\n" \
  "s_min_u32 s56, s56, s58\n" \
  "v_mov_b32 v229, s56\n" \
  "s_cmp_ge_u32 s56, s73\n" \
  "s_cbranch_scc1 LK_retplainp1\n" \
  "s_sub_u32 s71, s71, 1\n" \
  "s_cbranch_scc0 LK_ssplainp1\n" \
  "LK_spinplainp1:\n" \
  "buffer_load_dword v229, v233, s[52:55], 0 offen sc0 sc1\n" \
  "s_waitcnt vmcnt(0)\n" \
  "s_memrealtime s[80:81]\n" \
  "v_cmp_gt_u32 vcc, s73, v229\n" \
  "s_nop 0\n" \
  "s_cbranch_vccz LK_retplainp1\n" \
  "s_waitcnt lgkmcnt(0)\n" \
  "s_sub_u32 s70, s80, s78\n" \
  "s_cmp_lt_u32 s70, 100000\n" \
  "s_cbranch_scc1 LK_spinplainp1\n" \
  "global_load_dword v220, v219, s[20:21] sc1\n" \
  "s_sleep 8\n" \
  "s_waitcnt vmcnt(0)\n" \
  "v_readfirstlane_b32 s71, v220\n" \
  "s_cmp_eq_u32 s71, s72\n" \
  "s_cbranch_scc1 LK_toplainp1\n" \
  "s_cmp_lt_u32 s70, 30000000\n" \
  "s_cbranch_scc1 LK_spinplainp1\n" \
  "LK_toplainp1:\n" \
  "v_mov_b32 v220, s72\n" \
  "s_mov_b32 s73, 0\n" \
  "s_mov_b32 s74, 0\n" \
  "global_store_dword v219, v220, s[20:21] sc1\n" \
  "s_waitcnt vmcnt(0)\n" \
  "s_branch LK_retplainp1\n" \
  "LK_slowplainp2:\n" \
  "s_memrealtime s[78:79]\n" \
  "s_add_u32 s82, s82, 1\n" \
  "s_mov_b32 s71, 59\n" \
  "LK_ssplainp2:\n" \
  "s_load_dwordx4 s[56:59], s[76:77], 0x0 glc\n" \
  "s_load_dwordx4 s[48:51], s[76:77], s12 glc\n" \
  "s_waitcnt lgkmcnt(0)\n" \
  "s_min_u32 s56, s56, s48\n" \
  "s_min_u32 s57, s57, s49\n" \
  "s_min_u32 s58, s58, s50\n" \
  "s_min_u32 s59, s59, s51\n" \
  "s_min_u32 s56, s56, s57\n" \
  "s_min_u32 s58, s58, s59\n" \
  "s_min_u32 s56, s56, s58\n" \
  "v_mov_b32 v229, s56\n" \
  "s_cmp_ge_u32 s56, s73\n" \
  "s_cbranch_scc1 LK_retplainp2\n" \
  "s_sub_u32 s71, s71, 1\n" \
  "s_cbranch_scc0 LK_ssplainp2\n" \
  "LK_spinplainp2:\n" \
  "buffer_load_dword v229, v233, s[52:55], 0 offen sc0 sc1\n" \
  "s_waitcnt vmcnt(0)\n" \
  "s_memrealtime s[80:81]\n" \
  "v_cmp_gt_u32 vcc, s73, v229\n" \
  "s_nop 0\n" \
  "s_cbranch_vccz LK_retplainp2\n" \
  "s_waitcnt lgkmcnt(0)\n" \
  "s_sub_u32 s70, s80, s78\n" \
  "s_cmp_lt_u32 s70, 100000\n" \
  "s_cbranch_scc1 LK_spinplainp2\n" \
  "global_load_dword v220, v219, s[20:21] sc1\n" \
  "s_sleep 8\n" \
  "s_waitcnt vmcnt(0)\n" \
  "v_readfirstlane_b32 s71, v220\n" \
  "s_cmp_eq_u32 s71, s72\n" \
  "s_cbranch_scc1 LK_toplainp2\n" \
  "s_cmp_lt_u32 s70, 30000000\n" \
  "s_cbranch_scc1 LK_spinplainp2\n" \
  "LK_toplainp2:\n" \
  "v_mov_b32 v220, s72\n" \
  "s_mov_b32 s73, 0\n" \
  "s_mov_b32 s74, 0\n" \
  "global_store_dword v219, v220, s[20:21] sc1\n" \
  "s_waitcnt vmcnt(0)\n" \
  "s_branch LK_retplainp2\n" \
  "LK_slowplainp3:\n" \
  "s_memrealtime s[78:79]\n" \
  "s_add_u32 s82, s82, 1\n" \
  "s_mov_b32 s71, 59\n" \
  "LK_ssplainp3:\n" \
  "s_load_dwordx4 s[56:59], s[76:77], 0x0 glc\n" \
  "s_waitcnt lgkmcnt(0)\n" \
  "s_min_u32 s56, s56, s57\n" \
  "s_min_u32 s58, s58, s59\n" \
  "s_min_u32 s56, s56, s58\n" \
  "v_mov_b32 v229, s56\n" \
  "s_cmp_ge_u32 s56, s73\n" \
  "s_cbranch_scc1 LK_retplainp3\n" \
  "s_sub_u32 s71, s71, 1\n" \
  "s_cbranch_scc0 LK_ssplainp3\n" \
  "LK_spinplainp3:\n" \
  "buffer_load_dword v229, v230, s[52:55], 0 offen sc0 sc1\n" \
  "s_waitcnt vmcnt(0)\n" \
  "s_memrealtime s[80:81]\n" \
  "v_cmp_gt_u32 vcc, s73, v229\n" \
  "s_nop 0\n" \
  "s_cbranch_vccz LK_retplainp3\n" \
  "s_waitcnt lgkmcnt(0)\n" \
  "s_sub_u32 s70, s80, s78\n" \
  "s_cmp_lt_u32 s70, 100000\n" \
  "s_cbranch_scc1 LK_spinplainp3\n" \
  "global_load_dword v220, v219, s[20:21] sc1\n" \
  "s_sleep 8\n" \
  "s_waitcnt vmcnt(0)\n" \
  "v_readfirstlane_b32 s71, v220\n" \
  "s_cmp_eq_u32 s71, s72\n" \
  "s_cbranch_scc1 LK_toplainp3\n" \
  "s_cmp_lt_u32 s70, 30000000\n" \
  "s_cbranch_scc1 LK_spinplainp3\n" \
  "LK_toplainp3:\n" \
  "v_mov_b32 v220, s72\n" \
  "s_mov_b32 s73, 0\n" \
  "s_mov_b32 s74, 0\n" \
  "global_store_dword v219, v220, s[20:21] sc1\n" \
  "s_waitcnt vmcnt(0)\n" \
  "s_branch LK_retplainp3\n" \
  "LK_slowpush1:\n" \
  "s_memrealtime s[78:79]\n" \
  "s_add_u32 s82, s82, 1\n" \
  "s_mov_b32 s71, 39\n" \
  "LK_sspush1:\n" \
  "s_load_dwordx4 s[56:59], s[76:77], 0x0 glc\n" \
  "s_waitcnt lgkmcnt(0)\n" \
  "s_min_u32 s56, s56, s57\n" \
  "s_min_u32 s58, s58, s59\n" \
  "s_min_u32 s56, s56, s58\n" \
  "v_mov_b32 v229, s56\n" \
  "s_cmp_ge_u32 s56, s73\n" \
  "s_cbranch_scc1 LK_retpush1\n" \
  "s_sub_u32 s71, s71, 1\n" \
  "s_cbranch_scc0 LK_sspush1\n" \
  "LK_spinpush1:\n" \
  "buffer_load_dword v229, v230, s[52:55], 0 offen sc0 sc1\n" \
  "s_waitcnt vmcnt(0)\n" \
  "s_memrealtime s[80:81]\n" \
  "v_cmp_gt_u32 vcc, s73, v229\n" \
  "s_nop 0\n" \
  "s_cbranch_vccz LK_retpush1\n" \
  "s_waitcnt lgkmcnt(0)\n" \
  "s_sub_u32 s70, s80, s78\n" \
  "s_cmp_lt_u32 s70, 100000\n" \
  "s_cbranch_scc1 LK_spinpush1\n" \
  "global_load_dword v220, v219, s[20:21] sc1\n" \
  "s_sleep 8\n" \
  "s_waitcnt vmcnt(0)\n" \
  "v_readfirstlane_b32 s71, v220\n" \
  "s_cmp_eq_u32 s71, s72\n" \
  "s_cbranch_scc1 LK_topush1\n" \
  "s_cmp_lt_u32 s70, 30000000\n" \
  "s_cbranch_scc1 LK_spinpush1\n" \
  "LK_topush1:\n" \
  "v_mov_b32 v220, s72\n" \
  "s_mov_b32 s73, 0\n" \
  "s_mov_b32 s74, 0\n" \
  "global_store_dword v219, v220, s[20:21] sc1\n" \
  "s_waitcnt vmcnt(0)\n" \
  "s_branch LK_retpush1\n" \
  "LK_slowpush3:\n" \
  "s_memrealtime s[78:79]\n" \
  "s_add_u32 s82, s82, 1\n" \
  "s_mov_b32 s71, 39\n" \
  "LK_sspush3:\n" \
  "s_load_dwordx4 s[56:59], s[76:77], 0x0 glc\n" \
  "s_waitcnt lgkmcnt(0)\n" \
  "s_min_u32 s56, s56, s57\n" \
  "s_min_u32 s58, s58, s59\n" \
  "s_min_u32 s56, s56, s58\n" \
  "v_mov_b32 v229, s56\n" \
  "s_cmp_ge_u32 s56, s73\n" \
  "s_cbranch_scc1 LK_retpush3\n" \
  "s_sub_u32 s71, s71, 1\n" \
  "s_cbranch_scc0 LK_sspush3\n" \
  "LK_spinpush3:\n" \
  "buffer_load_dword v229, v230, s[52:55], 0 offen sc0 sc1\n" \
  "s_waitcnt vmcnt(0)\n" \
  "s_memrealtime s[80:81]\n" \
  "v_cmp_gt_u32 vcc, s73, v229\n" \
  "s_nop 0\n" \
  "s_cbranch_vccz LK_retpush3\n" \
  "s_waitcnt lgkmcnt(0)\n" \
  "s_sub_u32 s70, s80, s78\n" \
  "s_cmp_lt_u32 s70, 100000\n" \
  "s_cbranch_scc1 LK_spinpush3\n" \
  "global_load_dword v220, v219, s[20:21] sc1\n" \
  "s_sleep 8\n" \
  "s_waitcnt vmcnt(0)\n" \
  "v_readfirstlane_b32 s71, v220\n" \
  "s_cmp_eq_u32 s71, s72\n" \
  "s_cbranch_scc1 LK_topush3\n" \
  "s_cmp_lt_u32 s70, 30000000\n" \
  "s_cbranch_scc1 LK_spinpush3\n" \
  "LK_topush3:\n" \
  "v_mov_b32 v220, s72\n" \
  "s_mov_b32 s73, 0\n" \
  "s_mov_b32 s74, 0\n" \
  "global_store_dword v219, v220, s[20:21] sc1\n" \
  "s_waitcnt vmcnt(0)\n" \
  "s_branch LK_retpush3\n" \
  "LK_slowpushp3:\n" \
  "s_memrealtime s[78:79]\n" \
  "s_add_u32 s82, s82, 1\n" \
  "s_mov_b32 s71, 39\n" \
  "LK_sspushp3:\n" \
  "s_load_dwordx4 s[56:59], s[76:77], 0x0 glc\n" \
  "s_waitcnt lgkmcnt(0)\n" \
  "s_min_u32 s56, s56, s57\n" \
  "s_min_u32 s58, s58, s59\n" \
  "s_min_u32 s56, s56, s58\n" \
  "v_mov_b32 v229, s56\n" \
  "s_cmp_ge_u32 s56, s73\n" \
  "s_cbranch_scc1 LK_retpushp3\n" \
  "s_sub_u32 s71, s71, 1\n" \
  "s_cbranch_scc0 LK_sspushp3\n" \
  "LK_spinpushp3:\n" \
  "buffer_load_dword v229, v230, s[52:55], 0 offen sc0 sc1\n" \
  "s_waitcnt vmcnt(0)\n" \
  "s_memrealtime s[80:81]\n" \
  "v_cmp_gt_u32 vcc, s73, v229\n" \
  "s_nop 0\n" \
  "s_cbranch_vccz LK_retpushp3\n" \
  "s_waitcnt lgkmcnt(0)\n" \
  "s_sub_u32 s70, s80, s78\n" \
  "s_cmp_lt_u32 s70, 100000\n" \
  "s_cbranch_scc1 LK_spinpushp3\n" \
  "global_load_dword v220, v219, s[20:21] sc1\n" \
  "s_sleep 8\n" \
  "s_waitcnt vmcnt(0)\n" \
  "v_readfirstlane_b32 s71, v220\n" \
  "s_cmp_eq_u32 s71, s72\n" \
  "s_cbranch_scc1 LK_topushp3\n" \
  "s_cmp_lt_u32 s70, 30000000\n" \
  "s_cbranch_scc1 LK_spinpushp3\n" \
  "LK_topushp3:\n" \
  "v_mov_b32 v220, s72\n" \
  "s_mov_b32 s73, 0\n" \
  "s_mov_b32 s74, 0\n" \
  "global_store_dword v219, v220, s[20:21] sc1\n" \
  "s_waitcnt vmcnt(0)\n" \
  "s_branch LK_retpushp3\n" \
  "LK_end:\n" \
  ""
#define K_CLOB "v0","v1","v2","v3","v4","v5","v6","v7","v8","v9","v10","v11","v12","v13","v14","v15","v16","v17","v18","v19","v20","v21","v22","v23","v24","v25","v26","v27","v28","v29","v30","v31","v32","v33","v34","v35","v36","v37","v38","v39","v40","v41","v42","v43","v44","v45","v46","v47","v48","v49","v50","v51","v52","v53","v54","v55","v56","v57","v58","v59","v60","v61","v62","v63","v64","v65","v66","v67","v68","v69","v70","v71","v72","v73","v74","v75","v76","v77","v78","v79","v80","v81","v82","v83","v84","v85","v86","v87","v88","v89","v90","v91","v92","v93","v94","v95","v96","v97","v98","v99","v100","v101","v102","v103","v104","v105","v106","v107","v108","v109","v110","v111","v112","v113","v114","v115","v116","v117","v118","v119","v120","v121","v122","v123","v124","v125","v126","v127","v128","v129","v130","v131","v132","v133","v134","v135","v136","v137","v138","v139","v140","v141","v142","v143","v144","v145","v146","v147","v148","v149","v150","v151","v152","v153","v154","v155","v156","v157","v158","v159","v160","v161","v162","v163","v164","v165","v166","v167","v168","v169","v170","v171","v172","v173","v174","v175","v176","v177","v178","v179","v180","v181","v182","v183","v184","v185","v186","v187","v188","v189","v190","v191","v192","v193","v194","v195","v196","v197","v198","v199","v200","v201","v202","v203","v204","v205","v206","v207","v208","v209","v210","v211","v212","v213","v214","v215","v216","v217","v218","v219","v220","v221","v222","v223","v224","v225","v226","v227","v228","v229","v230","v231","v232","v233","v234","v235","v236","v237","v238","v240","v241","v242","v243","v244","v245","v246","v247","v248","v249","v250","v251","v252","v253","v254","v255","a0","a1","a2","a3","a4","a5","a6","a7","a8","a9","a10","a11","a12","a13","a14","a15","a16","a17","a18","a19","a20","a21","a22","a23","a24","a25","a26","a27","a28","a29","a30","a31","a32","a33","a34","a35","a36","a37","a38","a39","a40","a41","a42","a43","a44","a45","a46","a47","a48","a49","a50","a51","a52","a53","a54","a55","a56","a57","a58","a59","a60","a61","a62","a63","a64","a65","a66","a67","a68","a69","a70","a71","a72","a73","a74","a75","a76","a77","a78","a79","a80","a81","a82","a83","a84","a85","a86","a87","a88","a89","a90","a91","a92","a93","a94","a95","a96","a97","a98","a99","a100","a101","a102","a103","a104","a105","a106","a107","a108","a109","a110","a111","a112","a113","a114","a115","a116","a117","a118","a119","a120","a121","a122","a123","a124","a125","a126","a127","a128","a129","a130","a131","a132","a133","a134","a135","a136","a137","a138","a139","a140","a141","a142","a143","a144","a145","a146","a147","a148","a149","a150","a151","a152","a153","a154","a155","a156","a157","a158","a159","a160","a161","a162","a163","a164","a165","a166","a167","a168","a169","a170","a171","a172","a173","a174","a175","a176","a177","a178","a179","a180","a181","a182","a183","a184","a185","a186","a187","a188","a189","a190","a191","a192","a193","a194","a195","a196","a197","a198","a199","a200","a201","a202","a203","a204","a205","a206","a207","a208","a209","a210","a211","a212","a213","a214","a215","a216","a217","a218","a219","a220","a221","a222","a223","a224","a225","a226","a227","a228","a229","a230","a231","a232","a233","a234","a235","a236","a237","a238","a239","a240","a241","a242","a243","a244","a245","a246","a247","a248","a249","a250","a251","a252","a253","a254","a255","s4","s5","s6","s7","s8","s9","s10","s11","s12","s13","s14","s15","s16","s17","s18","s19","s20","s21","s22","s23","s24","s25","s26","s27","s28","s29","s30","s31","s34","s35","s36","s37","s38","s39","s40","s41","s42","s43","s44","s45","s46","s47","s48","s49","s50","s51","s52","s53","s54","s55","s56","s57","s58","s59","s60","s61","s62","s63","s64","s65","s66","s67","s68","s69","s70","s71","s72","s73","s74","s75","s76","s77","s78","s79","s80","s81","s82","s83","vcc","scc","memory"


namespace {
constexpr size_t WS_NEED = (size_t)K_FLG_OFF + 32 * (size_t)K_FLG_PANEL;
constexpr int LDS_BYTES = 131072 + 64;
struct Peer { char* data; char* flag; };
struct Args {
  const char* x; const char* w; char* out; char* ws;
  unsigned rank, wbytes; unsigned pad[6];
  Peer peer[8];
};
}

__global__ __launch_bounds__(256) __attribute__((amdgpu_waves_per_eu(1, 1)))
void fused_ag_gemm_silu(Args) {
  asm volatile(K_ASM :: "v"(threadIdx.x), "s"(blockIdx.x), "s"(__builtin_amdgcn_kernarg_segment_ptr()) : K_CLOB);
}

extern "C" void kernel_launch(int rank, int nranks, ncclComm_t comm, hipStream_t stream,
                              void* const* d_in, const int* in_sizes, int n_in,
                              void* d_out, int out_size,
                              const int* in_dtypes, int out_dtype,
                              void* const* d_out_all,
                              void* const* d_ws_all, size_t ws_bytes) {
  if (nranks != 8 || (unsigned)rank >= 8u || ws_bytes < WS_NEED) { fprintf(stderr, "unsupported configuration\n"); return; }
  hipError_t e = hipFuncSetAttribute((const void*)fused_ag_gemm_silu, hipFuncAttributeMaxDynamicSharedMemorySize, LDS_BYTES);
  if (e != hipSuccess) { fprintf(stderr, "HIP error %s\n", hipGetErrorString(e)); return; }
  Args a = {};
  a.x = (const char*)d_in[0];
  a.w = (const char*)d_in[1] + (size_t)rank * 8192 - 2688;
  a.out = (char*)d_out;
  a.ws = (char*)d_ws_all[rank];
  a.rank = (unsigned)rank;
  a.wbytes = 4096u * 65536u - (unsigned)rank * 8192u + 2688u;
  const int grp[8] = {0, 1, 0, 0, 1, 1, 0, 1};
  int n0 = 0, n1 = 4;
  for (int p = 0; p < 8; ++p) {
    const int k = (p == rank ? 3 : grp[p] == grp[rank] ? n0++ : n1++) ^ K_MAP;
    a.peer[k].data = (char*)d_ws_all[p] + (size_t)(2 * rank) * K_PANEL_STRIDE;
    if (K_FIN) { a.peer[k].flag = a.peer[k].data; a.peer[k].data += 16384; } else
    a.peer[k].flag = (char*)d_ws_all[p] + K_FLG_OFF + (size_t)((p == rank ? 16 : 0) + 2 * rank) * K_FLG_PANEL;
  }
  hipLaunchKernelGGL(fused_ag_gemm_silu, dim3(256), dim3(256), LDS_BYTES, stream, a);
}
```
